```python
import numpy as np

kernel_name = "gpu_ag_gemm_m4096_k4096_n4096_f32_silu"


ABSMAX_RTOL = 0.001

COMPUTE_DTYPE = "float32"
EPILOGUE = "silu"
IN_QUANT = None
OUT_QUANT = None
IN_DTYPES = {"x": "float32", "w_mat": "float32"}
WORLD_SIZE = 8

def setup_inputs():

    x = np.random.randn(4096, 4096).astype(np.float32)
    w_mat = (np.random.randn(4096, 4096) / np.sqrt(4096)).astype(np.float32)
    return {"x": x, "w_mat": w_mat}

def _epilogue(y):
    return (y / (1.0 + np.exp(-np.clip(y, -60.0, 60.0)))).astype(np.float32)

def reference(x, w_mat):
    y = (x @ w_mat).astype(np.float32)
    return _epilogue(y)
```

```cpp
#include <hip/hip_runtime.h>
#include <rccl/rccl.h>
#include <cstdio>

#define HIP_CHECK(expr) do { hipError_t _e = (expr); if (_e != hipSuccess) { \
    fprintf(stderr, "HIP error %s at %s:%d\n", hipGetErrorString(_e), __FILE__, __LINE__); \
    return; } } while (0)

namespace agk {

typedef unsigned u32x4 __attribute__((ext_vector_type(4)));
typedef float f32x4 __attribute__((ext_vector_type(4)));
typedef float f32x16 __attribute__((ext_vector_type(16)));
typedef _Float16 f16x8 __attribute__((ext_vector_type(8)));
typedef __attribute__((address_space(1))) unsigned gu32;
typedef __amdgpu_buffer_rsrc_t rsrc_t;

constexpr int NR = 8, MS = 512, NS = 512, KD = 4096;
constexpr int BLK = 32 * KD * 2;
constexpr int A_BYTES = 64 * BLK, B_BYTES = 32 * BLK, BUF_BYTES = A_BYTES + B_BYTES;
constexpr int CNT_OFF = 0;
constexpr int TMO_OFF = 32768;
constexpr int FA_OFF = 65536;
constexpr int FB_OFF = FA_OFF + 64 * 256;
constexpr int FD_OFF = FB_OFF + 8 * 256;
constexpr int CTRL_BYTES = 1 << 20;
constexpr int WS_USED = CTRL_BYTES + 2 * BUF_BYTES;
constexpr int OUT_BYTES = 4096 * NS * 4;
constexpr int IN_BYTES = MS * KD * 4;
constexpr int AUX_SYS = 17;
constexpr int RSRC_W3 = 0x00020000;
constexpr unsigned long long DEADLINE = 30000000ull;
constexpr int NTILES_REMOTE = 38;
constexpr int LDS_BYTES = 4 * 128 * 68 * 4;

#define PW_I 2
#define PW_C 2

struct Params {
  const float* x;
  const float* w;
  void* out[NR];
  void* ws[NR];
  int rank;
  int mode;
};

#define STAMP(ws_, idx_, val_) do {} while (0)
#define NOW() 0ull

__device__ __forceinline__ rsrc_t mk(const void* p, int bytes) {
  const unsigned long long a = (unsigned long long)p;
  const unsigned lo = __builtin_amdgcn_readfirstlane((unsigned)a), hi = __builtin_amdgcn_readfirstlane((unsigned)(a >> 32));
  return __builtin_amdgcn_make_buffer_rsrc((void*)(((unsigned long long)hi << 32) | lo), 0, bytes, RSRC_W3);
}
__device__ __forceinline__ unsigned ld_sys(const void* base, int off) {
  return __hip_atomic_load((gu32*)((char*)base + off), __ATOMIC_RELAXED, __HIP_MEMORY_SCOPE_SYSTEM);
}
__device__ __forceinline__ void st_sys(void* base, int off, unsigned v) {
  __hip_atomic_store((gu32*)((char*)base + off), v, __ATOMIC_RELAXED, __HIP_MEMORY_SCOPE_SYSTEM);
}
__device__ __forceinline__ unsigned ld_agent(const void* base, int off) {
  return __hip_atomic_load((gu32*)((char*)base + off), __ATOMIC_RELAXED, __HIP_MEMORY_SCOPE_AGENT);
}
__device__ __forceinline__ void st_agent(void* base, int off, unsigned v) {
  __hip_atomic_store((gu32*)((char*)base + off), v, __ATOMIC_RELAXED, __HIP_MEMORY_SCOPE_AGENT);
}
#define DRAIN() asm volatile("s_waitcnt vmcnt(0)" ::: "memory")

__device__ __forceinline__ u32x4 cvt8(f32x4 a, f32x4 b) {
  f16x8 h;
  h[0] = (_Float16)a[0]; h[1] = (_Float16)a[1]; h[2] = (_Float16)a[2]; h[3] = (_Float16)a[3];
  h[4] = (_Float16)b[0]; h[5] = (_Float16)b[1]; h[6] = (_Float16)b[2]; h[7] = (_Float16)b[3];
  return __builtin_bit_cast(u32x4, h);
}

__device__ __forceinline__ void x_load(f32x4 (&v)[8], rsrc_t rx, int lane, int b, int u) {
  const int voff = (lane & 31) * (KD * 4) + (lane >> 5) * 32;
  const int soff = b * (32 * KD * 4) + u * 256;
#pragma unroll
  for (int s = 0; s < 4; ++s) {
    v[2 * s] = __builtin_bit_cast(f32x4, __builtin_amdgcn_raw_buffer_load_b128(rx, voff + s * 64, soff, 0));
    v[2 * s + 1] = __builtin_bit_cast(f32x4, __builtin_amdgcn_raw_buffer_load_b128(rx, voff + s * 64 + 16, soff, 0));
  }
}

__device__ __forceinline__ void x_stream(const Params& p, unsigned call, int lane, int first, int step, int npieces,
                                         int blk_base, const int (&dst)[4], const int (&slot0)[4], const int (&wstride)[4],
                                         f32x4 (&v)[8], rsrc_t rx, int stamp_base) {
  rsrc_t rd[4];
#pragma unroll
  for (int i = 0; i < 4; ++i) rd[i] = mk(p.ws[dst[i]], WS_USED);
  char* fbase = (char*)(lane == 0 ? p.ws[dst[0]] : lane == 1 ? p.ws[dst[1]] : lane == 2 ? p.ws[dst[2]] : p.ws[dst[3]]);
  const int fslot0 = lane == 0 ? slot0[0] : lane == 1 ? slot0[1] : lane == 2 ? slot0[2] : slot0[3];
  const int fws = lane == 0 ? wstride[0] : lane == 1 ? wstride[1] : lane == 2 ? wstride[2] : wstride[3];
  const int abase = CTRL_BYTES + (call & 1) * BUF_BYTES;
  for (int pc = first; pc < npieces; pc += step) {
    const int win = pc >> 8, u = (pc >> 2) & 63, bb = pc & 3;
    u32x4 f[4];
#pragma unroll
    for (int s = 0; s < 4; ++s) f[s] = cvt8(v[2 * s], v[2 * s + 1]);
    __builtin_amdgcn_sched_barrier(0);
    {
      int nx = pc + step; if (nx >= npieces) nx = pc;
      x_load(v, rx, lane, blk_base + (nx >> 8) * 4 + (nx & 3), (nx >> 2) & 63);
    }
    STAMP(p.ws[p.rank], stamp_base + 2 * pc, NOW());
#pragma unroll
    for (int i = 0; i < 4; ++i) {
      const int soff = abase + (slot0[i] + win * wstride[i] + bb) * BLK + u * 4096;
#pragma unroll
      for (int s = 0; s < 4; ++s)
        __builtin_amdgcn_raw_buffer_store_b128(f[s], rd[i], lane * 16 + s * 1024, soff, AUX_SYS);
    }
    DRAIN();
    if (lane < 4) st_sys(fbase, FA_OFF + (fslot0 + win * fws + bb) * 256 + (u & 3) * 64 + (u >> 2) * 4, call);
    STAMP(p.ws[p.rank], stamp_base + 2 * pc + 1, NOW());
  }
}

__device__ __forceinline__ void w_load(f32x4 (&v)[32], rsrc_t rw, int lane, int q, int u) {
  const int voff = (lane >> 5) * (8 * NS * 4) + (lane & 31) * 16;
  const int soff = u * (64 * NS * 4) + q * 512;
#pragma unroll
  for (int s = 0; s < 4; ++s)
#pragma unroll
    for (int j = 0; j < 8; ++j)
      v[s * 8 + j] = __builtin_bit_cast(f32x4, __builtin_amdgcn_raw_buffer_load_b128(
          rw, voff + (j & 1) * (NS * 4), soff + (s * 16 + (j & 6)) * (NS * 4), 0));
}

__device__ __forceinline__ float silu(float y) {
  const float t = fminf(fmaxf(y, -60.f), 60.f);
  return y * __builtin_amdgcn_rcpf(1.f + __expf(-t));
}

__device__ __forceinline__ int wait_ready(void* ws, int offa, int offb, unsigned call, int need) {
  unsigned long long t0 = __builtin_amdgcn_s_memrealtime();
  for (;;) {
    const unsigned fa = ld_sys(ws, offa), fb = ld_sys(ws, offb);
    const unsigned long long m = __ballot(fa >= call && fb >= call);
    const unsigned r = (unsigned)(m & (m >> 16) & (m >> 32) & (m >> 48)) & 0xffffu;
    const int n = __builtin_ctz(~r);
    if (n > need) { asm volatile("" ::: "memory"); return n; }
    if (ld_agent(ws, TMO_OFF) == call || __builtin_amdgcn_s_memrealtime() - t0 > DEADLINE) {
      st_agent(ws, TMO_OFF, call);
      return 17;
    }
    __builtin_amdgcn_s_sleep(8);
  }
}

template <int MB, int NB>
__device__ __forceinline__ void gemm_tile(const Params& p, unsigned call, float* lds, int slot0, int qslot, int nt,
                                          int tile_id, int stamp_base) {
  const int lane = threadIdx.x & 63;
  const int w = __builtin_amdgcn_readfirstlane(threadIdx.x >> 6);
  void* ws = p.ws[p.rank];
  const rsrc_t rs = mk(ws, WS_USED);
  const int abase = CTRL_BYTES + (call & 1) * BUF_BYTES + slot0 * BLK + w * 4096;
  const int bbase = CTRL_BYTES + (call & 1) * BUF_BYTES + A_BYTES + (qslot * 4 + nt * NB) * BLK + w * 4096;
  const int offa = FA_OFF + (slot0 + ((lane >> 4) & (MB - 1))) * 256 + w * 64 + (lane & 15) * 4;
  const int offb = FB_OFF + qslot * 256 + w * 64 + (lane & 15) * 4;
  const int voff = lane * 16;

  f32x16 acc[MB][NB];
#pragma unroll
  for (int m = 0; m < MB; ++m)
#pragma unroll
    for (int n = 0; n < NB; ++n)
#pragma unroll
      for (int r = 0; r < 16; ++r) acc[m][n][r] = 0.f;
  u32x4 fa0[4][MB], fb0[4][NB], fa1[4][MB], fb1[4][NB];

#define ISSUE(FA, FB, kc) do { \
    _Pragma("unroll") for (int s = 0; s < 4; ++s) { \
      _Pragma("unroll") for (int n = 0; n < NB; ++n) \
        FB[s][n] = __builtin_amdgcn_raw_buffer_load_b128(rs, voff + s * 1024, bbase + n * BLK + (kc) * 16384, AUX_SYS); \
      _Pragma("unroll") for (int m = 0; m < MB; ++m) \
        FA[s][m] = __builtin_amdgcn_raw_buffer_load_b128(rs, voff + s * 1024, abase + m * BLK + (kc) * 16384, AUX_SYS); \
    } \
    __builtin_amdgcn_sched_barrier(0); } while (0)
#define COMPUTE(FA, FB) do { \
    _Pragma("unroll") for (int s = 0; s < 4; ++s) \
      _Pragma("unroll") for (int m = 0; m < MB; ++m) \
        _Pragma("unroll") for (int n = 0; n < NB; ++n) \
          acc[m][n] = __builtin_amdgcn_mfma_f32_32x32x16_f16(__builtin_bit_cast(f16x8, FB[s][n]), \
                                                             __builtin_bit_cast(f16x8, FA[s][m]), acc[m][n], 0, 0, 0); \
    __builtin_amdgcn_sched_barrier(0); } while (0)
#define READY(kc) do { if (nready <= (kc)) nready = wait_ready(ws, offa, offb, call, (kc)); \
    STAMP(ws, stamp_base + w * 32 + (kc), NOW()); } while (0)

  int nready = 0;
  STAMP(ws, stamp_base + w * 32 + 16, NOW());
  READY(0); ISSUE(fa0, fb0, 0);
  READY(1); ISSUE(fa1, fb1, 1);
  for (int kc = 0; kc < 14; kc += 2) {
    COMPUTE(fa0, fb0);
    READY(kc + 2); ISSUE(fa0, fb0, kc + 2);
    COMPUTE(fa1, fb1);
    READY(kc + 3); ISSUE(fa1, fb1, kc + 3);
  }
  COMPUTE(fa0, fb0);
  COMPUTE(fa1, fb1);
#undef ISSUE
#undef COMPUTE
#undef READY
  STAMP(ws, stamp_base + w * 32 + 17, NOW());

  constexpr int LD = NB * 32 + 4;
  {
    const bool bad = nready > 16 || ld_agent(ws, TMO_OFF) == call;
    float* dst = lds + w * (MB * 32 * LD) + (lane & 31) * LD + (lane >> 5) * 16;
#pragma unroll
    for (int m = 0; m < MB; ++m)
#pragma unroll
      for (int n = 0; n < NB; ++n)
#pragma unroll
        for (int t = 0; t < 4; ++t) {
          f32x4 o = {acc[m][n][t], acc[m][n][4 + t], acc[m][n][8 + t], acc[m][n][12 + t]};
          *(f32x4*)(dst + m * 32 * LD + n * 32 + t * 4) = o;
        }
    if (bad) {
      const float q = __builtin_nanf("");
      for (int m = 0; m < MB; ++m)
        for (int i = 0; i < NB * 4; ++i) *(f32x4*)(dst + m * 32 * LD + (i >> 2) * 32 + (i & 3) * 4) = f32x4{q, q, q, q};
    }
  }
  __syncthreads();
  const int grp = p.rank >> 2, jme = p.rank & 3;
  const bool remote = qslot >= 4;
  const int owner = remote ? ((grp ^ 1) * 4 + (qslot - 4)) : p.rank;
  const int col0 = (remote ? jme : qslot) * 128 + nt * NB * 32;
  const rsrc_t ro = mk(p.out[owner], OUT_BYTES);
  constexpr int LPR = NB * 8;
  constexpr int RPI = 64 / LPR;
  constexpr int RPW = MB * 8;
#pragma unroll
  for (int i = 0; i < RPW / RPI; ++i) {
    const int row = w * RPW + i * RPI + lane / LPR, c4 = (lane % LPR) * 4;
    f32x4 s = *(const f32x4*)(lds + row * LD + c4);
#pragma unroll
    for (int k = 1; k < 4; ++k) s += *(const f32x4*)(lds + k * (MB * 32 * LD) + row * LD + c4);
    f32x4 o;
#pragma unroll
    for (int e = 0; e < 4; ++e) o[e] = silu(s[e]);
    const int slot = slot0 + ((w * RPW + i * RPI) >> 5);
    int src, b;
    if (slot < 12) { src = p.rank; b = slot; }
    else if (slot < 48) { const int t = slot - 12, win = t / 12, r = t - win * 12; src = grp * 4 + ((jme + 1 + (r >> 2)) & 3); b = win * 4 + (r & 3); }
    else { src = (grp ^ 1) * 4 + ((slot - 48) >> 2); b = 12 + (slot & 3); }
    const int grow = src * MS + b * 32;
    __builtin_amdgcn_raw_buffer_store_b128(__builtin_bit_cast(u32x4, o), ro,
                                           ((grow + (row & 31)) * NS + col0 + c4) * 4, 0, AUX_SYS);
  }
  DRAIN();
  __syncthreads();
  if (remote && threadIdx.x == 0) st_sys(p.ws[owner], FD_OFF + jme * 256 + tile_id * 4, call);
  STAMP(ws, stamp_base + w * 32 + 18, NOW());
}

__global__ void __launch_bounds__(256, 1) ag_gemm(Params p) {
  extern __shared__ __attribute__((aligned(16))) float lds[];
  const int lane = threadIdx.x & 63;
  const int w = __builtin_amdgcn_readfirstlane(threadIdx.x >> 6);
  const int xcd = blockIdx.x & 7, idx = blockIdx.x >> 3;
  const int grp = p.rank >> 2, jme = p.rank & 3;
  void* ws = p.ws[p.rank];
  STAMP(ws, 65536 + blockIdx.x * 4 + w, NOW());

  if (idx < 3) {
    if (w < PW_I) {
      const rsrc_t rx = mk(p.x, IN_BYTES);
      const int first = (idx * 8 + xcd) * PW_I + w, step = 24 * PW_I;
      f32x4 v[8];
      x_load(v, rx, lane, (first >> 8) * 4 + (first & 3), (first >> 2) & 63);
      const unsigned call = __builtin_amdgcn_readfirstlane(ld_agent(ws, CNT_OFF + blockIdx.x * 128 + w * 4)) + 1;
      if (lane == 0) st_agent(ws, CNT_OFF + blockIdx.x * 128 + w * 4, call);
      int dst[4], slot0[4], wstr[4];
#pragma unroll
      for (int i = 0; i < 3; ++i) {
        dst[i] = grp * 4 + ((jme + 1 + i) & 3); slot0[i] = 12 + (2 - i) * 4; wstr[i] = 12;
      }
      dst[3] = p.rank; slot0[3] = 0; wstr[3] = 4;
      x_stream(p, call, lane, first, step, 768, 0, dst, slot0, wstr, v, rx, 0);
    }
  } else if (idx < 6) {
    const unsigned call = __builtin_amdgcn_readfirstlane(ld_agent(ws, CNT_OFF + blockIdx.x * 128 + w * 4)) + 1;
    if (lane == 0) st_agent(ws, CNT_OFF + blockIdx.x * 128 + w * 4, call);
    if (w < PW_C) {
      const rsrc_t rx = mk(p.x, IN_BYTES), rw = mk(p.w, IN_BYTES);
      const int first = ((idx - 3) * 8 + xcd) * PW_C + w, step = 24 * PW_C;
      const int q = first & 3;
      const int peer = (grp ^ 1) * 4 + q;
      f32x4 xv[8];
      {
        f32x4 v[32];
        w_load(v, rw, lane, q, first >> 2);
        const rsrc_t rl = mk(ws, WS_USED), rp = mk(p.ws[peer], WS_USED);
        char* fbase = (char*)(lane == 0 ? ws : p.ws[peer]);
        const int fq = lane == 0 ? q : 4 + jme;
        const int bbase = CTRL_BYTES + (call & 1) * BUF_BYTES + A_BYTES;
        const int voff = ((lane & 31) >> 3) * BLK + ((lane >> 5) * 32 + (lane & 7)) * 16;
        for (int pc = first; pc < 256; pc += step) {
          const int u = pc >> 2;
          u32x4 f[16];
#pragma unroll
          for (int s = 0; s < 4; ++s)
#pragma unroll
            for (int e = 0; e < 4; ++e) {
              f32x4 a = {v[s * 8][e], v[s * 8 + 1][e], v[s * 8 + 2][e], v[s * 8 + 3][e]};
              f32x4 b = {v[s * 8 + 4][e], v[s * 8 + 5][e], v[s * 8 + 6][e], v[s * 8 + 7][e]};
              f[s * 4 + e] = cvt8(a, b);
            }
          __builtin_amdgcn_sched_barrier(0);
          if (pc + step < 256) w_load(v, rw, lane, q, (pc + step) >> 2);
          else x_load(xv, rx, lane, 12 + (first & 3), (first >> 2) & 63);
          STAMP(ws, 2048 + 2 * pc, NOW());
#pragma unroll
          for (int i = 0; i < 16; ++i)
            __builtin_amdgcn_raw_buffer_store_b128(f[i], rp, voff + (i >> 2) * 1024 + (i & 3) * 128,
                                                   bbase + (4 + jme) * 4 * BLK + u * 4096, AUX_SYS);
#pragma unroll
          for (int i = 0; i < 16; ++i)
            __builtin_amdgcn_raw_buffer_store_b128(f[i], rl, voff + (i >> 2) * 1024 + (i & 3) * 128,
                                                   bbase + q * 4 * BLK + u * 4096, AUX_SYS);
          DRAIN();
          if (lane < 2) st_sys(fbase, FB_OFF + fq * 256 + (u & 3) * 64 + (u >> 2) * 4, call);
          STAMP(ws, 2048 + 2 * pc + 1, NOW());
        }
      }
      int dst[4], slot0[4], wstr[4];
#pragma unroll
      for (int i = 0; i < 4; ++i) { dst[i] = (grp ^ 1) * 4 + ((jme + i) & 3); slot0[i] = 48 + jme * 4; wstr[i] = 0; }
      x_stream(p, call, lane, first, step, 256, 12, dst, slot0, wstr, xv, rx, 3072);
    }
    if (blockIdx.x == 24 && !(p.mode & 1)) {
      unsigned long long t0 = __builtin_amdgcn_s_memrealtime();
      for (;;) {
        const unsigned f = ld_sys(ws, FD_OFF + w * 256 + lane * 4);
        if (__all(f >= call || lane >= NTILES_REMOTE)) break;
        if (ld_agent(ws, TMO_OFF) == call || __builtin_amdgcn_s_memrealtime() - t0 > DEADLINE) {
          st_agent(ws, TMO_OFF, call);
          if (lane == 0) ((float*)p.out[p.rank])[((grp ^ 1) * 4 * MS) * NS + w * 128] = __builtin_nanf("");
          break;
        }
        __builtin_amdgcn_s_sleep(4);
      }
      STAMP(ws, 65536 + 2048 + w, NOW());
    }
  } else {
    const unsigned call = __builtin_amdgcn_readfirstlane(ld_agent(ws, CNT_OFF + blockIdx.x * 128 + w * 4)) + 1;
    if (lane == 0) st_agent(ws, CNT_OFF + blockIdx.x * 128 + w * 4, call);
    if (!(p.mode & 1)) {
      const int t = idx < 18 ? idx - 6 : idx - 18;
      const int mt = t >> 1, nt = t & 1;
      const int slot0 = idx < 18 ? mt * 4 : (mt < 3 ? 24 + mt * 4 : 36 + mt * 4);
      gemm_tile<4, 2>(p, call, lds, slot0, xcd, nt, idx - 6, 4096 + blockIdx.x * 128);
      if (idx < 18) gemm_tile<2, 2>(p, call, lds, 36 + mt * 2, xcd, nt, 26 + t, 4096 + 32768 + (blockIdx.x - 48) * 128);
    }
  }
  STAMP(ws, 65536 + 1024 + blockIdx.x * 4 + w, NOW());
}

}

#define AG_MODE 0

extern "C" void kernel_launch(int rank, int nranks, ncclComm_t comm,
                              hipStream_t stream,
                              void* const* d_in, const int* in_sizes, int n_in,
                              void* d_out, int out_size,
                              const int* in_dtypes, int out_dtype,
                              void* const* d_out_all,
                              void* const* d_ws_all, size_t ws_bytes) {
  using namespace agk;
  if (nranks != NR || (unsigned)rank >= (unsigned)NR || n_in < 2 || in_sizes[0] != MS * KD || in_sizes[1] != KD * NS ||
      out_size != 4096 * NS || ws_bytes < (size_t)WS_USED) {
    fprintf(stderr, "kernel_launch: unexpected problem shape\n");
    return;
  }
  HIP_CHECK(hipFuncSetAttribute((const void*)ag_gemm, hipFuncAttributeMaxDynamicSharedMemorySize, LDS_BYTES));
  Params p;
  p.x = (const float*)d_in[0];
  p.w = (const float*)d_in[1];
  for (int i = 0; i < NR; ++i) { p.out[i] = d_out_all[i]; p.ws[i] = d_ws_all[i]; }
  p.rank = rank;
  p.mode = AG_MODE;
  hipLaunchKernelGGL(ag_gemm, dim3(256), dim3(256), LDS_BYTES, stream, p);
  HIP_CHECK(hipGetLastError());
}
```

```cpp
#include <hip/hip_runtime.h>
#include <rccl/rccl.h>
#include <cstdio>

#define HIP_CHECK(expr) do { hipError_t _e = (expr); if (_e != hipSuccess) { \
    fprintf(stderr, "HIP error %s at %s:%d\n", hipGetErrorString(_e), __FILE__, __LINE__); \
    return; } } while (0)

namespace agk {

typedef unsigned u32x4 __attribute__((ext_vector_type(4)));
typedef float f32x4 __attribute__((ext_vector_type(4)));
typedef float f32x16 __attribute__((ext_vector_type(16)));
typedef _Float16 f16x8 __attribute__((ext_vector_type(8)));
typedef __attribute__((address_space(1))) unsigned gu32;
typedef __amdgpu_buffer_rsrc_t rsrc_t;

constexpr int NR = 8, MS = 512, NS = 512, KD = 4096;
constexpr int BLK = 32 * KD * 2;
constexpr int A_SLOTS = 69;
constexpr int A_BYTES = A_SLOTS * BLK, B_BYTES = 32 * BLK, OX_BYTES = 4 << 20;
constexpr int EX_PIECE = 7168 + 256, EX_BYTES = 8 << 20;
constexpr int B_OFF = A_BYTES, OX_OFF = B_OFF + B_BYTES, EX_OFF = OX_OFF + OX_BYTES, BUF_BYTES = EX_OFF + EX_BYTES;
static_assert(1024 * EX_PIECE <= EX_BYTES, "export buffer");
constexpr int CNT_OFF = 0;
constexpr int TMO_OFF = 32768;
constexpr int TK_OFF = 33024;
constexpr int OQ_OFF = 34816;
constexpr int FA_OFF = 65536;
constexpr int FB_OFF = FA_OFF + A_SLOTS * 256;
constexpr int SF_OFF = 131072;
constexpr int CTRL_BYTES = 1 << 20;
constexpr int WS_USED = CTRL_BYTES + 2 * BUF_BYTES;
constexpr int OUT_BYTES = 4096 * NS * 4;
constexpr int IN_BYTES = MS * KD * 4;
constexpr int AUX_SYS = 17;
constexpr int RSRC_W3 = 0x00020000;
constexpr unsigned long long DEADLINE = 30000000ull;
constexpr int LDS_RED = 4 * 128 * 68 * 4, LDS_BYTES = LDS_RED + 64;
constexpr int LDS_WAVE = 16 * 528 + 512;

#define PACK 1
#define PW 2
#define NR0 16
#define PACE 1
#define PWIN 2
#define WIN 6
#define NSINGLE 8
#define AHEAD 3
#define NE 40
#define P0 16
#define EP0 40
#define EPS 4
#define ILV 0
#define NSB 5
#define WARM 0
constexpr int NPUSH = 8 * PW;
constexpr int NFW = 8;
constexpr int NP_INTRA = 352, T_AWAY = 128, T_OUT = 288, NP_CROSS = T_OUT + 118;
constexpr unsigned D_RAW = 1u << 13;
__device__ __forceinline__ bool desc_ok(unsigned d, unsigned call) { return (d >> 14) == (call & 0x3ffffu); }

struct Params {
  const float* x;
  const float* w;
  void* out[NR];
  void* ws[NR];
  int rank;
  int mode;
};

#define STAMP(ws_, idx_, val_) do {} while (0)
#define NOW() 0ull

__device__ __forceinline__ char* uni(const void* p) {
  const unsigned long long a = (unsigned long long)p;
  const unsigned lo = __builtin_amdgcn_readfirstlane((unsigned)a), hi = __builtin_amdgcn_readfirstlane((unsigned)(a >> 32));
  return (char*)(((unsigned long long)hi << 32) | lo);
}
__device__ __forceinline__ rsrc_t mk(const void* p, int bytes) {
  return __builtin_amdgcn_make_buffer_rsrc((void*)uni(p), 0, bytes, RSRC_W3);
}
__device__ __forceinline__ unsigned ld_sys(const void* base, int off) {
  return __hip_atomic_load((gu32*)((char*)base + off), __ATOMIC_RELAXED, __HIP_MEMORY_SCOPE_SYSTEM);
}
__device__ __forceinline__ void st_sys(void* base, int off, unsigned v) {
  __hip_atomic_store((gu32*)((char*)base + off), v, __ATOMIC_RELAXED, __HIP_MEMORY_SCOPE_SYSTEM);
}
__device__ __forceinline__ unsigned ld_agent(const void* base, int off) {
  return __hip_atomic_load((gu32*)((char*)base + off), __ATOMIC_RELAXED, __HIP_MEMORY_SCOPE_AGENT);
}
__device__ __forceinline__ void st_agent(void* base, int off, unsigned v) {
  __hip_atomic_store((gu32*)((char*)base + off), v, __ATOMIC_RELAXED, __HIP_MEMORY_SCOPE_AGENT);
}
#define DRAIN() asm volatile("s_waitcnt vmcnt(0)" ::: "memory")

__device__ __forceinline__ u32x4 cvt8(f32x4 a, f32x4 b) {
  f16x8 h;
  h[0] = (_Float16)a[0]; h[1] = (_Float16)a[1]; h[2] = (_Float16)a[2]; h[3] = (_Float16)a[3];
  h[4] = (_Float16)b[0]; h[5] = (_Float16)b[1]; h[6] = (_Float16)b[2]; h[7] = (_Float16)b[3];
  return __builtin_bit_cast(u32x4, h);
}

__device__ __forceinline__ int slot_row(int rank, int slot) {
  const int grp = rank >> 2, j = rank & 3;
  int src, b;
  if (slot < 16) { src = rank; b = slot; }
  else if (slot < 49) { const int r = (slot - 16) / 11; src = grp * 4 + ((j + 1 + r) & 3); b = slot - 16 - r * 11; }
  else { const int i = (slot - 49) / 5; src = (grp ^ 1) * 4 + i; b = 11 + slot - 49 - i * 5; }
  return src * MS + b * 32;
}
__device__ __forceinline__ int mate_slot(int r, int b) { return 16 + r * 11 + b; }
__device__ __forceinline__ int away_slot(int i, int b) { return 49 + i * 5 + (b - 11); }
__device__ __forceinline__ int out_slot(int ow, int ob) {
  if (ob < 11) return ob;
  if (ob < 26) { const int r = (ob - 11) / 5; return mate_slot(r, ob - 11 - r * 5); }
  if (ob < 41) { const int r = (ob - 26) / 5; return away_slot((ow + 1 + r) & 3, 11 + ob - 26 - r * 5); }
  if (ob < 47) return mate_slot((ob - 41) >> 1, 6 - (ob & 1));
  return mate_slot((ob - 47) >> 2, 7 + ((ob - 47) & 3));
}
__device__ __forceinline__ int own_slot(int i) {
  if (i < 16) return i;
  if (i < 31) return out_slot(0, i - 5);
  if (i < 51) return i + 18;
  return out_slot(0, i - 10);
}
__device__ __forceinline__ int a_flag(int slot, int u) { return FA_OFF + slot * 256 + (u & 3) * 64 + (u >> 2) * 4; }
__device__ __forceinline__ int b_flag(int qslot, int h) { return FB_OFF + qslot * 512 + ((h >> 1) & 3) * 128 + (h & 1) * 64 + (h >> 3) * 4; }

__device__ __forceinline__ void x_piece(int e, int& b, int& j) {
  if (e < 160) { j = e / 5; b = e - j * 5; }
  else if (e < 224) { b = 5 + (e & 1); j = (e - 160) >> 1; }
  else if (e < NP_INTRA) { b = 7 + (e & 3); j = (e - 224) >> 2; }
  else { j = (e - NP_INTRA) / 5; b = 11 + e - NP_INTRA - j * 5; }
}
__device__ __forceinline__ void x_load(f32x4 (&v)[16], rsrc_t rx, int lane, int b, int j) {
  const int voff = (lane >> 5) * (KD * 4) + (lane & 31) * 16, soff = b * (32 * KD * 4) + j * 512;
#pragma unroll
  for (int i = 0; i < 16; ++i)
    v[i] = __builtin_bit_cast(f32x4, __builtin_amdgcn_raw_buffer_load_b128(rx, voff, soff + i * (2 * KD * 4), 0));
}
__device__ __forceinline__ void x_frags(u32x4 (&f)[8], const f32x4 (&v)[16], char* lw, int lane) {
  typedef _Float16 f16x4 __attribute__((ext_vector_type(4)));
  char* wp = lw + ((lane & 31) >> 1) * 528 + (lane >> 5) * 16 + (lane & 1) * 8;
#pragma unroll
  for (int i = 0; i < 16; ++i) {
    f16x4 h = {(_Float16)v[i][0], (_Float16)v[i][1], (_Float16)v[i][2], (_Float16)v[i][3]};
    *(f16x4*)(wp + i * 32) = h;
  }
  asm volatile("" ::: "memory");
  const char* rp = lw + (lane >> 5) * 528 + (lane & 31) * 16;
#pragma unroll
  for (int s = 0; s < 8; ++s) f[s] = *(const u32x4*)(rp + s * 1056);
  asm volatile("" ::: "memory");
}
__device__ __forceinline__ void w_load(f32x4 (&v)[16], rsrc_t rw, int lane, int q, int h) {
  const int voff = (lane >> 5) * (8 * NS * 4) + (lane & 31) * 16, soff = h * (32 * NS * 4) + q * 512;
#pragma unroll
  for (int i = 0; i < 16; ++i)
    v[i] = __builtin_bit_cast(f32x4, __builtin_amdgcn_raw_buffer_load_b128(
        rw, voff + (i & 1) * (NS * 4), soff + ((i >> 3) * 16 + (i & 6)) * (NS * 4), 0));
}
__device__ __forceinline__ void w_frags(u32x4 (&f)[8], const f32x4 (&v)[16]) {
#pragma unroll
  for (int s = 0; s < 2; ++s)
#pragma unroll
    for (int e = 0; e < 4; ++e) {
      f32x4 a = {v[s * 8][e], v[s * 8 + 1][e], v[s * 8 + 2][e], v[s * 8 + 3][e]};
      f32x4 c = {v[s * 8 + 4][e], v[s * 8 + 5][e], v[s * 8 + 6][e], v[s * 8 + 7][e]};
      f[s * 4 + e] = cvt8(a, c);
    }
}
__device__ __forceinline__ int w_lane_off(int lane) { return ((lane & 31) >> 3) * BLK + ((lane >> 5) * 32 + (lane & 7)) * 16; }

__device__ __forceinline__ unsigned pack_piece(const u32x4 (&f)[8], rsrc_t rl, int exoff, char* lw, int lane) {
  typedef unsigned short u16x2 __attribute__((ext_vector_type(2)));
  unsigned* list = (unsigned*)(lw + 16 * 528);
  list[lane] = 0;
  unsigned se = 0;
#pragma unroll
  for (int d = 0; d < 4; ++d) se += ((f[0][d] >> 10) & 0x001f001fu) + ((f[4][d] >> 10) & 0x001f001fu);
  se = (se & 0xffffu) + (se >> 16);
  se += __builtin_amdgcn_update_dpp(0u, se, 0x111, 0xf, 0xf, false);
  se += __builtin_amdgcn_update_dpp(0u, se, 0x112, 0xf, 0xf, false);
  se += __builtin_amdgcn_update_dpp(0u, se, 0x114, 0xf, 0xf, false);
  se += __builtin_amdgcn_update_dpp(0u, se, 0x118, 0xf, 0xf, false);
  se += __builtin_amdgcn_update_dpp(0u, se, 0x142, 0xa, 0xf, false);
  se += __builtin_amdgcn_update_dpp(0u, se, 0x143, 0xc, 0xf, false);
  int base = (int)(((unsigned)__builtin_amdgcn_readlane(se, 63) + 1843 + 1023) >> 10) - 7;
  base = base < 0 ? 0 : base > 24 ? 24 : base;
  const unsigned b2 = (unsigned)base * 0x04000400u;
  unsigned pk[28], esc[8];
#pragma unroll
  for (int g = 0; g < 4; ++g) {
    unsigned t[8], wd[8];
#pragma unroll
    for (int i = 0; i < 8; ++i) {
      wd[i] = f[2 * g + (i >> 2)][i & 3];
      t[i] = __builtin_bit_cast(unsigned, (u16x2)(__builtin_bit_cast(u16x2, wd[i]) - __builtin_bit_cast(u16x2, b2)));
    }
    const unsigned z = (t[7] & 0x1fff1fffu) | ((wd[7] >> 2) & 0x20002000u);
#pragma unroll
    for (int i = 0; i < 7; ++i) pk[7 * g + i] = (t[i] & 0x1fff1fffu) | (wd[i] & 0x80008000u) | ((z << (13 - 2 * i)) & 0x60006000u);
#pragma unroll
    for (int u = 0; u < 2; ++u) {
      unsigned a = 0;
#pragma unroll
      for (int d = 0; d < 4; ++d) a |= ((t[4 * u + d] ^ (wd[4 * u + d] & 0x80008000u)) >> (4 * d)) & (0xe000e000u >> (4 * d));
      esc[2 * g + u] = a;
    }
  }
#pragma unroll
  for (int k = 0; k < 7; ++k)
    __builtin_amdgcn_raw_buffer_store_b128(u32x4{pk[4 * k], pk[4 * k + 1], pk[4 * k + 2], pk[4 * k + 3]}, rl,
                                           exoff + k * 1024 + lane * 16, 0, AUX_SYS);
  int cnt = 0;
  asm volatile("" ::: "memory");
#pragma unroll
  for (int u = 0; u < 8; ++u) {
    unsigned a = esc[u];
    unsigned long long m = __ballot(a != 0);
    do {
      const unsigned sh = __builtin_ctz(a | 0x80000000u) & 28u;
      const unsigned q = (a >> (sh + 1)) & 7u;
      const unsigned e = ((u * 512 + lane * 8 + 6) << 4) + (sh & 16u) - ((sh & 12u) << 3) + q;
      unsigned at = __builtin_amdgcn_mbcnt_hi((unsigned)(m >> 32), __builtin_amdgcn_mbcnt_lo((unsigned)m, cnt));
      at = (a != 0 && at < 128u) ? at : 128u + lane;
      ((unsigned short*)list)[at] = (unsigned short)e;
      a &= ~(0xfu << sh);
      cnt += __builtin_popcountll(m);
      m = __ballot(a != 0);
    } while (m != 0);
  }
  asm volatile("" ::: "memory");
  __builtin_amdgcn_raw_buffer_store_b32(list[lane], rl, exoff + 7168 + lane * 4, 0, AUX_SYS);
  asm volatile("" ::: "memory");
  return cnt > 128 ? D_RAW : (unsigned)(cnt << 5 | base);
}
__device__ __forceinline__ void unpack_piece(u32x4 (&r)[8], unsigned desc, unsigned patch, char* img, int lane) {
  const unsigned b2 = (desc & 31u) * 0x04000400u;
  unsigned pk[28];
#pragma unroll
  for (int i = 0; i < 28; ++i) pk[i] = r[i >> 2][i & 3];
#pragma unroll
  for (int g = 0; g < 4; ++g) {
    unsigned o[8], z = 0;
#pragma unroll
    for (int i = 0; i < 7; ++i) {
      o[i] = (pk[7 * g + i] & 0x9fff9fffu) + b2;
      z |= (pk[7 * g + i] >> (13 - 2 * i)) & (0x00030003u << (2 * i));
    }
    o[7] = ((z & 0x1fff1fffu) | ((z << 2) & 0x80008000u)) + b2;
    *(u32x4*)(img + g * 2048 + lane * 16) = u32x4{o[0], o[1], o[2], o[3]};
    *(u32x4*)(img + g * 2048 + 1024 + lane * 16) = u32x4{o[4], o[5], o[6], o[7]};
  }
  asm volatile("" ::: "memory");
#pragma unroll
  for (int k = 0; k < 2; ++k) {
    const unsigned e = k ? patch >> 16 : patch & 0xffffu;
    const unsigned hi = e << 29;
    if ((e & 7u) != 0)
      __hip_atomic_fetch_add((unsigned*)(img + (e >> 5) * 4), (e & 16u) ? hi : (unsigned)((int)hi >> 16),
                             __ATOMIC_RELAXED, __HIP_MEMORY_SCOPE_WORKGROUP);
  }
  asm volatile("" ::: "memory");
#pragma unroll
  for (int i = 0; i < 8; ++i) r[i] = *(const u32x4*)(img + i * 1024 + lane * 16);
  asm volatile("" ::: "memory");
}

__device__ __forceinline__ void convert(const Params& p, char* ws, char* lds, int e, bool agg, int w, int lane, unsigned& call, int round) {
  const int grp = p.rank >> 2, jme = p.rank & 3;
  const bool isx = e < 512, home = e < NP_INTRA;
  f32x4 v[16];
  int b = 0, j = 0;
  const int q = (e >> 7) & 3, h = e & 127;
  const unsigned cnt0 = ld_agent(ws, CNT_OFF + blockIdx.x * 128 + w * 4);
  if (isx) {
    x_piece(e, b, j);
    x_load(v, mk(p.x, IN_BYTES), lane, b, j);
  } else {
    w_load(v, mk(p.w, IN_BYTES), lane, q, h);
  }
  if (call == 0) {
    call = __builtin_amdgcn_readfirstlane(cnt0) + 1;
    if (lane == 0) st_agent(ws, CNT_OFF + blockIdx.x * 128 + w * 4, call);
  }
  const int t = home ? e : isx ? e - NP_INTRA + T_AWAY : h;
  const int d = lane >> 2, nd = home ? 3 : isx ? 4 : 1;
  char* const sfp = (char*)p.ws[home ? grp * 4 + ((jme + 1 + d) & 3) : (grp ^ 1) * 4 + (isx ? d & 3 : q)] +
                    SF_OFF + ((call & 1) * 8 + p.rank) * 2048 + (t + (lane & 3)) * 4;
  const int base = CTRL_BYTES + (call & 1) * BUF_BYTES;
  const rsrc_t rl = mk(ws, WS_USED);
  char* lw = lds + w * LDS_WAVE;
  u32x4 f[8];
  if (isx) {
    x_frags(f, v, lw, lane);
    const int o = base + b * BLK + j * 8192 + lane * 16;
#pragma unroll
    for (int i = 0; i < 8; ++i) __builtin_amdgcn_raw_buffer_store_b128(f[i], rl, o + i * 1024, 0, AUX_SYS);
  } else {
    w_frags(f, v);
    const int o = base + B_OFF + q * 4 * BLK + w_lane_off(lane) + h * 2048;
#pragma unroll
    for (int i = 0; i < 8; ++i) __builtin_amdgcn_raw_buffer_store_b128(f[i], rl, o + (i >> 2) * 1024 + (i & 3) * 128, 0, AUX_SYS);
  }
  STAMP(ws, 80000 + e * 4, NOW());
  unsigned desc = D_RAW;
  if (PACK && t >= NPUSH + NR0) desc = pack_piece(f, rl, base + EX_OFF + e * EX_PIECE, lw, lane);
  STAMP(ws, 80000 + e * 4 + 1, NOW());
  DRAIN();
  STAMP(ws, 80000 + e * 4 + 2, NOW());
  if (!agg) {
    if (d < nd && (lane & 3) == 0) st_sys(sfp, 0, call << 14 | desc);
  } else if (t >= NPUSH) {
    unsigned* dl = (unsigned*)(lds + LDS_RED) + round * 4;
    if (lane == 0) dl[w] = call << 14 | desc;
    __syncthreads();
    if (w == 0 && d < nd) st_sys(sfp, 0, dl[lane & 3]);
  }
  if (isx) { if (lane < 2) st_sys(ws, a_flag(b, 2 * j + lane), call); }
  else if (lane == 0) st_sys(ws, b_flag(q, h), call);
}

__device__ __forceinline__ unsigned push_first(const Params& p, char* ws, char* lds, int link, int t, int w, int lane) {
  const int peer = p.rank ^ (link + 1), jme = p.rank & 3, jp = peer & 3;
  const bool isx = link < 3;
  const unsigned cnt0 = ld_agent(ws, CNT_OFF + blockIdx.x * 128 + w * 4);
  f32x4 v[16];
  int b = 0, j = 0;
  if (isx) { x_piece(t, b, j); x_load(v, mk(p.x, IN_BYTES), lane, b, j); }
  else w_load(v, mk(p.w, IN_BYTES), lane, jp, t);
  asm volatile("s_barrier" ::: "memory");
  const unsigned call = __builtin_amdgcn_readfirstlane(cnt0) + 1;
  const int base = CTRL_BYTES + (call & 1) * BUF_BYTES;
  const rsrc_t rp = mk(p.ws[peer], WS_USED);
  const int slot = mate_slot((jme - jp - 1) & 3, b);
  const int o0 = isx ? base + slot * BLK + j * 8192 + lane * 16
                     : base + B_OFF + (4 + jme) * 4 * BLK + w_lane_off(lane) + t * 2048;
  u32x4 f[8];
  if (isx) x_frags(f, v, lds, lane); else w_frags(f, v);
#pragma unroll
  for (int i = 0; i < 8; ++i) {
    if (i > PWIN) asm volatile("s_waitcnt vmcnt(%0)" :: "i"(PWIN) : "memory");
    __builtin_amdgcn_raw_buffer_store_b128(f[i], rp, o0 + (isx ? i * 1024 : (i >> 2) * 1024 + (i & 3) * 128), 0, AUX_SYS);
  }
  DRAIN();
  if (isx) { if (lane < 2) st_sys(p.ws[peer], a_flag(slot, 2 * j + lane), call); }
  else if (lane == 0) st_sys(p.ws[peer], b_flag(4 + jme, t), call);
  if (lane == 0) st_agent(ws, CNT_OFF + blockIdx.x * 128 + w * 4, call);
  STAMP(ws, link * 512 + t, NOW());
  return call;
}

struct Piece {
  int src, ex, dst;
  int kind;
  int fa, fb;
};
__device__ __forceinline__ Piece piece_of(int rank, int peer, int t, unsigned desc, int base) {
  Piece c;
  const int jme = rank & 3, jp = peer & 3;
  const bool intra = (peer >> 2) == (rank >> 2);
  if (intra || (t >= T_AWAY && t < T_OUT)) {
    int b, j;
    c.ex = intra ? t : t - T_AWAY + NP_INTRA;
    x_piece(c.ex, b, j);
    const int slot = intra ? mate_slot((jp - jme - 1) & 3, b) : away_slot(jp, b);
    c.src = base + b * BLK + j * 8192; c.dst = base + slot * BLK + j * 8192; c.kind = 0;
    c.fa = a_flag(slot, 2 * j); c.fb = a_flag(slot, 2 * j + 1);
  } else if (t < T_AWAY) {
    c.src = base + B_OFF + jme * 4 * BLK + t * 2048; c.ex = 512 + jme * 128 + t;
    c.dst = base + B_OFF + (4 + jp) * 4 * BLK + t * 2048; c.kind = 1;
    c.fa = c.fb = b_flag(4 + jp, t);
  } else {
    const int o = min((int)(desc & 127u), NP_CROSS - T_OUT - 1);
    c.src = base + OX_OFF + jme * (1 << 20) + o * 8192; c.ex = 0;
    c.dst = ((slot_row(peer, out_slot(jme, o >> 1)) + (o & 1) * 16) * NS + jp * 128) * 4; c.kind = 2;
    c.fa = c.fb = 0;
  }
  c.ex = base + EX_OFF + c.ex * EX_PIECE;
  return c;
}

constexpr int NSLOT = 8, SLOT_B = 8192 + 256, F_PUSH = NSLOT * SLOT_B, F_CTL = F_PUSH + 2 * LDS_WAVE;
enum { C_QN = 0, C_FIN = 1, C_ISSUED = 2, C_LANDED = 3, C_DONE = 4, C_ENT = 16 };
constexpr int OOB = 0x7ff00000;
__device__ __forceinline__ unsigned lds_ld(const unsigned* a) {
  return __builtin_amdgcn_readfirstlane(__hip_atomic_load(a, __ATOMIC_RELAXED, __HIP_MEMORY_SCOPE_WORKGROUP));
}
__device__ __forceinline__ void lds_st(unsigned* a, unsigned v) {
  asm volatile("" ::: "memory");
  __hip_atomic_store(a, v, __ATOMIC_RELAXED, __HIP_MEMORY_SCOPE_WORKGROUP);
  asm volatile("" ::: "memory");
}

__device__ __forceinline__ void fetch_schedule(const Params& p, char* ws, unsigned* ctl, unsigned call, int link, int lane) {
  const int rank = __builtin_amdgcn_readfirstlane(p.rank);
  const int peer = rank ^ (link + 1);
  const int np = link < 3 ? NP_INTRA : (__builtin_amdgcn_readfirstlane(p.mode) & 1) ? T_OUT : NP_CROSS;
  const rsrc_t rl = mk(ws, WS_USED);
  const int base = CTRL_BYTES + (call & 1) * BUF_BYTES;
  const int sf = SF_OFF + ((call & 1) * 8 + peer) * 2048;
  const int v2 = (np - NPUSH - NSINGLE) >> 1, nv = v2 + NSINGLE;
  const unsigned tkbase = (call - 1) * (unsigned)nv - (unsigned)NFW;
  int tko = TK_OFF + link * 128;
  asm volatile("" : "+v"(tko));
  unsigned tk = 0;
  int v = blockIdx.x & 7, n = 0, t = 0;
  bool fail = false;
  while ((unsigned)v < (unsigned)nv && !fail) {
    const int k = v < v2 ? 2 : 1, tf = NPUSH + (v < v2 ? 2 * v : v + v2);
    while ((int)lds_ld(ctl + C_ISSUED) + AHEAD <= n || (int)lds_ld(ctl + C_DONE + (n & 1)) + NSLOT <= n ||
           (int)lds_ld(ctl + C_DONE + (~n & 1)) + NSLOT <= n + 1) __builtin_amdgcn_s_sleep(1);
    if (lane == 0) tk = __builtin_amdgcn_raw_ptr_buffer_atomic_add_i32(1, rl, tko, 0, 0);
    unsigned dv = ld_sys(ws, sf + (tf + (lane & 1)) * 4);
    for (int i = 0; i < k; ++i) {
      t = tf + i;
      unsigned dc = __builtin_amdgcn_readlane(dv, i);
      if (!desc_ok(dc, call)) {
        const unsigned long long ts = __builtin_amdgcn_s_memrealtime();
        for (unsigned it = 1;; ++it) {
          dv = ld_sys(ws, sf + (tf + (lane & 1)) * 4);
          dc = __builtin_amdgcn_readlane(dv, i);
          if (desc_ok(dc, call)) break;
          if ((it & 15) == 0 && (__builtin_amdgcn_readfirstlane(ld_agent(ws, TMO_OFF)) == call ||
                                 __builtin_amdgcn_s_memrealtime() - ts > DEADLINE)) { fail = true; break; }
          __builtin_amdgcn_s_sleep(1);
        }
        if (fail) break;
      }
      const Piece c = piece_of(rank, peer, t, dc, base);
      unsigned* e = ctl + C_ENT + (n & (NSLOT - 1)) * 8;
      *(u32x4*)e = u32x4{(unsigned)((dc & D_RAW) ? c.src : c.ex), dc, (unsigned)c.kind, (unsigned)c.dst};
      *(u32x4*)(e + 4) = u32x4{(unsigned)c.fa, (unsigned)c.fb, (unsigned)t, 0u};
      lds_st(ctl + C_QN, ++n);
      if (t < 64) STAMP(ws, 72000 + link * 512 + t, NOW());
    }
    v = (int)(__builtin_amdgcn_readfirstlane(tk) - tkbase);
  }
  lds_st(ctl + C_FIN, n + 1);
  if (fail) {
    st_agent(ws, TMO_OFF, call);
    if (lane == 0) ((float*)p.out[rank])[rank * MS * NS] = __builtin_nanf("");
  }
}

#define DMA(op, voff, ldsaddr) do { unsigned keep_; \
  asm volatile("s_mov_b32 %0, m0\n\ts_mov_b32 m0, %2\n\ts_waitcnt vmcnt(%4)\n\t" op " %1, %3, 0 offen sc0 sc1 lds\n\t" \
               "s_mov_b32 m0, %0" : "=&s"(keep_) : "v"(voff), "s"(ldsaddr), "s"(rp), "i"(WIN - 1) : "memory"); } while (0)
__device__ __forceinline__ void fetch_load(const Params& p, char* lds, unsigned* ctl, int link, int lane) {
  const rsrc_t rp = mk(p.ws[p.rank ^ (link + 1)], WS_USED);
  const unsigned ring = (unsigned)(unsigned long long)(__attribute__((address_space(3))) char*)lds;
  const int lx = lane * 16, lw = w_lane_off(lane);
  if (WARM) {
    DMA("buffer_load_dword", lane * (WS_USED / 64 / 128 * 128), ring + 7 * SLOT_B);
    DRAIN();
  }
  unsigned q = 0;
  u32x4 e = {0, 0, 0, 0};
  int pub = 0;
  for (int n = 0;; ++n) {
    if ((int)__builtin_amdgcn_readfirstlane(q) <= n) {
      if ((int)lds_ld(ctl + C_QN) <= n) {
        DRAIN();
        pub = n;
        lds_st(ctl + C_LANDED, n);
        bool end = false;
        while ((int)lds_ld(ctl + C_QN) <= n) {
          if ((int)lds_ld(ctl + C_FIN) == n + 1) { end = true; break; }
          __builtin_amdgcn_s_sleep(1);
        }
        if (end) break;
      }
      asm volatile("" ::: "memory");
      e = *(const u32x4*)(ctl + C_ENT + (n & (NSLOT - 1)) * 8);
    }
    lds_st(ctl + C_ISSUED, n + 1);
    const unsigned dc = __builtin_amdgcn_readfirstlane(e[1]);
    const bool raw = dc & D_RAW, wraw = raw && __builtin_amdgcn_readfirstlane(e[2]) == 1;
    const int src = __builtin_amdgcn_readfirstlane(e[0]);
    const int a0 = (wraw ? lw : lx) + src, st = wraw ? 128 : 1024, hb = wraw ? 1024 : 4096;
    const unsigned slot = ring + (n & (NSLOT - 1)) * SLOT_B;
#pragma unroll
    for (int j = 0; j < 8; ++j) {
      if (j < 7 || raw) DMA("buffer_load_dwordx4", a0 + (j >> 2) * hb + (j & 3) * st, slot + j * 1024);
      else DMA("buffer_load_dword", (lane == 0 || 2 * lane < (int)((dc >> 5) & 255u)) ? src + 7168 + lane * 4 : OOB, slot + 8192);
      if (j == (WIN - 1) % 8) {
        const int l = n - (WIN - 1) / 8;
        if (l > pub) { pub = l; lds_st(ctl + C_LANDED, l); }
      }
      if (j == 4) {
        q = __hip_atomic_load(ctl + C_QN, __ATOMIC_RELAXED, __HIP_MEMORY_SCOPE_WORKGROUP);
        asm volatile("" ::: "memory");
        e = *(const u32x4*)(ctl + C_ENT + ((n + 1) & (NSLOT - 1)) * 8);
      }
    }
  }
}

__device__ __forceinline__ void fetch_work(const Params& p, char* ws, char* lds, unsigned* ctl, unsigned call, int link, int k, int lane) {
  const rsrc_t rl = mk(ws, WS_USED), ro = mk(p.out[p.rank], OUT_BYTES);
  const int lx = lane * 16, lw = w_lane_off(lane), lo = (lane >> 5) * (NS * 4) + (lane & 31) * 16;
  for (int n = k;; n += 2) {
    bool end = false;
    while ((int)lds_ld(ctl + C_LANDED) <= n) {
      const int fin = (int)lds_ld(ctl + C_FIN);
      if (fin != 0 && fin - 1 <= n) { end = true; break; }
      __builtin_amdgcn_s_sleep(1);
    }
    if (end) break;
    asm volatile("" ::: "memory");
    char* img = lds + (n & (NSLOT - 1)) * SLOT_B;
    const u32x4 e0 = *(const u32x4*)(ctl + C_ENT + (n & (NSLOT - 1)) * 8), e1 = *(const u32x4*)(ctl + C_ENT + (n & (NSLOT - 1)) * 8 + 4);
    u32x4 r[8];
#pragma unroll
    for (int i = 0; i < 8; ++i) r[i] = *(const u32x4*)(img + i * 1024 + lane * 16);
    const unsigned pt = *(const unsigned*)(img + 8192 + lane * 4);
    asm volatile("" ::: "memory");
    const unsigned dc = __builtin_amdgcn_readfirstlane(e0[1]);
    const int kind = __builtin_amdgcn_readfirstlane(e0[2]);
    if (!(dc & D_RAW)) unpack_piece(r, dc, 2 * lane < (int)((dc >> 5) & 255u) ? pt : 0u, img, lane);
    const int d0 = (kind == 0 ? lx : kind == 1 ? lw : lo) + (int)e0[3];
    const int dst_st = kind == 0 ? 1024 : kind == 1 ? 128 : 2 * NS * 4, dst_hb = kind == 1 ? 1024 : 4 * dst_st;
    const rsrc_t rd = kind == 2 ? ro : rl;
#pragma unroll
    for (int i = 0; i < 8; ++i)
      __builtin_amdgcn_raw_buffer_store_b128(r[i], rd, d0 + (i >> 2) * dst_hb + (i & 3) * dst_st, 0, AUX_SYS);
    lds_st(ctl + C_DONE + k, n + 1);
    DRAIN();
    if (lane < 2 && kind != 2) st_sys(ws, lane ? e1[1] : e1[0], call);
    STAMP(ws, link * 512 + e1[2], NOW());
  }
}

__device__ __forceinline__ float silu(float y) {
  const float t = fminf(fmaxf(y, -60.f), 60.f);
  return y * __builtin_amdgcn_rcpf(1.f + __expf(-t));
}

template <int SLEEP>
__device__ __forceinline__ int wait_ready(void* ws, int offa, int offb, unsigned call, int need) {
  unsigned long long t0 = __builtin_amdgcn_s_memrealtime();
  for (unsigned it = 1;; ++it) {
    const unsigned fa = ld_sys(ws, offa), fb = ld_sys(ws, offb);
    const unsigned long long m = __ballot(fa >= call && fb >= call);
    const unsigned r = (unsigned)(m & (m >> 16) & (m >> 32) & (m >> 48)) & 0xffffu;
    const int n = __builtin_ctz(~r);
    if (n > need) { asm volatile("" ::: "memory"); return n; }
    if ((it & 15) == 0 && (ld_agent(ws, TMO_OFF) == call || __builtin_amdgcn_s_memrealtime() - t0 > DEADLINE)) {
      st_agent(ws, TMO_OFF, call);
      return 17;
    }
    if (need == 0) __builtin_amdgcn_s_sleep(16); else if (need < 12) __builtin_amdgcn_s_sleep(SLEEP); else __builtin_amdgcn_s_sleep(1);
  }
}

template <int MB>
__device__ __forceinline__ void gemm_tile(const Params& p, char* ws, unsigned call, float* lds, int slot0, int slot1, int qslot,
                                          int ob, int stamp_base) {
  constexpr int NB = 4;
  const int lane = threadIdx.x & 63;
  const int w = __builtin_amdgcn_readfirstlane(threadIdx.x >> 6);
  const rsrc_t rs = mk(ws, WS_USED);
  const int base = CTRL_BYTES + (call & 1) * BUF_BYTES;
  const int abase[2] = {base + slot0 * BLK + w * 4096, base + slot1 * BLK + w * 4096};
  const int bbase = base + B_OFF + qslot * 4 * BLK + w * 4096;
  const int offa = FA_OFF + ((lane & 16) ? slot1 : slot0) * 256 + w * 64 + (lane & 15) * 4;
  const int offb = FB_OFF + qslot * 512 + w * 128 + (lane & 31) * 4;
  const int voff = lane * 16;

  f32x16 acc[MB][NB];
#pragma unroll
  for (int m = 0; m < MB; ++m)
#pragma unroll
    for (int n = 0; n < NB; ++n)
#pragma unroll
      for (int r = 0; r < 16; ++r) acc[m][n][r] = 0.f;
  u32x4 fa0[4][MB], fb0[4][NB], fa1[4][MB], fb1[4][NB];

#define ISSUE(FA, FB, kc) do { \
    _Pragma("unroll") for (int s = 0; s < 4; ++s) { \
      _Pragma("unroll") for (int n = 0; n < NB; ++n) \
        FB[s][n] = __builtin_amdgcn_raw_buffer_load_b128(rs, voff + s * 1024, bbase + n * BLK + (kc) * 16384, AUX_SYS); \
      _Pragma("unroll") for (int m = 0; m < MB; ++m) \
        FA[s][m] = __builtin_amdgcn_raw_buffer_load_b128(rs, voff + s * 1024, abase[m] + (kc) * 16384, AUX_SYS); \
    } \
    __builtin_amdgcn_sched_barrier(0); } while (0)
#define COMPUTE(FA, FB) do { \
    _Pragma("unroll") for (int s = 0; s < 4; ++s) \
      _Pragma("unroll") for (int m = 0; m < MB; ++m) \
        _Pragma("unroll") for (int n = 0; n < NB; ++n) \
          acc[m][n] = __builtin_amdgcn_mfma_f32_32x32x16_f16(__builtin_bit_cast(f16x8, FB[s][n]), \
                                                             __builtin_bit_cast(f16x8, FA[s][m]), acc[m][n], 0, 0, 0); \
    __builtin_amdgcn_sched_barrier(0); } while (0)
#define STEP(FA, FB, kc) do { \
    _Pragma("unroll") for (int s = 0; s < 4; ++s) { \
      _Pragma("unroll") for (int n = 0; n < NB; ++n) { \
        _Pragma("unroll") for (int m = 0; m < MB; ++m) \
          acc[m][n] = __builtin_amdgcn_mfma_f32_32x32x16_f16(__builtin_bit_cast(f16x8, FB[s][n]), \
                                                             __builtin_bit_cast(f16x8, FA[s][m]), acc[m][n], 0, 0, 0); \
        FB[s][n] = __builtin_amdgcn_raw_buffer_load_b128(rs, voff + s * 1024, bbase + n * BLK + (kc) * 16384, AUX_SYS); \
        __builtin_amdgcn_sched_barrier(0); \
      } \
      _Pragma("unroll") for (int m = 0; m < MB; ++m) \
        FA[s][m] = __builtin_amdgcn_raw_buffer_load_b128(rs, voff + s * 1024, abase[m] + (kc) * 16384, AUX_SYS); \
      __builtin_amdgcn_sched_barrier(0); \
    } } while (0)
#define READY(kc) do { if (nready <= (kc)) nready = wait_ready<(MB == 2 ? 8 : 2)>(ws, offa, offb, call, (kc)); \
    STAMP(ws, stamp_base + w * 32 + (kc), NOW()); } while (0)

  int nready = 0;
  STAMP(ws, stamp_base + w * 32 + 16, NOW());
  READY(0); ISSUE(fa0, fb0, 0);
  READY(1); ISSUE(fa1, fb1, 1);
  for (int kc = 0; kc < 14; kc += 2) {
    COMPUTE(fa0, fb0);
    READY(kc + 2); ISSUE(fa0, fb0, kc + 2);
    COMPUTE(fa1, fb1);
    READY(kc + 3); ISSUE(fa1, fb1, kc + 3);
  }
  COMPUTE(fa0, fb0);
  COMPUTE(fa1, fb1);
#undef ISSUE
#undef STEP
#undef COMPUTE
#undef READY
  STAMP(ws, stamp_base + w * 32 + 17, NOW());

  constexpr int LD = NB * 32 + 4;
  const bool remote = qslot >= 4;
  unsigned oq = 0;
  if (remote && threadIdx.x == 0) {
    int oo = OQ_OFF + (qslot - 4) * 128;
    asm volatile("" : "+v"(oo));
    oq = __builtin_amdgcn_raw_ptr_buffer_atomic_add_i32(2 * MB, rs, oo, 0, 0);
  }
  {
    const bool bad = nready > 16 || ld_agent(ws, TMO_OFF) == call;
    float* dst = lds + w * (MB * 32 * LD) + (lane & 31) * LD + (lane >> 5) * 16;
#pragma unroll
    for (int m = 0; m < MB; ++m)
#pragma unroll
      for (int n = 0; n < NB; ++n)
#pragma unroll
        for (int t = 0; t < 4; ++t) {
          f32x4 o = {acc[m][n][t], acc[m][n][4 + t], acc[m][n][8 + t], acc[m][n][12 + t]};
          *(f32x4*)(dst + m * 32 * LD + n * 32 + t * 4) = o;
        }
    if (bad) {
      const float q = __builtin_nanf("");
      for (int m = 0; m < MB; ++m)
        for (int i = 0; i < NB * 4; ++i) *(f32x4*)(dst + m * 32 * LD + (i >> 2) * 32 + (i & 3) * 4) = f32x4{q, q, q, q};
    }
  }
  __syncthreads();
  const rsrc_t ro = remote ? rs : mk(p.out[p.rank], OUT_BYTES);
  constexpr int LPR = NB * 8;
  constexpr int RPI = 64 / LPR;
  constexpr int RPW = MB * 8;
  const int grow = slot_row(p.rank, (w * RPW) >> 5 ? slot1 : slot0) + ((w * RPW) & 31);
  const int o0 = remote ? base + OX_OFF + (qslot - 4) * (1 << 20) + ob * 16384 + w * (RPW * 512) + lane * 16
                        : ((grow + lane / LPR) * NS + qslot * 128) * 4 + (lane % LPR) * 16;
  const int ost = remote ? 1024 : RPI * NS * 4;
#pragma unroll
  for (int i = 0; i < RPW / RPI; ++i) {
    const int row = w * RPW + i * RPI + lane / LPR, c4 = (lane % LPR) * 4;
    f32x4 s = *(const f32x4*)(lds + row * LD + c4);
#pragma unroll
    for (int k = 1; k < 4; ++k) s += *(const f32x4*)(lds + k * (MB * 32 * LD) + row * LD + c4);
    f32x4 o;
#pragma unroll
    for (int e = 0; e < 4; ++e) o[e] = silu(s[e]);
    __builtin_amdgcn_raw_buffer_store_b128(__builtin_bit_cast(u32x4, o), ro, o0 + i * ost, 0, AUX_SYS);
  }
  DRAIN();
  __syncthreads();
  if (remote && w == 0) {
    const unsigned at = __builtin_amdgcn_readfirstlane(oq) - (call - 1) * (unsigned)(NP_CROSS - T_OUT) + lane;
    if (lane < 2 * MB && at < (unsigned)(NP_CROSS - T_OUT))
      st_sys(p.ws[((p.rank >> 2) ^ 1) * 4 + (qslot - 4)], SF_OFF + ((call & 1) * 8 + p.rank) * 2048 + (T_OUT + at) * 4,
             call << 14 | D_RAW | (2 * ob + lane));
  }
  STAMP(ws, stamp_base + w * 32 + 18, NOW());
}

__global__ void __launch_bounds__(256, 1) ag_gemm(Params p) {
  extern __shared__ __attribute__((aligned(16))) float lds[];
  const int lane = threadIdx.x & 63;
  const int w = __builtin_amdgcn_readfirstlane(threadIdx.x >> 6);
  const int xcd = blockIdx.x & 7, idx = blockIdx.x >> 3;
  char* const ws = uni(p.ws[p.rank]);
  if (p.mode & 2) return;
  STAMP(ws, 65536 + blockIdx.x * 4 + w, NOW());

  if (idx < 7) {
    const int link = (idx + xcd) % 7;
    unsigned* ctl = (unsigned*)((char*)lds + F_CTL);
    unsigned call = 0;
    if (w == 0) {
      if (lane < 16) ctl[lane] = 0;
      asm volatile("s_waitcnt lgkmcnt(0)\n\ts_barrier" ::: "memory");
      call = __builtin_amdgcn_readfirstlane(ld_agent(ws, CNT_OFF + blockIdx.x * 128)) + 1;
      if (lane == 0) st_agent(ws, CNT_OFF + blockIdx.x * 128, call);
      fetch_schedule(p, ws, ctl, call, link, lane);
    } else if (w == 1) {
      asm volatile("s_barrier" ::: "memory");
      fetch_load(p, (char*)lds, ctl, link, lane);
    } else {
      if (PW > 0 && w >= 4 - PW) {
        call = push_first(p, ws, (char*)lds + F_PUSH + (w - 2) * LDS_WAVE, link, (3 - w) * 8 + xcd, w, lane);
      } else {
        asm volatile("s_barrier" ::: "memory");
        call = __builtin_amdgcn_readfirstlane(ld_agent(ws, CNT_OFF + blockIdx.x * 128 + w * 4)) + 1;
        if (lane == 0) st_agent(ws, CNT_OFF + blockIdx.x * 128 + w * 4, call);
      }
      fetch_work(p, ws, (char*)lds, ctl, call, link, w - 2, lane);
    }
  } else {
    unsigned call = 0;
    {
      const int g = blockIdx.x - 56;
      constexpr int A0 = NPUSH / 4, A1 = (NPUSH + NE) / 4, NG = 32 - NE / 4, NTASK = 5 * NG + 96;
      auto task_piece = [](int o) {
        if (o < 5 * NG) {
          int a = o / 5; const int r = o - a * 5;
          a = a < 8 ? a + A1 : a < 8 + A0 ? a - 8 : a + NE / 4;
          return r == 0 ? a * 4 : 384 + r * 128 + a * 4;
        }
        const int r = o - 5 * NG;
        return r < 80 ? ((r & 1) ? NP_INTRA + (r >> 1) * 4 : (32 + (r >> 1)) * 4) : (r - 8) * 4;
      };
      const bool early = g < 5 * NE;
      if (!(early && w == 0)) for (int i = -P0; i < g; ++i) __builtin_amdgcn_s_sleep(PACE);
      else for (int i = EP0; i < g; i += EPS) __builtin_amdgcn_s_sleep(1);
      const int r1 = g >= 400 - NTASK ? 3 : 2;
#pragma clang loop unroll(disable)
      for (int r = (early && w == 0) ? 0 : 1; r < r1; ++r) {
        const int s5 = g % 5;
        const int e = r == 0 ? (s5 == 0 ? 0 : 384 + s5 * 128) + NPUSH + g / 5 : task_piece(r == 1 ? g : g + NTASK - 200) + w;
        convert(p, ws, (char*)lds, e, r != 0, w, lane, call, r);
      }
      STAMP(ws, 65536 + 3072 + g * 4 + w, NOW());
    }
    if (!(p.mode & 1)) {
      const int gi = idx - 7;
      const bool own = xcd < 4;
      int ta, tb = -1;
      if (own) {
        ta = 4 * gi + 1;
        if (gi < 3) tb = 4 * (25 + gi) + 1; else if (gi == 3) tb = 2 * 56; else if (gi < 6) tb = 2 * (49 + 2 * gi) + 1; else if (gi < 14) tb = 2 * (55 + gi);
      } else {
        if (gi < 12) { ta = 2 * (2 + 2 * gi) + 1; tb = 2 * (47 + gi); }
        else if (gi < 14) { ta = 2 * (gi - 12); tb = 2 * (14 + gi); }
        else ta = gi < 17 ? 2 * (14 + gi) : 2 * (2 * gi - 3) + 1;
      }
#pragma clang loop unroll(disable)
      for (; ta >= 0; ta = tb, tb = -1) {
        const int ob = ta >> 1, s0 = own ? own_slot(ob) : out_slot(xcd - 4, ob);
        if (ta & 1) gemm_tile<2>(p, ws, call, lds, s0, own ? own_slot(ob + 1) : out_slot(xcd - 4, ob + 1), xcd, ob, 4096 + ((ob >> 1) * 8 + xcd) * 128);
        else gemm_tile<1>(p, ws, call, lds, s0, s0, xcd, ob, 4096 + 36864 + ((ob < 47 ? 0 : ob - 46) * 8 + xcd) * 128);
      }
    }
  }
  STAMP(ws, 65536 + 1024 + blockIdx.x * 4 + w, NOW());
}

}

#define AG_MODE 0

extern "C" void kernel_launch(int rank, int nranks, ncclComm_t comm,
                              hipStream_t stream,
                              void* const* d_in, const int* in_sizes, int n_in,
                              void* d_out, int out_size,
                              const int* in_dtypes, int out_dtype,
                              void* const* d_out_all,
                              void* const* d_ws_all, size_t ws_bytes) {
  using namespace agk;
  if (nranks != NR || (unsigned)rank >= (unsigned)NR || n_in < 2 || in_sizes[0] != MS * KD || in_sizes[1] != KD * NS ||
      out_size != 4096 * NS || ws_bytes < (size_t)WS_USED) {
    fprintf(stderr, "kernel_launch: unexpected problem shape\n");
    return;
  }
  HIP_CHECK(hipFuncSetAttribute((const void*)ag_gemm, hipFuncAttributeMaxDynamicSharedMemorySize, LDS_BYTES));
  Params p;
  p.x = (const float*)d_in[0];
  p.w = (const float*)d_in[1];
  for (int i = 0; i < NR; ++i) { p.out[i] = d_out_all[i]; p.ws[i] = d_ws_all[i]; }
  p.rank = rank;
  p.mode = AG_MODE;
  hipLaunchKernelGGL(ag_gemm, dim3(256), dim3(256), LDS_BYTES, stream, p);
  HIP_CHECK(hipGetLastError());
}
```

```cpp
#include <hip/hip_runtime.h>
#include <rccl/rccl.h>
#include <cstdio>

#define HIP_CHECK(expr) do { hipError_t _e = (expr); if (_e != hipSuccess) { \
    fprintf(stderr, "HIP error %s at %s:%d\n", hipGetErrorString(_e), __FILE__, __LINE__); \
    return; } } while (0)

namespace agk {

typedef unsigned u32x4 __attribute__((ext_vector_type(4)));
typedef float f32x4 __attribute__((ext_vector_type(4)));
typedef float f32x16 __attribute__((ext_vector_type(16)));
typedef _Float16 f16x8 __attribute__((ext_vector_type(8)));
typedef __attribute__((address_space(1))) unsigned gu32;
typedef __amdgpu_buffer_rsrc_t rsrc_t;

constexpr int NR = 8, MS = 512, NS = 512, KD = 4096;
constexpr int BLK = 32 * KD * 2;
constexpr int A_SLOTS = 72;
constexpr int A_BYTES = A_SLOTS * BLK, B_BYTES = 32 * BLK, OX_BYTES = 4 << 20;
constexpr int EX_PIECE = 7168 + 256, EX_BYTES = 8 << 20;
constexpr int B_OFF = A_BYTES, OX_OFF = B_OFF + B_BYTES, EX_OFF = OX_OFF + OX_BYTES, BUF_BYTES = EX_OFF + EX_BYTES;
static_assert(1024 * EX_PIECE <= EX_BYTES, "export buffer");
constexpr int CNT_OFF = 0;
constexpr int TMO_OFF = 32768;
constexpr int TK_OFF = 33024;
constexpr int OQ_OFF = 34816;
constexpr int FA_OFF = 65536, FLG_PAR = 32768;
constexpr int FB_OFF = FA_OFF + A_SLOTS * 256;
constexpr int SF_OFF = 131072, SF_SRC = 32768;
constexpr int CTRL_BYTES = 1 << 20;
constexpr int WS_USED = CTRL_BYTES + 2 * BUF_BYTES;
constexpr int OUT_BYTES = 4096 * NS * 4;
constexpr int IN_BYTES = MS * KD * 4;
constexpr int AUX_SYS = 17;
constexpr int RSRC_W3 = 0x00020000;
constexpr unsigned long long DEADLINE = 30000000ull;
constexpr int LDS_RED = 4 * 128 * 68 * 4, LDS_BYTES = LDS_RED + 64;
constexpr int LDS_WAVE = 16 * 528 + 512;

#define PACK 1
#define NPUSH 16
#define NR0 24
#define PACE 1
#define PWIN 2
#define WIN 6
#define NSINGLE 12
#define AHEAD 2
#define NE 56
#define P0 40
#define EP0 60
#define EPS 8
#define NRI 8
#define SFS 2
#define NSI 24
#define D2 4
#define D1 4
#define SKEW 0
#define PR2 4
#define PR1 8
#define NRL 0
#define SL1 2
#define SL2 1
#define NWI 0
#define NWC 12
#define WWIN 16
constexpr int NFW = 8;
constexpr int N_HOME = 352, E_ALL = 480, NP_INTRA = 384, T_AWAY = 128, T_OUT = 288, N_OUT = 112, NP_CROSS = T_OUT + N_OUT;
constexpr unsigned D_RAW = 1u << 13;
__device__ __forceinline__ bool desc_ok(unsigned d, unsigned call) { return (d >> 18) == (call & 0x3fffu); }

struct Params {
  const float* x;
  const float* w;
  void* out[NR];
  void* ws[NR];
  int rank;
  int mode;
};

#define ACC(ws_, idx_, t0_) do {} while (0)
#define STAMP(ws_, idx_, val_) do {} while (0)
#define NOW() 0ull

__device__ __forceinline__ char* uni(const void* p) {
  const unsigned long long a = (unsigned long long)p;
  const unsigned lo = __builtin_amdgcn_readfirstlane((unsigned)a), hi = __builtin_amdgcn_readfirstlane((unsigned)(a >> 32));
  return (char*)(((unsigned long long)hi << 32) | lo);
}
__device__ __forceinline__ rsrc_t mk(const void* p, int bytes) {
  return __builtin_amdgcn_make_buffer_rsrc((void*)uni(p), 0, bytes, RSRC_W3);
}
__device__ __forceinline__ unsigned ld_sys(const void* base, int off) {
  return __hip_atomic_load((gu32*)((char*)base + off), __ATOMIC_RELAXED, __HIP_MEMORY_SCOPE_SYSTEM);
}
__device__ __forceinline__ void st_sys(void* base, int off, unsigned v) {
  __hip_atomic_store((gu32*)((char*)base + off), v, __ATOMIC_RELAXED, __HIP_MEMORY_SCOPE_SYSTEM);
}
__device__ __forceinline__ unsigned ld_agent(const void* base, int off) {
  return __hip_atomic_load((gu32*)((char*)base + off), __ATOMIC_RELAXED, __HIP_MEMORY_SCOPE_AGENT);
}
__device__ __forceinline__ void st_agent(void* base, int off, unsigned v) {
  __hip_atomic_store((gu32*)((char*)base + off), v, __ATOMIC_RELAXED, __HIP_MEMORY_SCOPE_AGENT);
}
#define DRAIN() asm volatile("s_waitcnt vmcnt(0)" ::: "memory")

__device__ __forceinline__ u32x4 cvt8(f32x4 a, f32x4 b) {
  f16x8 h;
  h[0] = (_Float16)a[0]; h[1] = (_Float16)a[1]; h[2] = (_Float16)a[2]; h[3] = (_Float16)a[3];
  h[4] = (_Float16)b[0]; h[5] = (_Float16)b[1]; h[6] = (_Float16)b[2]; h[7] = (_Float16)b[3];
  return __builtin_bit_cast(u32x4, h);
}

__device__ __forceinline__ float silu(float y) {
  const float t = fminf(fmaxf(y, -60.f), 60.f);
  return y * __builtin_amdgcn_rcpf(1.f + __expf(-t));
}

__device__ __forceinline__ int slot_row(int rank, int slot) {
  const int grp = rank >> 2, j = rank & 3;
  int src, b;
  if (slot < 16) { src = rank; b = slot; }
  else if (slot < 52) { const int r = (slot - 16) / 12; src = grp * 4 + ((j + 1 + r) & 3); b = slot - 16 - r * 12; b = b == 11 ? 15 : b; }
  else { const int i = (slot - 52) / 5; src = (grp ^ 1) * 4 + i; b = 11 + slot - 52 - i * 5; }
  return src * MS + b * 32;
}
__device__ __forceinline__ int mate_slot(int r, int b) { return 16 + r * 12 + (b == 15 ? 11 : b); }
__device__ __forceinline__ int away_slot(int i, int b) { return 52 + i * 5 + (b - 11); }
__device__ __forceinline__ int out_slot(int ow, int ob) {
  if (ob < 11) return ob;
  if (ob < 26) { const int r = (ob - 11) / 5; return mate_slot(r, ob - 11 - r * 5); }
  if (ob < 38) return away_slot((ow + 1 + ((ob - 26) >> 2)) & 3, 11 + ((ob - 26) & 3));
  if (ob < 50) return mate_slot((ob - 38) >> 2, 5 + ((ob - 38) & 3));
  return mate_slot((ob - 50) >> 1, 9 + (ob & 1));
}
__device__ __forceinline__ int own_slot(int i) {
  if (i < 16) return i;
  if (i < 31) return out_slot(0, i - 5);
  if (i < 47) return away_slot((i - 31) >> 2, 11 + ((i - 31) & 3));
  if (i < 59) return out_slot(0, i - 9);
  if (i < 63) return away_slot(i - 59, 15);
  if (i < 69) return out_slot(0, i - 13);
  return mate_slot(i - 69, 15);
}
__device__ __forceinline__ int sf_pos(int t) {
  return t < NPUSH + NE ? 16384 + (t >> 1) * 128 + (t & 1) * 4 : (t >> SFS) * 128 + (t & ((1 << SFS) - 1)) * 4;
}
__device__ __forceinline__ int a_flag(int slot, int u) { return FA_OFF + slot * 256 + (u & 3) * 64 + (u >> 2) * 4; }
__device__ __forceinline__ int b_flag(int qslot, int h) { return FB_OFF + qslot * 512 + ((h >> 1) & 3) * 128 + (h & 1) * 64 + (h >> 3) * 4; }

__device__ __forceinline__ void x_piece(int e, int& b, int& j) {
  if (e < 160) { j = e / 5; b = e - j * 5; }
  else if (e < 288) { b = 5 + (e & 3); j = (e - 160) >> 2; }
  else if (e < N_HOME) { b = 9 + (e & 1); j = (e - 288) >> 1; }
  else if (e < E_ALL) { b = 11 + (e & 3); j = (e - N_HOME) >> 2; }
  else { b = 15; j = e - E_ALL; }
}
__device__ __forceinline__ void x_load(f32x4 (&v)[16], rsrc_t rx, int lane, int b, int j) {
  const int voff = (lane >> 5) * (KD * 4) + (lane & 31) * 16, soff = b * (32 * KD * 4) + j * 512;
#pragma unroll
  for (int i = 0; i < 16; ++i)
    v[i] = __builtin_bit_cast(f32x4, __builtin_amdgcn_raw_buffer_load_b128(rx, voff, soff + i * (2 * KD * 4), 0));
}
__device__ __forceinline__ unsigned scale_piece(f32x4 (&v)[16]) {
  float m = 0.f;
#pragma unroll
  for (int i = 0; i < 16; ++i)
    m = fmaxf(fmaxf(m, fmaxf(fabsf(v[i][0]), fabsf(v[i][1]))), fmaxf(fabsf(v[i][2]), fabsf(v[i][3])));
  unsigned u = __builtin_bit_cast(unsigned, m);
  u = max(u, (unsigned)__builtin_amdgcn_update_dpp(0u, u, 0x111, 0xf, 0xf, false));
  u = max(u, (unsigned)__builtin_amdgcn_update_dpp(0u, u, 0x112, 0xf, 0xf, false));
  u = max(u, (unsigned)__builtin_amdgcn_update_dpp(0u, u, 0x114, 0xf, 0xf, false));
  u = max(u, (unsigned)__builtin_amdgcn_update_dpp(0u, u, 0x118, 0xf, 0xf, false));
  u = max(u, (unsigned)__builtin_amdgcn_update_dpp(0u, u, 0x142, 0xa, 0xf, false));
  u = max(u, (unsigned)__builtin_amdgcn_update_dpp(0u, u, 0x143, 0xc, 0xf, false));
  const unsigned top = __builtin_amdgcn_readlane(u, 63);
  const int e = top >> 23;
  if (top == 0 || e == 255 || (e >= 117 && e <= 141)) return 0;
  const int z = min((141 - e) >> 4, 7);
#pragma unroll
  for (int i = 0; i < 16; ++i)
#pragma unroll
    for (int k = 0; k < 4; ++k) v[i][k] = __builtin_amdgcn_ldexpf(v[i][k], 16 * z);
  return z & 15;
}
__device__ __forceinline__ void x_frags(u32x4 (&f)[8], const f32x4 (&v)[16], char* lw, int lane) {
  typedef _Float16 f16x4 __attribute__((ext_vector_type(4)));
  char* wp = lw + ((lane & 31) >> 1) * 528 + (lane >> 5) * 16 + (lane & 1) * 8;
#pragma unroll
  for (int i = 0; i < 16; ++i) {
    f16x4 h = {(_Float16)v[i][0], (_Float16)v[i][1], (_Float16)v[i][2], (_Float16)v[i][3]};
    *(f16x4*)(wp + i * 32) = h;
  }
  asm volatile("" ::: "memory");
  const char* rp = lw + (lane >> 5) * 528 + (lane & 31) * 16;
#pragma unroll
  for (int s = 0; s < 8; ++s) f[s] = *(const u32x4*)(rp + s * 1056);
  asm volatile("" ::: "memory");
}
__device__ __forceinline__ void w_load(f32x4 (&v)[16], rsrc_t rw, int lane, int q, int h) {
  const int voff = (lane >> 5) * (8 * NS * 4) + (lane & 31) * 16, soff = h * (32 * NS * 4) + q * 512;
#pragma unroll
  for (int i = 0; i < 16; ++i)
    v[i] = __builtin_bit_cast(f32x4, __builtin_amdgcn_raw_buffer_load_b128(
        rw, voff + (i & 1) * (NS * 4), soff + ((i >> 3) * 16 + (i & 6)) * (NS * 4), 0));
}
__device__ __forceinline__ void w_frags(u32x4 (&f)[8], const f32x4 (&v)[16]) {
#pragma unroll
  for (int s = 0; s < 2; ++s)
#pragma unroll
    for (int e = 0; e < 4; ++e) {
      f32x4 a = {v[s * 8][e], v[s * 8 + 1][e], v[s * 8 + 2][e], v[s * 8 + 3][e]};
      f32x4 c = {v[s * 8 + 4][e], v[s * 8 + 5][e], v[s * 8 + 6][e], v[s * 8 + 7][e]};
      f[s * 4 + e] = cvt8(a, c);
    }
}
__device__ __forceinline__ int w_lane_off(int lane) { return ((lane & 31) >> 3) * BLK + ((lane >> 5) * 32 + (lane & 7)) * 16; }

__device__ __forceinline__ unsigned pack_piece(const u32x4 (&f)[8], rsrc_t rl, int exoff, char* lw, int lane) {
  typedef unsigned short u16x2 __attribute__((ext_vector_type(2)));
  unsigned* list = (unsigned*)(lw + 16 * 528);
  list[lane] = 0;
  unsigned se = 0;
#pragma unroll
  for (int d = 0; d < 4; ++d) se += ((f[0][d] >> 10) & 0x001f001fu) + ((f[4][d] >> 10) & 0x001f001fu);
  se = (se & 0xffffu) + (se >> 16);
  se += __builtin_amdgcn_update_dpp(0u, se, 0x111, 0xf, 0xf, false);
  se += __builtin_amdgcn_update_dpp(0u, se, 0x112, 0xf, 0xf, false);
  se += __builtin_amdgcn_update_dpp(0u, se, 0x114, 0xf, 0xf, false);
  se += __builtin_amdgcn_update_dpp(0u, se, 0x118, 0xf, 0xf, false);
  se += __builtin_amdgcn_update_dpp(0u, se, 0x142, 0xa, 0xf, false);
  se += __builtin_amdgcn_update_dpp(0u, se, 0x143, 0xc, 0xf, false);
  int base = (int)(((unsigned)__builtin_amdgcn_readlane(se, 63) + 1843 + 1023) >> 10) - 7;
  base = base < 0 ? 0 : base > 24 ? 24 : base;
  const unsigned b2 = (unsigned)base * 0x04000400u;
  unsigned pk[28], esc[8];
#pragma unroll
  for (int g = 0; g < 4; ++g) {
    unsigned t[8], wd[8];
#pragma unroll
    for (int i = 0; i < 8; ++i) {
      wd[i] = f[2 * g + (i >> 2)][i & 3];
      t[i] = __builtin_bit_cast(unsigned, (u16x2)(__builtin_bit_cast(u16x2, wd[i]) - __builtin_bit_cast(u16x2, b2)));
    }
    const unsigned z = (t[7] & 0x1fff1fffu) | ((wd[7] >> 2) & 0x20002000u);
#pragma unroll
    for (int i = 0; i < 7; ++i) pk[7 * g + i] = (t[i] & 0x1fff1fffu) | (wd[i] & 0x80008000u) | ((z << (13 - 2 * i)) & 0x60006000u);
#pragma unroll
    for (int u = 0; u < 2; ++u) {
      unsigned a = 0;
#pragma unroll
      for (int d = 0; d < 4; ++d) a |= ((t[4 * u + d] ^ (wd[4 * u + d] & 0x80008000u)) >> (4 * d)) & (0xe000e000u >> (4 * d));
      esc[2 * g + u] = a;
    }
  }
#pragma unroll
  for (int k = 0; k < 7; ++k)
    __builtin_amdgcn_raw_buffer_store_b128(u32x4{pk[4 * k], pk[4 * k + 1], pk[4 * k + 2], pk[4 * k + 3]}, rl,
                                           exoff + k * 1024 + lane * 16, 0, AUX_SYS);
  int cnt = 0;
  asm volatile("" ::: "memory");
#pragma unroll
  for (int u = 0; u < 8; ++u) {
    unsigned a = esc[u];
    unsigned long long m = __ballot(a != 0);
    do {
      const unsigned sh = __builtin_ctz(a | 0x80000000u) & 28u;
      const unsigned q = (a >> (sh + 1)) & 7u;
      const unsigned e = ((u * 512 + lane * 8 + 6) << 4) + (sh & 16u) - ((sh & 12u) << 3) + q;
      unsigned at = __builtin_amdgcn_mbcnt_hi((unsigned)(m >> 32), __builtin_amdgcn_mbcnt_lo((unsigned)m, cnt));
      at = (a != 0 && at < 128u) ? at : 128u + lane;
      ((unsigned short*)list)[at] = (unsigned short)e;
      a &= ~(0xfu << sh);
      cnt += __builtin_popcountll(m);
      m = __ballot(a != 0);
    } while (m != 0);
  }
  asm volatile("" ::: "memory");
  __builtin_amdgcn_raw_buffer_store_b32(list[lane], rl, exoff + 7168 + lane * 4, 0, AUX_SYS);
  asm volatile("" ::: "memory");
  return cnt > 128 ? D_RAW : (unsigned)(cnt << 5 | base);
}
__device__ __forceinline__ void unpack_piece(u32x4 (&r)[8], unsigned desc, unsigned patch, char* img, int lane) {
  const unsigned b2 = (desc & 31u) * 0x04000400u;
  unsigned pk[28];
#pragma unroll
  for (int i = 0; i < 28; ++i) pk[i] = r[i >> 2][i & 3];
#pragma unroll
  for (int g = 0; g < 4; ++g) {
    unsigned o[8], z = 0;
#pragma unroll
    for (int i = 0; i < 7; ++i) {
      o[i] = (pk[7 * g + i] & 0x9fff9fffu) + b2;
      z |= (pk[7 * g + i] >> (13 - 2 * i)) & (0x00030003u << (2 * i));
    }
    o[7] = ((z & 0x1fff1fffu) | ((z << 2) & 0x80008000u)) + b2;
    *(u32x4*)(img + g * 2048 + lane * 16) = u32x4{o[0], o[1], o[2], o[3]};
    *(u32x4*)(img + g * 2048 + 1024 + lane * 16) = u32x4{o[4], o[5], o[6], o[7]};
  }
  asm volatile("" ::: "memory");
#pragma unroll
  for (int k = 0; k < 2; ++k) {
    const unsigned e = k ? patch >> 16 : patch & 0xffffu;
    const unsigned hi = e << 29;
    if ((e & 7u) != 0)
      __hip_atomic_fetch_add((unsigned*)(img + (e >> 5) * 4), (e & 16u) ? hi : (unsigned)((int)hi >> 16),
                             __ATOMIC_RELAXED, __HIP_MEMORY_SCOPE_WORKGROUP);
  }
  asm volatile("" ::: "memory");
#pragma unroll
  for (int i = 0; i < 8; ++i) r[i] = *(const u32x4*)(img + i * 1024 + lane * 16);
  asm volatile("" ::: "memory");
}

__device__ __forceinline__ void convert(const Params& p, char* ws, char* lds, int e, bool agg, int w, int lane, unsigned& call, int round) {
  const int grp = p.rank >> 2, jme = p.rank & 3;
  const bool isx = e < 512, home = e < N_HOME;
  f32x4 v[16];
  int b = 0, j = 0;
  const int q = (e >> 7) & 3, h = e & 127;
  const unsigned cnt0 = ld_agent(ws, CNT_OFF + blockIdx.x * 128 + w * 4);
  if (isx) {
    x_piece(e, b, j);
    x_load(v, mk(p.x, IN_BYTES), lane, b, j);
  } else {
    w_load(v, mk(p.w, IN_BYTES), lane, q, h);
  }
  if (call == 0) {
    call = __builtin_amdgcn_readfirstlane(cnt0) + 1;
    if (lane == 0) st_agent(ws, CNT_OFF + blockIdx.x * 128 + w * 4, call);
  }
  const int t = home ? e : isx ? e - N_HOME + T_AWAY : h;
  const int d = lane >> 2, nd = home ? 3 : !isx ? 1 : e < E_ALL ? 4 : 7;
  const bool tomate = home || d >= 4;
  char* const sfp = (char*)p.ws[tomate ? grp * 4 + ((jme + 1 + d) & 3) : (grp ^ 1) * 4 + (isx ? d : q)] +
                    SF_OFF + ((call & 1) * 8 + p.rank) * SF_SRC + sf_pos((tomate && !home ? t + 96 : t) + (lane & 3));
  const int base = CTRL_BYTES + (call & 1) * BUF_BYTES;
  const rsrc_t rl = mk(ws, WS_USED);
  char* lw = lds + w * LDS_WAVE;
  unsigned sc = 0;
  u32x4 f[8];
  for (;;) {
    if (isx) {
      x_frags(f, v, lw, lane);
      const int o = base + b * BLK + j * 8192 + lane * 16;
#pragma unroll
      for (int i = 0; i < 8; ++i) __builtin_amdgcn_raw_buffer_store_b128(f[i], rl, o + i * 1024, 0, AUX_SYS);
    } else {
      w_frags(f, v);
      const int o = base + B_OFF + q * 4 * BLK + w_lane_off(lane) + h * 2048;
#pragma unroll
      for (int i = 0; i < 8; ++i) __builtin_amdgcn_raw_buffer_store_b128(f[i], rl, o + (i >> 2) * 1024 + (i & 3) * 128, 0, AUX_SYS);
    }
    __builtin_amdgcn_sched_barrier(0);
    if (sc != 0 || (sc = scale_piece(v)) == 0) break;
    DRAIN();
  }
  STAMP(ws, 80000 + e * 4, NOW());
  unsigned desc = D_RAW;
  if (PACK && t >= NPUSH + NR0) desc = pack_piece(f, rl, base + EX_OFF + e * EX_PIECE, lw, lane);
  STAMP(ws, 80000 + e * 4 + 1, NOW());
  DRAIN();
  STAMP(ws, 80000 + e * 4 + 2, NOW());
  if (!agg) {
    if (d < nd && (lane & 3) == 0) st_sys(sfp, 0, call << 18 | sc << 14 | desc);
  } else if (t >= NPUSH) {
    unsigned* dl = (unsigned*)(lds + LDS_RED) + round * 4;
    if (lane == 0) dl[w] = call << 18 | sc << 14 | desc;
    __syncthreads();
    if (w == 0 && d < nd) st_sys(sfp, 0, dl[lane & 3]);
  }
  if (lane < (isx ? 2 : 1)) st_sys(ws, (call & 1) * FLG_PAR + (isx ? a_flag(b, 2 * j + lane) : b_flag(q, h)), call << 4 | sc);
}

__device__ __forceinline__ unsigned push_first(const Params& p, char* ws, char* lds, int link, int t, int w, int lane) {
  const int peer = p.rank ^ (link + 1), jme = p.rank & 3, jp = peer & 3;
  const bool isx = link < 3;
  const unsigned cnt0 = ld_agent(ws, CNT_OFF + blockIdx.x * 128 + w * 4);
  f32x4 v[16];
  int b = 0, j = 0;
  if (isx) { x_piece(t, b, j); x_load(v, mk(p.x, IN_BYTES), lane, b, j); }
  else w_load(v, mk(p.w, IN_BYTES), lane, jp, t);
  const unsigned call = __builtin_amdgcn_readfirstlane(cnt0) + 1;
  const int base = CTRL_BYTES + (call & 1) * BUF_BYTES;
  const rsrc_t rp = mk(p.ws[peer], WS_USED);
  const int slot = mate_slot((jme - jp - 1) & 3, b);
  const int o0 = isx ? base + slot * BLK + j * 8192 + lane * 16
                     : base + B_OFF + (4 + jme) * 4 * BLK + w_lane_off(lane) + t * 2048;
  unsigned sc = 0;
  for (;;) {
    u32x4 f[8];
    if (isx) x_frags(f, v, lds, lane); else w_frags(f, v);
#pragma unroll
    for (int i = 0; i < 8; ++i) {
      if (i > PWIN) asm volatile("s_waitcnt vmcnt(%0)" :: "i"(PWIN) : "memory");
      __builtin_amdgcn_raw_buffer_store_b128(f[i], rp, o0 + (isx ? i * 1024 : (i >> 2) * 1024 + (i & 3) * 128), 0, AUX_SYS);
    }
    __builtin_amdgcn_sched_barrier(0);
    if (sc != 0 || (sc = scale_piece(v)) == 0) break;
    DRAIN();
  }
  DRAIN();
  if (lane < (isx ? 2 : 1))
    st_sys(p.ws[peer], (call & 1) * FLG_PAR + (isx ? a_flag(slot, 2 * j + lane) : b_flag(4 + jme, t)), call << 4 | sc);
  if (lane == 0) st_agent(ws, CNT_OFF + blockIdx.x * 128 + w * 4, call);
  STAMP(ws, link * 512 + t, NOW());
  return call;
}

struct Piece {
  int src, ex, dst;
  int kind;
  int fa, fb;
};
__device__ __forceinline__ Piece piece_of(int rank, int peer, int t, unsigned desc, int base) {
  Piece c;
  const int jme = rank & 3, jp = peer & 3;
  const bool intra = (peer >> 2) == (rank >> 2);
  if (intra || (t >= T_AWAY && t < T_OUT)) {
    int b, j;
    c.ex = intra ? (t < N_HOME ? t : t + (E_ALL - N_HOME)) : t - T_AWAY + N_HOME;
    x_piece(c.ex, b, j);
    const int slot = intra ? mate_slot((jp - jme - 1) & 3, b) : away_slot(jp, b);
    c.src = base + b * BLK + j * 8192; c.dst = base + slot * BLK + j * 8192; c.kind = 0;
    c.fa = a_flag(slot, 2 * j); c.fb = a_flag(slot, 2 * j + 1);
  } else if (t < T_AWAY) {
    c.src = base + B_OFF + jme * 4 * BLK + t * 2048; c.ex = 512 + jme * 128 + t;
    c.dst = base + B_OFF + (4 + jp) * 4 * BLK + t * 2048; c.kind = 1;
    c.fa = c.fb = b_flag(4 + jp, t);
  } else {
    const int o = min((int)(desc & 127u), N_OUT - 1);
    c.src = base + OX_OFF + jme * (1 << 20) + o * 8192; c.ex = 0;
    c.kind = o < 100 ? 2 : 3;
    c.dst = (slot_row(peer, out_slot(jme, o >> 1)) * NS + jp * 128 + (o & 1) * (o < 100 ? 16 * NS : 64)) * 4;
    c.fa = c.fb = 0;
  }
  c.ex = base + EX_OFF + c.ex * EX_PIECE;
  return c;
}

constexpr int NSLOT = 8, SLOT_B = 8192 + 256, F_CTL = NSLOT * SLOT_B;
enum { C_QN = 0, C_FIN = 1, C_ISSUED = 2, C_LANDED = 3, C_DONE = 4, C_ENT = 16 };
constexpr int OOB = 0x7ff00000;
__device__ __forceinline__ unsigned lds_ld(const unsigned* a) {
  return __builtin_amdgcn_readfirstlane(__hip_atomic_load(a, __ATOMIC_RELAXED, __HIP_MEMORY_SCOPE_WORKGROUP));
}
__device__ __forceinline__ void lds_st(unsigned* a, unsigned v) {
  asm volatile("" ::: "memory");
  __hip_atomic_store(a, v, __ATOMIC_RELAXED, __HIP_MEMORY_SCOPE_WORKGROUP);
  asm volatile("" ::: "memory");
}

__device__ __forceinline__ void fetch_schedule(const Params& p, char* ws, unsigned* ctl, unsigned call, int link, int lane) {
  const int rank = __builtin_amdgcn_readfirstlane(p.rank);
  const int peer = rank ^ (link + 1);
  const int np = link < 3 ? NP_INTRA : (__builtin_amdgcn_readfirstlane(p.mode) & 1) ? T_OUT : NP_CROSS;
  const rsrc_t rl = mk(ws, WS_USED);
  const int base = CTRL_BYTES + (call & 1) * BUF_BYTES;
  const int sf = SF_OFF + ((call & 1) * 8 + peer) * SF_SRC;
  const int nsg = link < 3 ? NSI : NSINGLE, v2 = (np - NPUSH - nsg) >> 1, nv = v2 + nsg;
  const unsigned tkbase = (call - 1) * (unsigned)nv - (unsigned)NFW;
  int tko = TK_OFF + link * 128;
  asm volatile("" : "+v"(tko));
  unsigned tk = 0;
  int v = blockIdx.x & 7, n = 0, t = 0;
  bool fail = false;
  while ((unsigned)v < (unsigned)nv && !fail) {
    const int k = v < v2 ? 2 : 1, tf = NPUSH + v + (v < v2 ? max(0, v - SKEW) : v2), tg = v < v2 ? NPUSH + v + min(v2, v + SKEW + 1) : tf;
    const int sfo = sf + sf_pos((lane & 1) ? tg : tf);
    while ((int)lds_ld(ctl + C_ISSUED) + AHEAD <= n || (int)lds_ld(ctl + C_DONE + (n & 1)) + NSLOT <= n ||
           (int)lds_ld(ctl + C_DONE + (~n & 1)) + NSLOT <= n + 1) __builtin_amdgcn_s_sleep(1);
    if (lane == 0) tk = __builtin_amdgcn_raw_ptr_buffer_atomic_add_i32(1, rl, tko, 0, 0);
    unsigned dv = ld_sys(ws, sfo);
    for (int i = 0; i < k; ++i) {
      t = i ? tg : tf;
      unsigned dc = __builtin_amdgcn_readlane(dv, i);
      if (!desc_ok(dc, call)) {
        const unsigned long long ts = __builtin_amdgcn_s_memrealtime();
        for (unsigned it = 1;; ++it) {
          dv = ld_sys(ws, sfo);
          dc = __builtin_amdgcn_readlane(dv, i);
          if (desc_ok(dc, call)) break;
          if ((it & 15) == 0 && (__builtin_amdgcn_readfirstlane(ld_agent(ws, TMO_OFF)) == call ||
                                 __builtin_amdgcn_s_memrealtime() - ts > DEADLINE)) { fail = true; break; }
          __builtin_amdgcn_s_sleep(1);
        }
        if (fail) break;
      }
      if (link < 3 && ((t >= N_HOME - NRI && t < N_HOME) || t >= NP_INTRA - NRL)) dc |= D_RAW;
      const Piece c = piece_of(rank, peer, t, dc, base);
      const bool wide = t >= np - (link < 3 ? NWI : NWC);
      unsigned* e = ctl + C_ENT + (n & (NSLOT - 1)) * 8;
      *(u32x4*)e = u32x4{(unsigned)((dc & D_RAW) ? c.src : c.ex), dc, (unsigned)(c.kind | (wide ? 4 : 0)), (unsigned)c.dst};
      *(u32x4*)(e + 4) = u32x4{(unsigned)c.fa, (unsigned)c.fb, (unsigned)t, 0u};
      ++n;
      if (!wide || i + 1 == k || !desc_ok(__builtin_amdgcn_readlane(dv, 1), call)) lds_st(ctl + C_QN, n);
      if (t < 64) STAMP(ws, 72000 + link * 512 + t, NOW());
    }
    v = (int)(__builtin_amdgcn_readfirstlane(tk) - tkbase);
  }
  lds_st(ctl + C_FIN, n + 1);
  if (fail) {
    st_agent(ws, TMO_OFF, call);
    if (lane == 0) ((float*)p.out[rank])[rank * MS * NS] = __builtin_nanf("");
  }
}

#define DMA(W, op, voff, ldsaddr) do { unsigned keep_; \
  asm volatile("s_mov_b32 %0, m0\n\ts_mov_b32 m0, %2\n\ts_waitcnt vmcnt(%4)\n\t" op " %1, %3, 0 offen sc0 sc1 lds\n\t" \
               "s_mov_b32 m0, %0" : "=&s"(keep_) : "v"(voff), "s"(ldsaddr), "s"(rp), "i"((W) - 1) : "memory"); } while (0)
__device__ __forceinline__ void fetch_load(const Params& p, char* lds, unsigned* ctl, int link, int lane) {
  const rsrc_t rp = mk(p.ws[p.rank ^ (link + 1)], WS_USED);
  const unsigned ring = (unsigned)(unsigned long long)(__attribute__((address_space(3))) char*)lds;
  const int lx = lane * 16, lw = w_lane_off(lane);
  unsigned q = 0;
  u32x4 e = {0, 0, 0, 0};
  int pub = 0;
  for (int n = 0;; ++n) {
    if ((int)__builtin_amdgcn_readfirstlane(q) <= n) {
      if ((int)lds_ld(ctl + C_QN) <= n) {
        if (pub < n - 1) {
          asm volatile("s_waitcnt vmcnt(8)" ::: "memory");
          lds_st(ctl + C_LANDED, n - 1);
        }
        DRAIN();
        pub = n;
        lds_st(ctl + C_LANDED, n);
        bool end = false;
        while ((int)lds_ld(ctl + C_QN) <= n) {
          if ((int)lds_ld(ctl + C_FIN) == n + 1) { end = true; break; }
          __builtin_amdgcn_s_sleep(1);
        }
        if (end) break;
      }
      asm volatile("" ::: "memory");
      e = *(const u32x4*)(ctl + C_ENT + (n & (NSLOT - 1)) * 8);
    }
    lds_st(ctl + C_ISSUED, n + 1);
    const unsigned dc = __builtin_amdgcn_readfirstlane(e[1]);
    asm volatile("" :: "v"(e));
    const unsigned kind = __builtin_amdgcn_readfirstlane(e[2]);
    const bool raw = dc & D_RAW, wraw = raw && (kind & 3) == 1;
    const int src = __builtin_amdgcn_readfirstlane(e[0]);
    const int a0 = (wraw ? lw : lx) + src, st = wraw ? 128 : 1024, hb = wraw ? 1024 : 4096;
    const unsigned slot = ring + (n & (NSLOT - 1)) * SLOT_B;
#define PIECE(W) _Pragma("unroll") for (int j = 0; j < 8; ++j) { \
      if (j < 7 || raw) DMA(W, "buffer_load_dwordx4", a0 + (j >> 2) * hb + (j & 3) * st, slot + j * 1024); \
      else DMA(W, "buffer_load_dword", (lane == 0 || 2 * lane < (int)((dc >> 5) & 255u)) ? src + 7168 + lane * 4 : OOB, slot + 8192); \
      if (j == ((W) - 1) % 8) {                      \
        const int l = n - ((W) - 1) / 8; \
        if (l > pub) { pub = l; lds_st(ctl + C_LANDED, l); } \
      } \
      if (j == 4) {                                  \
        q = __hip_atomic_load(ctl + C_QN, __ATOMIC_RELAXED, __HIP_MEMORY_SCOPE_WORKGROUP); \
        asm volatile("" ::: "memory"); \
        e = *(const u32x4*)(ctl + C_ENT + ((n + 1) & (NSLOT - 1)) * 8); \
      } }
    if (kind & 4) { PIECE(WWIN) } else { PIECE(WIN) }
#undef PIECE
  }
}

__device__ __forceinline__ void fetch_work(const Params& p, char* ws, char* lds, unsigned* ctl, unsigned call, int link, int k, int lane, unsigned long long tl0) {
  const rsrc_t rl = mk(ws, WS_USED), ro = mk(p.out[p.rank], OUT_BYTES);
  const int lx = lane * 16, lw = w_lane_off(lane), lo = (lane >> 5) * (NS * 4) + (lane & 31) * 16,
            lo3 = (lane >> 4) * (NS * 4) + (lane & 15) * 16;
  for (int n = k;; n += 2) {
    bool end = false;
    while ((int)lds_ld(ctl + C_LANDED) <= n) {
      const int fin = (int)lds_ld(ctl + C_FIN);
      if (fin != 0 && fin - 1 <= n) { end = true; break; }
      __builtin_amdgcn_s_sleep(1);
    }
    if (end) break;
    asm volatile("" ::: "memory");
    char* img = lds + (n & (NSLOT - 1)) * SLOT_B;
    const u32x4 e0 = *(const u32x4*)(ctl + C_ENT + (n & (NSLOT - 1)) * 8), e1 = *(const u32x4*)(ctl + C_ENT + (n & (NSLOT - 1)) * 8 + 4);
    u32x4 r[8];
#pragma unroll
    for (int i = 0; i < 8; ++i) r[i] = *(const u32x4*)(img + i * 1024 + lane * 16);
    const unsigned pt = *(const unsigned*)(img + 8192 + lane * 4);
    asm volatile("" ::: "memory");
    const unsigned dc = __builtin_amdgcn_readfirstlane(e0[1]);
    const int kind = __builtin_amdgcn_readfirstlane(e0[2]) & 3;
    if (!(dc & D_RAW)) unpack_piece(r, dc, 2 * lane < (int)((dc >> 5) & 255u) ? pt : 0u, img, lane);
    const int d0 = (kind == 0 ? lx : kind == 1 ? lw : kind == 2 ? lo : lo3) + (int)e0[3];
    const int dst_st = kind == 0 ? 1024 : kind == 1 ? 128 : kind == 2 ? 2 * NS * 4 : 4 * NS * 4, dst_hb = kind == 1 ? 1024 : 4 * dst_st;
    const rsrc_t rd = kind >= 2 ? ro : rl;
#pragma unroll
    for (int i = 0; i < 8; ++i)
      __builtin_amdgcn_raw_buffer_store_b128(r[i], rd, d0 + (i >> 2) * dst_hb + (i & 3) * dst_st, 0, AUX_SYS);
    lds_st(ctl + C_DONE + k, n + 1);
    DRAIN();
    if (lane < 2 && kind < 2) st_sys(ws, (call & 1) * FLG_PAR + (lane ? e1[1] : e1[0]), call << 4 | ((dc >> 14) & 15u));
    STAMP(ws, link * 512 + e1[2], NOW());
    ACC(ws, link * 512 + e1[2], tl0);
  }
}

struct Ready { int na, nb; unsigned f; };
template <int SLEEP, bool ISA, int SLEEP0 = 16>
__device__ __forceinline__ void wait_ready(void* ws, int off, unsigned call, int need, Ready& r) {
  unsigned long long t0 = __builtin_amdgcn_s_memrealtime();
  for (unsigned it = 1;; ++it) {
    r.f = ld_sys(ws, off);
    const unsigned long long m = __ballot(r.f >= call << 4);
    r.na = __builtin_ctz(~((unsigned)(m & (m >> 16)) & 0xffffu));
    r.nb = __builtin_ctz(~((unsigned)((m >> 32) & (m >> 48)) & 0xffffu));
    if ((ISA ? r.na : r.nb) > need) break;
    if ((it & 15) == 0 && (ld_agent(ws, TMO_OFF) == call || __builtin_amdgcn_s_memrealtime() - t0 > DEADLINE)) {
      st_agent(ws, TMO_OFF, call);
      r.na = r.nb = 17;
      break;
    }
    if (need == 0) __builtin_amdgcn_s_sleep(SLEEP0); else if (need < 12) __builtin_amdgcn_s_sleep(SLEEP); else __builtin_amdgcn_s_sleep(SL2);
  }
}
__device__ __forceinline__ int scale_of(unsigned f, int lane) { return (int)((unsigned)__builtin_amdgcn_readlane(f, lane) << 28) >> 24; }

#define MMA(B, A, C) __builtin_amdgcn_mfma_f32_32x32x16_f16(__builtin_bit_cast(f16x8, B), __builtin_bit_cast(f16x8, A), C, 0, 0, 0)
template <int NB>
__device__ __forceinline__ void unscale_partials(rsrc_t rs, unsigned f, float* dst, int mb, int a0, int a1, int b0, int lane) {
  constexpr int LD = NB * 32 + 4;
  const unsigned lead = __shfl(f, lane < 32 ? lane & 16 : 32);
  const bool uniform = __ballot(((f ^ lead) & 15u) != 0) == 0;
#pragma clang loop unroll(disable)
  for (int mn = 0; mn < mb * NB; ++mn) {
    {
      const int m = mn / NB, n = mn % NB;
      float* d = dst + m * 32 * LD + n * 32;
      if (uniform) {
        const int sh = -scale_of(f, 16 * m) - scale_of(f, 32);
        for (int i = 0; i < 16; ++i) d[i] = __builtin_amdgcn_ldexpf(d[i], sh);
      } else {
        f32x16 acc = {0, 0, 0, 0, 0, 0, 0, 0, 0, 0, 0, 0, 0, 0, 0, 0};
#pragma clang loop unroll(disable)
        for (int kc = 0; kc < 16; ++kc) {
          u32x4 a[4], b[4];
#pragma unroll
          for (int s = 0; s < 4; ++s) {
            a[s] = __builtin_amdgcn_raw_buffer_load_b128(rs, lane * 16 + s * 1024 + (m ? a1 : a0) + kc * 16384, 0, AUX_SYS);
            b[s] = __builtin_amdgcn_raw_buffer_load_b128(rs, lane * 16 + s * 1024 + b0 + n * BLK + kc * 16384, 0, AUX_SYS);
          }
#pragma unroll
          for (int hh = 0; hh < 2; ++hh) {
            const f32x16 z = {0, 0, 0, 0, 0, 0, 0, 0, 0, 0, 0, 0, 0, 0, 0, 0};
            const f32x16 t = MMA(b[2 * hh + 1], a[2 * hh + 1], MMA(b[2 * hh], a[2 * hh], z));
            const int sh = -scale_of(f, 16 * m + kc) - scale_of(f, 32 + 16 * hh + kc);
#pragma unroll
            for (int r = 0; r < 16; ++r) acc[r] += __builtin_amdgcn_ldexpf(t[r], sh);
          }
        }
#pragma unroll
        for (int r = 0; r < 16; ++r) d[(r & 3) * 4 + (r >> 2)] = acc[r];
      }
    }
  }
}

template <int MB, int NB, int D, int PRE>
__device__ __forceinline__ void gemm_tile(const Params& p, char* ws, unsigned call, float* lds, int slot0, int slot1, int qslot,
                                          int nt, int ob, int stamp_base) {
  const int lane = threadIdx.x & 63;
  const int w = __builtin_amdgcn_readfirstlane(threadIdx.x >> 6);
  const rsrc_t rs = mk(ws, WS_USED);
  const int base = CTRL_BYTES + (call & 1) * BUF_BYTES;
  const int abase[2] = {base + slot0 * BLK + w * 4096, base + slot1 * BLK + w * 4096};
  const int bbase = base + B_OFF + (qslot * 4 + nt * NB) * BLK + w * 4096;
  const int foff = (call & 1) * FLG_PAR + (lane < 32 ? FA_OFF + ((lane & 16) ? slot1 : slot0) * 256 + w * 64 + (lane & 15) * 4
                             : FB_OFF + qslot * 512 + w * 128 + (lane & 31) * 4);
  const int voff = lane * 16;

  f32x16 acc[MB][NB];
#pragma unroll
  for (int m = 0; m < MB; ++m)
#pragma unroll
    for (int n = 0; n < NB; ++n)
#pragma unroll
      for (int r = 0; r < 16; ++r) acc[m][n][r] = 0.f;
  u32x4 fa[D][4][MB], fb[D][4][NB];

#define ISSUE_B(FB, kc, FROM_LDS) do { \
    if (FROM_LDS) { \
      _Pragma("unroll") for (int s = 0; s < 4; ++s) \
        _Pragma("unroll") for (int n = 0; n < NB; ++n) \
          FB[s][n] = *(const u32x4*)(pre + (((kc) - (16 - PRE)) * 4 * NB + s * NB + n) * 1024); \
    } else { \
      if (!PRE && rd.nb <= (kc)) wait_ready<(MB == 2 ? 8 : SL1), false>(ws, foff, call, (kc), rd); \
      asm volatile("" ::: "memory"); \
      _Pragma("unroll") for (int s = 0; s < 4; ++s) \
        _Pragma("unroll") for (int n = 0; n < NB; ++n) \
          FB[s][n] = __builtin_amdgcn_raw_buffer_load_b128(rs, voff + s * 1024, bbase + n * BLK + (kc) * 16384, AUX_SYS); \
    } \
    __builtin_amdgcn_sched_barrier(0); } while (0)
#define ISSUE_A(FA, kc) do { \
    if (rd.na <= (kc)) wait_ready<(MB == 2 ? 8 : SL1), true, (PRE ? 4 : 16)>(ws, foff, call, (kc), rd); \
    asm volatile("" ::: "memory"); \
    STAMP(ws, stamp_base + w * 32 + (kc), NOW()); \
    _Pragma("unroll") for (int s = 0; s < 4; ++s) \
      _Pragma("unroll") for (int m = 0; m < MB; ++m) \
        FA[s][m] = __builtin_amdgcn_raw_buffer_load_b128(rs, voff + s * 1024, abase[m] + (kc) * 16384, AUX_SYS); \
    __builtin_amdgcn_sched_barrier(0); } while (0)
#define ISSUE(FA, FB, kc, FROM_LDS) do { ISSUE_B(FB, kc, FROM_LDS); ISSUE_A(FA, kc); } while (0)
#define COMPUTE(FA, FB) do { \
    _Pragma("unroll") for (int s = 0; s < 4; ++s) \
      _Pragma("unroll") for (int m = 0; m < MB; ++m) \
        _Pragma("unroll") for (int n = 0; n < NB; ++n) acc[m][n] = MMA(FB[s][n], FA[s][m], acc[m][n]); \
    __builtin_amdgcn_sched_barrier(0); } while (0)

  Ready rd = {0, 0, 0u};
  STAMP(ws, stamp_base + w * 32 + 16, NOW());
  constexpr int PSTR = PRE ? LDS_RED / 16 : MB * 32 * (NB * 32 + 4);
  static_assert(PRE % D == 0 && PRE * 4 * NB * 1024 <= LDS_RED / 4, "LDS region of a wave");
  char* const pre = (char*)(lds + w * PSTR) + lane * 16;
  if (PRE) {
    wait_ready<8, false>(ws, foff, call, 15, rd);
    asm volatile("" ::: "memory");
#pragma clang loop unroll(disable)
    for (int kc = 16 - PRE; kc < 16; ++kc) {
#pragma unroll
      for (int i = 0; i < 4 * NB; ++i)
        *(u32x4*)(pre + ((kc - (16 - PRE)) * 4 * NB + i) * 1024) = __builtin_amdgcn_raw_buffer_load_b128(
            rs, voff + (i / NB) * 1024, bbase + (i % NB) * BLK + kc * 16384, AUX_SYS);
    }
  }
#pragma unroll
  for (int d = 0; d < D; ++d) { ISSUE_B(fb[d], d, false); if (!PRE) ISSUE_A(fa[d], d); }
  if (PRE) {
#pragma unroll
    for (int d = 0; d < D; ++d) ISSUE_A(fa[d], d);
  }
#pragma clang loop unroll(disable)
  for (int kc = D; kc < 16 - PRE; kc += D) {
#pragma unroll
    for (int d = 0; d < D; ++d) {
      COMPUTE(fa[d], fb[d]);
      ISSUE(fa[d], fb[d], kc + d, false);
    }
  }
  if (PRE) {
#pragma clang loop unroll(disable)
    for (int kc = 16 - PRE; kc < 16; kc += D) {
#pragma unroll
      for (int d = 0; d < D; ++d) {
        COMPUTE(fa[d], fb[d]);
        ISSUE(fa[d], fb[d], kc + d, true);
      }
    }
  }
  const unsigned tmo = ld_agent(ws, TMO_OFF);
#pragma unroll
  for (int d = 0; d < D; ++d) COMPUTE(fa[d], fb[d]);
#undef ISSUE
#undef ISSUE_A
#undef ISSUE_B
#undef COMPUTE
  STAMP(ws, stamp_base + w * 32 + 17, NOW());

  constexpr int LD = NB * 32 + 4;
  const bool remote = qslot >= 4;
  unsigned oq = 0;
  if (remote && threadIdx.x == 0) {
    int oo = OQ_OFF + (qslot - 4) * 128;
    asm volatile("" : "+v"(oo));
    oq = __builtin_amdgcn_raw_ptr_buffer_atomic_add_i32(MB * NB / 2, rs, oo, 0, 0);
  }
  {
    const bool bad = rd.na > 16 || tmo == call;
    float* dst = lds + w * PSTR + (lane & 31) * LD + (lane >> 5) * 16;
#pragma unroll
    for (int m = 0; m < MB; ++m)
#pragma unroll
      for (int n = 0; n < NB; ++n)
#pragma unroll
        for (int t = 0; t < 4; ++t) {
          f32x4 o = {acc[m][n][t], acc[m][n][4 + t], acc[m][n][8 + t], acc[m][n][12 + t]};
          *(f32x4*)(dst + m * 32 * LD + n * 32 + t * 4) = o;
        }
    if (__ballot((rd.f & 15u) != 0) != 0) unscale_partials<NB>(rs, rd.f, dst, MB, abase[0], abase[1], bbase, lane);
    if (bad) {
      const float q = __builtin_nanf("");
      for (int m = 0; m < MB; ++m)
        for (int i = 0; i < NB * 4; ++i) *(f32x4*)(dst + m * 32 * LD + (i >> 2) * 32 + (i & 3) * 4) = f32x4{q, q, q, q};
    }
  }
  __syncthreads();
  const rsrc_t ro = remote ? rs : mk(p.out[p.rank], OUT_BYTES);
  constexpr int LPR = NB * 8;
  constexpr int RPI = 64 / LPR;
  constexpr int RPW = MB * 8;
  const int grow = slot_row(p.rank, (w * RPW) >> 5 ? slot1 : slot0) + ((w * RPW) & 31);
  const int o0 = remote ? base + OX_OFF + (qslot - 4) * (1 << 20) + (2 * ob + nt) * 8192 + w * (RPW * NB * 128) + lane * 16
                        : ((grow + lane / LPR) * NS + qslot * 128 + nt * NB * 32) * 4 + (lane % LPR) * 16;
  const int ost = remote ? 1024 : RPI * NS * 4;
#pragma unroll
  for (int i = 0; i < RPW / RPI; ++i) {
    const int row = w * RPW + i * RPI + lane / LPR, c4 = (lane % LPR) * 4;
    f32x4 s = *(const f32x4*)(lds + row * LD + c4);
#pragma unroll
    for (int k = 1; k < 4; ++k) s += *(const f32x4*)(lds + k * PSTR + row * LD + c4);
    f32x4 o;
#pragma unroll
    for (int e = 0; e < 4; ++e) o[e] = silu(s[e]);
    __builtin_amdgcn_raw_buffer_store_b128(__builtin_bit_cast(u32x4, o), ro, o0 + i * ost, 0, AUX_SYS);
  }
  DRAIN();
  __syncthreads();
  if (remote && w == 0) {
    const unsigned at = __builtin_amdgcn_readfirstlane(oq) - (call - 1) * (unsigned)N_OUT + lane;
    if (lane < MB * NB / 2 && at < (unsigned)N_OUT)
      st_sys(p.ws[((p.rank >> 2) ^ 1) * 4 + (qslot - 4)], SF_OFF + ((call & 1) * 8 + p.rank) * SF_SRC + sf_pos(T_OUT + at),
             call << 18 | D_RAW | (2 * ob + nt + lane));
  }
  STAMP(ws, stamp_base + w * 32 + 18, NOW());
}

__global__ void __launch_bounds__(256, 1) ag_gemm(Params p) {
  extern __shared__ __attribute__((aligned(16))) float lds[];
  const int lane = threadIdx.x & 63;
  const int w = __builtin_amdgcn_readfirstlane(threadIdx.x >> 6);
  const int xcd = blockIdx.x & 7, idx = blockIdx.x >> 3;
  char* const ws = uni(p.ws[p.rank]);
  if (p.mode & 2) return;
  STAMP(ws, 65536 + blockIdx.x * 4 + w, NOW());
  const unsigned long long tl0 = 0;

  if (idx < 7) {
    const int link = (idx + xcd) % 7;
    unsigned* ctl = (unsigned*)((char*)lds + F_CTL);
    unsigned call = 0;
    if (w == 0) {
      if (lane < 16) ctl[lane] = 0;
      asm volatile("s_waitcnt lgkmcnt(0)\n\ts_barrier" ::: "memory");
      call = __builtin_amdgcn_readfirstlane(ld_agent(ws, CNT_OFF + blockIdx.x * 128)) + 1;
      if (lane == 0) st_agent(ws, CNT_OFF + blockIdx.x * 128, call);
      fetch_schedule(p, ws, ctl, call, link, lane);
    } else if (w == 1) {
      asm volatile("s_barrier" ::: "memory");
      fetch_load(p, (char*)lds, ctl, link, lane);
    } else {
      asm volatile("s_barrier" ::: "memory");
      call = __builtin_amdgcn_readfirstlane(ld_agent(ws, CNT_OFF + blockIdx.x * 128 + w * 4)) + 1;
      if (lane == 0) st_agent(ws, CNT_OFF + blockIdx.x * 128 + w * 4, call);
      fetch_work(p, ws, (char*)lds, ctl, call, link, w - 2, lane, tl0);
    }
  } else {
    unsigned call = 0;
    {
      const int g = blockIdx.x - 56;
      constexpr int A0 = NPUSH / 4, A1 = (NPUSH + NE) / 4, NG = 32 - NE / 4, NTASK = 5 * NG + 96;
      auto task_piece = [](int o) {
        if (o < 5 * NG) {
          int a = o / 5; const int r = o - a * 5;
          a = a < 8 ? a + A1 : a < 8 + A0 ? a - 8 : a + NE / 4;
          return r == 0 ? a * 4 : 384 + r * 128 + a * 4;
        }
        const int r = o - 5 * NG;
        return r < 80 ? ((r & 1) ? N_HOME + (r >> 1) * 4 : (32 + (r >> 1)) * 4) : (r - 8) * 4;
      };
      const int ei = g < 80 ? (w == 0 ? g : w == 2 ? g + 80 : -1) : (w == 0 && g + 80 < 5 * NE ? g + 80 : -1);
      const int pk = g - (200 - 7 * NPUSH);
      if (w == 1 && pk >= 0) call = push_first(p, ws, (char*)lds + LDS_WAVE, pk % 7, pk / 7, w, lane);
      if (ei < 0) for (int i = call ? 64 : -P0; i < g; ++i) __builtin_amdgcn_s_sleep(PACE);
      else for (int i = EP0; i < ei; i += EPS) __builtin_amdgcn_s_sleep(1);
      const int r1 = g >= NTASK ? 1 : g >= 400 - NTASK ? 3 : 2;
#pragma clang loop unroll(disable)
      for (int r = ei >= 0 ? 0 : 1; r < r1; ++r) {
        const int s5 = ei % 5;
        const int e = r == 0 ? (s5 == 0 ? 0 : 384 + s5 * 128) + NPUSH + ei / 5 : task_piece(r == 1 ? g : g + NTASK - 200) + w;
        convert(p, ws, (char*)lds, e, r != 0, w, lane, call, r);
      }
      if (call == 0) {
        call = __builtin_amdgcn_readfirstlane(ld_agent(ws, CNT_OFF + blockIdx.x * 128 + w * 4)) + 1;
        if (lane == 0) st_agent(ws, CNT_OFF + blockIdx.x * 128 + w * 4, call);
      }
      STAMP(ws, 65536 + 3072 + g * 4 + w, NOW());
    }
    if (!(p.mode & 1)) {
      const int gi = idx - 7;
      const bool own = xcd < 4;
      __syncthreads();
#pragma clang loop unroll(disable)
      for (int ob = 2 * gi; ob < (own ? 58 : 50); ob += 50) {
        gemm_tile<2, 4, 2, 0>(p, ws, call, lds, own ? own_slot(ob) : out_slot(xcd - 4, ob), own ? own_slot(ob + 1) : out_slot(xcd - 4, ob + 1),
                        xcd, 0, ob, 4096 + ((ob >> 1) * 8 + xcd) * 128);
        if (w == 0) ACC(ws, 4096 + xcd * 128 + ob, tl0);
      }
      const int sb = 4096 + 36864 + (gi * 8 + xcd) * 128 * (gi < 23);
      const int k2 = own ? gi - 9 : gi;
      if (own && gi >= 4 && gi < 9) {
        const int s0 = own_slot(54 + gi);
        gemm_tile<1, 4, 2, 0>(p, ws, call, lds, s0, s0, xcd, 0, 0, sb);
        if (w == 0) ACC(ws, 4096 + xcd * 128 + 54 + gi, tl0);
      } else if (k2 >= 0 && k2 < 12) {
        const int s0 = out_slot(0, 50 + (k2 >> 1));
        gemm_tile<1, 2, D2, PR2>(p, ws, call, lds, s0, s0, xcd, k2 & 1, 50 + (k2 >> 1), sb);
        if (w == 0) ACC(ws, 4096 + xcd * 128 + 64 + k2, tl0);
      } else if (!own && gi < 24) {
        const int s0 = mate_slot((gi - 12) >> 2, 15);
        gemm_tile<1, 1, D1, PR1>(p, ws, call, lds, s0, s0, xcd & 3, gi & 3, 0, sb);
        if (w == 0) ACC(ws, 4096 + xcd * 128 + 64 + gi, tl0);
      }
    }
  }
  STAMP(ws, 65536 + 1024 + blockIdx.x * 4 + w, NOW());
}

}

#define AG_MODE 0

extern "C" void kernel_launch(int rank, int nranks, ncclComm_t comm,
                              hipStream_t stream,
                              void* const* d_in, const int* in_sizes, int n_in,
                              void* d_out, int out_size,
                              const int* in_dtypes, int out_dtype,
                              void* const* d_out_all,
                              void* const* d_ws_all, size_t ws_bytes) {
  using namespace agk;
  if (nranks != NR || (unsigned)rank >= (unsigned)NR || n_in < 2 || in_sizes[0] != MS * KD || in_sizes[1] != KD * NS ||
      out_size != 4096 * NS || ws_bytes < (size_t)WS_USED) {
    fprintf(stderr, "kernel_launch: unexpected problem shape\n");
    return;
  }
  HIP_CHECK(hipFuncSetAttribute((const void*)ag_gemm, hipFuncAttributeMaxDynamicSharedMemorySize, LDS_BYTES));
  Params p;
  p.x = (const float*)d_in[0];
  p.w = (const float*)d_in[1];
  for (int i = 0; i < NR; ++i) { p.out[i] = d_out_all[i]; p.ws[i] = d_ws_all[i]; }
  p.rank = rank;
  p.mode = AG_MODE;
  hipLaunchKernelGGL(ag_gemm, dim3(256), dim3(256), LDS_BYTES, stream, p);
  HIP_CHECK(hipGetLastError());
}
```

```cpp
#include <hip/hip_runtime.h>
#include <rccl/rccl.h>
#include <cstdio>

#define HIP_CHECK(expr) do { hipError_t _e = (expr); if (_e != hipSuccess) { \
    fprintf(stderr, "HIP error %s at %s:%d\n", hipGetErrorString(_e), __FILE__, __LINE__); \
    return; } } while (0)

namespace agk {

typedef unsigned u32x4 __attribute__((ext_vector_type(4)));
typedef float f32x4 __attribute__((ext_vector_type(4)));
typedef float f32x16 __attribute__((ext_vector_type(16)));
typedef _Float16 f16x8 __attribute__((ext_vector_type(8)));
typedef __attribute__((address_space(1))) unsigned gu32;
typedef __amdgpu_buffer_rsrc_t rsrc_t;

constexpr int NR = 8, MS = 512, NS = 512, KD = 4096;
constexpr int BLK = 32 * KD * 2;
constexpr int A_SLOTS = 72;
constexpr int A_BYTES = A_SLOTS * BLK, B_BYTES = 32 * BLK, OX_BYTES = 4 << 20;
constexpr int EX_PIECE = 7168 + 256, EX_BYTES = 8 << 20;
constexpr int B_OFF = A_BYTES, OX_OFF = B_OFF + B_BYTES, EX_OFF = OX_OFF + OX_BYTES, BUF_BYTES = EX_OFF + EX_BYTES;
static_assert(1024 * EX_PIECE <= EX_BYTES, "export buffer");
constexpr int CNT_OFF = 0;
constexpr int TMO_OFF = 32768;
constexpr int TK_OFF = 33024;
constexpr int OQ_OFF = 34816;
constexpr int FA_OFF = 65536, FLG_PAR = 65536;
constexpr int FB_OFF = FA_OFF + A_SLOTS * 512;
constexpr int SF_OFF = 262144, SF_SRC = 32768;
constexpr int CTRL_BYTES = 1 << 20;
static_assert(FB_OFF + 4096 <= FA_OFF + FLG_PAR && FA_OFF + 2 * FLG_PAR <= SF_OFF && SF_OFF + 16 * SF_SRC <= CTRL_BYTES, "control block");
constexpr int WS_USED = CTRL_BYTES + 2 * BUF_BYTES;
constexpr int OUT_BYTES = 4096 * NS * 4;
constexpr int IN_BYTES = MS * KD * 4;
constexpr int AUX_SYS = 17;
constexpr int RSRC_W3 = 0x00020000;
constexpr unsigned long long DEADLINE = 30000000ull;
constexpr int LDS_RED = 4 * 128 * 68 * 4, LDS_BYTES = LDS_RED + 64;
constexpr int LDS_WAVE = 16 * 528 + 512;

constexpr int NPUSH = 16;
constexpr int NR0 = 24;
constexpr int PWIN = 2;
constexpr int WIN = 6;
constexpr int NSINGLE = 12;
constexpr int AHEAD = 2;
constexpr int NE = 56;
constexpr int P0 = 40;
constexpr int EP0 = 60;
constexpr int EPS = 8;
constexpr int NRI = 8;
constexpr int NSI = 24;
constexpr int D2 = 4;
constexpr int D1 = 4;
constexpr int PR2 = 4;
constexpr int PR1 = 8;
constexpr int SL1 = 2;
constexpr int SL2 = 1;
constexpr int NWC = 12;
constexpr int WWIN = 16;
constexpr int NFW = 8;
constexpr int N_HOME = 352, E_ALL = 480, NP_INTRA = 384, T_AWAY = 128, T_OUT = 288, N_OUT = 112, NP_CROSS = T_OUT + N_OUT;
constexpr unsigned D_RAW = 1u << 13;
__device__ __forceinline__ bool desc_ok(unsigned d, unsigned call) { return (d >> 18) == (call & 0x3fffu); }

struct Params {
  const float* x;
  const float* w;
  float* out;
  void* ws[NR];
  int rank;
  int pad;
};


__device__ __forceinline__ char* uni(const void* p) {
  const unsigned long long a = (unsigned long long)p;
  const unsigned lo = __builtin_amdgcn_readfirstlane((unsigned)a), hi = __builtin_amdgcn_readfirstlane((unsigned)(a >> 32));
  return (char*)(((unsigned long long)hi << 32) | lo);
}
__device__ __forceinline__ rsrc_t mk(const void* p, int bytes) {
  return __builtin_amdgcn_make_buffer_rsrc((void*)uni(p), 0, bytes, RSRC_W3);
}
__device__ __forceinline__ unsigned ld_sys(const void* base, int off) {
  return __hip_atomic_load((gu32*)((char*)base + off), __ATOMIC_RELAXED, __HIP_MEMORY_SCOPE_SYSTEM);
}
__device__ __forceinline__ void st_sys(void* base, int off, unsigned v) {
  __hip_atomic_store((gu32*)((char*)base + off), v, __ATOMIC_RELAXED, __HIP_MEMORY_SCOPE_SYSTEM);
}
__device__ __forceinline__ unsigned ld_agent(const void* base, int off) {
  return __hip_atomic_load((gu32*)((char*)base + off), __ATOMIC_RELAXED, __HIP_MEMORY_SCOPE_AGENT);
}
__device__ __forceinline__ void st_agent(void* base, int off, unsigned v) {
  __hip_atomic_store((gu32*)((char*)base + off), v, __ATOMIC_RELAXED, __HIP_MEMORY_SCOPE_AGENT);
}
#define DRAIN() asm volatile("s_waitcnt vmcnt(0)" ::: "memory")

__device__ __forceinline__ u32x4 cvt8(f32x4 a, f32x4 b) {
  f16x8 h;
  h[0] = (_Float16)a[0]; h[1] = (_Float16)a[1]; h[2] = (_Float16)a[2]; h[3] = (_Float16)a[3];
  h[4] = (_Float16)b[0]; h[5] = (_Float16)b[1]; h[6] = (_Float16)b[2]; h[7] = (_Float16)b[3];
  return __builtin_bit_cast(u32x4, h);
}

__device__ __forceinline__ float silu(float y) {
  const float t = fminf(fmaxf(y, -60.f), 60.f);
  return y * __builtin_amdgcn_rcpf(1.f + __expf(-t));
}

__device__ __forceinline__ int slot_row(int rank, int slot) {
  const int grp = rank >> 2, j = rank & 3;
  int src, b;
  if (slot < 16) { src = rank; b = slot; }
  else if (slot < 52) { const int r = (slot - 16) / 12; src = grp * 4 + ((j + 1 + r) & 3); b = slot - 16 - r * 12; b = b == 11 ? 15 : b; }
  else { const int i = (slot - 52) / 5; src = (grp ^ 1) * 4 + i; b = 11 + slot - 52 - i * 5; }
  return src * MS + b * 32;
}
__device__ __forceinline__ int mate_slot(int r, int b) { return 16 + r * 12 + (b == 15 ? 11 : b); }
__device__ __forceinline__ int away_slot(int i, int b) { return 52 + i * 5 + (b - 11); }
__device__ __forceinline__ int out_slot(int ow, int ob) {
  if (ob < 11) return ob;
  if (ob < 26) { const int r = (ob - 11) / 5; return mate_slot(r, ob - 11 - r * 5); }
  if (ob < 38) return away_slot((ow + 1 + ((ob - 26) >> 2)) & 3, 11 + ((ob - 26) & 3));
  if (ob < 50) return mate_slot((ob - 38) >> 2, 5 + ((ob - 38) & 3));
  return mate_slot((ob - 50) >> 1, 9 + (ob & 1));
}
__device__ __forceinline__ int own_slot(int i) {
  if (i < 16) return i;
  if (i < 31) return out_slot(0, i - 5);
  if (i < 47) return away_slot((i - 31) >> 2, 11 + ((i - 31) & 3));
  if (i < 59) return out_slot(0, i - 9);
  if (i < 63) return away_slot(i - 59, 15);
  if (i < 69) return out_slot(0, i - 13);
  return mate_slot(i - 69, 15);
}
__device__ __forceinline__ int sf_pos(int t) {
  return t < NPUSH + NE ? 16384 + (t >> 1) * 128 + (t & 1) * 4 : (t >> 2) * 128 + (t & 3) * 4;
}
__device__ __forceinline__ int a_flag(int slot, int u) { return FA_OFF + slot * 512 + (u & 3) * 128 + (u >> 2) * 4; }
__device__ __forceinline__ int b_flag(int qslot, int h) { return FB_OFF + qslot * 512 + ((h >> 1) & 3) * 128 + (h & 1) * 64 + (h >> 3) * 4; }

__device__ __forceinline__ void x_piece(int e, int& b, int& j) {
  if (e < 160) { j = e / 5; b = e - j * 5; }
  else if (e < 288) { b = 5 + (e & 3); j = (e - 160) >> 2; }
  else if (e < N_HOME) { b = 9 + (e & 1); j = (e - 288) >> 1; }
  else if (e < E_ALL) { b = 11 + (e & 3); j = (e - N_HOME) >> 2; }
  else { b = 15; j = e - E_ALL; }
}
__device__ __forceinline__ void x_load(f32x4 (&v)[16], rsrc_t rx, int lane, int b, int j) {
  const int voff = (lane >> 5) * (KD * 4) + (lane & 31) * 16, soff = b * (32 * KD * 4) + j * 512;
#pragma unroll
  for (int i = 0; i < 16; ++i)
    v[i] = __builtin_bit_cast(f32x4, __builtin_amdgcn_raw_buffer_load_b128(rx, voff, soff + i * (2 * KD * 4), 0));
}
__device__ __forceinline__ unsigned scale_piece(f32x4 (&v)[16]) {
  float m = 0.f;
#pragma unroll
  for (int i = 0; i < 16; ++i)
    m = fmaxf(fmaxf(m, fmaxf(fabsf(v[i][0]), fabsf(v[i][1]))), fmaxf(fabsf(v[i][2]), fabsf(v[i][3])));
  unsigned u = __builtin_bit_cast(unsigned, m);
  u = max(u, (unsigned)__builtin_amdgcn_update_dpp(0u, u, 0x111, 0xf, 0xf, false));
  u = max(u, (unsigned)__builtin_amdgcn_update_dpp(0u, u, 0x112, 0xf, 0xf, false));
  u = max(u, (unsigned)__builtin_amdgcn_update_dpp(0u, u, 0x114, 0xf, 0xf, false));
  u = max(u, (unsigned)__builtin_amdgcn_update_dpp(0u, u, 0x118, 0xf, 0xf, false));
  u = max(u, (unsigned)__builtin_amdgcn_update_dpp(0u, u, 0x142, 0xa, 0xf, false));
  u = max(u, (unsigned)__builtin_amdgcn_update_dpp(0u, u, 0x143, 0xc, 0xf, false));
  const unsigned top = __builtin_amdgcn_readlane(u, 63);
  const int e = top >> 23;
  if (top == 0 || e == 255 || (e >= 117 && e <= 141)) return 0;
  const int z = min((141 - e) >> 4, 7);
#pragma unroll
  for (int i = 0; i < 16; ++i)
#pragma unroll
    for (int k = 0; k < 4; ++k) v[i][k] = __builtin_amdgcn_ldexpf(v[i][k], 16 * z);
  return z & 15;
}
__device__ __forceinline__ void x_frags(u32x4 (&f)[8], const f32x4 (&v)[16], char* lw, int lane) {
  typedef _Float16 f16x4 __attribute__((ext_vector_type(4)));
  char* wp = lw + ((lane & 31) >> 1) * 528 + (lane >> 5) * 16 + (lane & 1) * 8;
#pragma unroll
  for (int i = 0; i < 16; ++i) {
    f16x4 h = {(_Float16)v[i][0], (_Float16)v[i][1], (_Float16)v[i][2], (_Float16)v[i][3]};
    *(f16x4*)(wp + i * 32) = h;
  }
  asm volatile("" ::: "memory");
  const char* rp = lw + (lane >> 5) * 528 + (lane & 31) * 16;
#pragma unroll
  for (int s = 0; s < 8; ++s) f[s] = *(const u32x4*)(rp + s * 1056);
  asm volatile("" ::: "memory");
}
__device__ __forceinline__ void w_load(f32x4 (&v)[16], rsrc_t rw, int lane, int q, int h) {
  const int voff = (lane >> 5) * (8 * NS * 4) + (lane & 31) * 16, soff = h * (32 * NS * 4) + q * 512;
#pragma unroll
  for (int i = 0; i < 16; ++i)
    v[i] = __builtin_bit_cast(f32x4, __builtin_amdgcn_raw_buffer_load_b128(
        rw, voff + (i & 1) * (NS * 4), soff + ((i >> 3) * 16 + (i & 6)) * (NS * 4), 0));
}
__device__ __forceinline__ void w_frags(u32x4 (&f)[8], const f32x4 (&v)[16]) {
#pragma unroll
  for (int s = 0; s < 2; ++s)
#pragma unroll
    for (int e = 0; e < 4; ++e) {
      f32x4 a = {v[s * 8][e], v[s * 8 + 1][e], v[s * 8 + 2][e], v[s * 8 + 3][e]};
      f32x4 c = {v[s * 8 + 4][e], v[s * 8 + 5][e], v[s * 8 + 6][e], v[s * 8 + 7][e]};
      f[s * 4 + e] = cvt8(a, c);
    }
}
__device__ __forceinline__ int w_lane_off(int lane) { return ((lane & 31) >> 3) * BLK + ((lane >> 5) * 32 + (lane & 7)) * 16; }

__device__ __forceinline__ unsigned pack_piece(const u32x4 (&f)[8], rsrc_t rl, int exoff, char* lw, int lane) {
  typedef unsigned short u16x2 __attribute__((ext_vector_type(2)));
  unsigned* list = (unsigned*)(lw + 16 * 528);
  list[lane] = 0;
  unsigned se = 0;
#pragma unroll
  for (int d = 0; d < 4; ++d) se += ((f[0][d] >> 10) & 0x001f001fu) + ((f[4][d] >> 10) & 0x001f001fu);
  se = (se & 0xffffu) + (se >> 16);
  se += __builtin_amdgcn_update_dpp(0u, se, 0x111, 0xf, 0xf, false);
  se += __builtin_amdgcn_update_dpp(0u, se, 0x112, 0xf, 0xf, false);
  se += __builtin_amdgcn_update_dpp(0u, se, 0x114, 0xf, 0xf, false);
  se += __builtin_amdgcn_update_dpp(0u, se, 0x118, 0xf, 0xf, false);
  se += __builtin_amdgcn_update_dpp(0u, se, 0x142, 0xa, 0xf, false);
  se += __builtin_amdgcn_update_dpp(0u, se, 0x143, 0xc, 0xf, false);
  int base = (int)(((unsigned)__builtin_amdgcn_readlane(se, 63) + 1843 + 1023) >> 10) - 7;
  base = base < 0 ? 0 : base > 24 ? 24 : base;
  const unsigned b2 = (unsigned)base * 0x04000400u;
  unsigned pk[28], esc[8];
#pragma unroll
  for (int g = 0; g < 4; ++g) {
    unsigned t[8], wd[8];
#pragma unroll
    for (int i = 0; i < 8; ++i) {
      wd[i] = f[2 * g + (i >> 2)][i & 3];
      t[i] = __builtin_bit_cast(unsigned, (u16x2)(__builtin_bit_cast(u16x2, wd[i]) - __builtin_bit_cast(u16x2, b2)));
    }
    const unsigned z = (t[7] & 0x1fff1fffu) | ((wd[7] >> 2) & 0x20002000u);
#pragma unroll
    for (int i = 0; i < 7; ++i) pk[7 * g + i] = (t[i] & 0x1fff1fffu) | (wd[i] & 0x80008000u) | ((z << (13 - 2 * i)) & 0x60006000u);
#pragma unroll
    for (int u = 0; u < 2; ++u) {
      unsigned a = 0;
#pragma unroll
      for (int d = 0; d < 4; ++d) a |= ((t[4 * u + d] ^ (wd[4 * u + d] & 0x80008000u)) >> (4 * d)) & (0xe000e000u >> (4 * d));
      esc[2 * g + u] = a;
    }
  }
#pragma unroll
  for (int k = 0; k < 7; ++k)
    __builtin_amdgcn_raw_buffer_store_b128(u32x4{pk[4 * k], pk[4 * k + 1], pk[4 * k + 2], pk[4 * k + 3]}, rl,
                                           exoff + k * 1024 + lane * 16, 0, AUX_SYS);
  int cnt = 0;
  asm volatile("" ::: "memory");
#pragma unroll
  for (int u = 0; u < 8; ++u) {
    unsigned a = esc[u];
    unsigned long long m = __ballot(a != 0);
    do {
      const unsigned sh = __builtin_ctz(a | 0x80000000u) & 28u;
      const unsigned q = (a >> (sh + 1)) & 7u;
      const unsigned e = ((u * 512 + lane * 8 + 6) << 4) + (sh & 16u) - ((sh & 12u) << 3) + q;
      unsigned at = __builtin_amdgcn_mbcnt_hi((unsigned)(m >> 32), __builtin_amdgcn_mbcnt_lo((unsigned)m, cnt));
      at = (a != 0 && at < 128u) ? at : 128u + lane;
      ((unsigned short*)list)[at] = (unsigned short)e;
      a &= ~(0xfu << sh);
      cnt += __builtin_popcountll(m);
      m = __ballot(a != 0);
    } while (m != 0);
  }
  asm volatile("" ::: "memory");
  __builtin_amdgcn_raw_buffer_store_b32(list[lane], rl, exoff + 7168 + lane * 4, 0, AUX_SYS);
  asm volatile("" ::: "memory");
  return cnt > 128 ? D_RAW : (unsigned)(cnt << 5 | base);
}
__device__ __forceinline__ void unpack_piece(u32x4 (&r)[8], unsigned desc, unsigned patch, char* img, int lane) {
  const unsigned b2 = (desc & 31u) * 0x04000400u;
  unsigned pk[28];
#pragma unroll
  for (int i = 0; i < 28; ++i) pk[i] = r[i >> 2][i & 3];
#pragma unroll
  for (int g = 0; g < 4; ++g) {
    unsigned o[8], z = 0;
#pragma unroll
    for (int i = 0; i < 7; ++i) {
      o[i] = (pk[7 * g + i] & 0x9fff9fffu) + b2;
      z |= (pk[7 * g + i] >> (13 - 2 * i)) & (0x00030003u << (2 * i));
    }
    o[7] = ((z & 0x1fff1fffu) | ((z << 2) & 0x80008000u)) + b2;
    *(u32x4*)(img + g * 2048 + lane * 16) = u32x4{o[0], o[1], o[2], o[3]};
    *(u32x4*)(img + g * 2048 + 1024 + lane * 16) = u32x4{o[4], o[5], o[6], o[7]};
  }
  asm volatile("" ::: "memory");
#pragma unroll
  for (int k = 0; k < 2; ++k) {
    const unsigned e = k ? patch >> 16 : patch & 0xffffu;
    const unsigned hi = e << 29;
    if ((e & 7u) != 0)
      __hip_atomic_fetch_add((unsigned*)(img + (e >> 5) * 4), (e & 16u) ? hi : (unsigned)((int)hi >> 16),
                             __ATOMIC_RELAXED, __HIP_MEMORY_SCOPE_WORKGROUP);
  }
  asm volatile("" ::: "memory");
#pragma unroll
  for (int i = 0; i < 8; ++i) r[i] = *(const u32x4*)(img + i * 1024 + lane * 16);
  asm volatile("" ::: "memory");
}

__device__ __forceinline__ void convert(const Params& p, char* ws, char* lds, int e, bool agg, int w, int lane, unsigned& call, int round) {
  const int grp = p.rank >> 2, jme = p.rank & 3;
  const bool isx = e < 512, home = e < N_HOME;
  f32x4 v[16];
  int b = 0, j = 0;
  const int q = (e >> 7) & 3, h = e & 127;
  const unsigned cnt0 = ld_agent(ws, CNT_OFF + blockIdx.x * 128 + w * 4);
  if (isx) {
    x_piece(e, b, j);
    x_load(v, mk(p.x, IN_BYTES), lane, b, j);
  } else {
    w_load(v, mk(p.w, IN_BYTES), lane, q, h);
  }
  if (call == 0) {
    call = __builtin_amdgcn_readfirstlane(cnt0) + 1;
    if (lane == 0) st_agent(ws, CNT_OFF + blockIdx.x * 128 + w * 4, call);
  }
  const int t = home ? e : isx ? e - N_HOME + T_AWAY : h;
  const int d = lane >> 2, nd = home ? 3 : !isx ? 1 : e < E_ALL ? 4 : 7;
  const bool tomate = home || d >= 4;
  char* const sfp = (char*)p.ws[tomate ? grp * 4 + ((jme + 1 + d) & 3) : (grp ^ 1) * 4 + (isx ? d : q)] +
                    SF_OFF + ((call & 1) * 8 + p.rank) * SF_SRC + sf_pos((tomate && !home ? t + 96 : t) + (lane & 3));
  const int base = CTRL_BYTES + (call & 1) * BUF_BYTES;
  const rsrc_t rl = mk(ws, WS_USED);
  char* lw = lds + w * LDS_WAVE;
  unsigned sc = 0;
  u32x4 f[8];
  for (;;) {
    if (isx) {
      x_frags(f, v, lw, lane);
      const int o = base + b * BLK + j * 8192 + lane * 16;
#pragma unroll
      for (int i = 0; i < 8; ++i) __builtin_amdgcn_raw_buffer_store_b128(f[i], rl, o + i * 1024, 0, AUX_SYS);
    } else {
      w_frags(f, v);
      const int o = base + B_OFF + q * 4 * BLK + w_lane_off(lane) + h * 2048;
#pragma unroll
      for (int i = 0; i < 8; ++i) __builtin_amdgcn_raw_buffer_store_b128(f[i], rl, o + (i >> 2) * 1024 + (i & 3) * 128, 0, AUX_SYS);
    }
    __builtin_amdgcn_sched_barrier(0);
    if (sc != 0 || (sc = scale_piece(v)) == 0) break;
    DRAIN();
  }
  unsigned desc = D_RAW;
  if (t >= NPUSH + NR0) desc = pack_piece(f, rl, base + EX_OFF + e * EX_PIECE, lw, lane);
  DRAIN();
  if (!agg) {
    if (d < nd && (lane & 3) == 0) st_sys(sfp, 0, call << 18 | sc << 14 | desc);
  } else if (t >= NPUSH) {
    unsigned* dl = (unsigned*)(lds + LDS_RED) + round * 4;
    if (lane == 0) dl[w] = call << 18 | sc << 14 | desc;
    __syncthreads();
    if (w == 0 && d < nd) st_sys(sfp, 0, dl[lane & 3]);
  }
  if (lane < (isx ? 2 : 1)) st_sys(ws, (call & 1) * FLG_PAR + (isx ? a_flag(b, 2 * j + lane) : b_flag(q, h)), call << 4 | sc);
}

__device__ __forceinline__ unsigned push_first(const Params& p, char* ws, char* lds, int link, int t, int w, int lane) {
  const int peer = p.rank ^ (link + 1), jme = p.rank & 3, jp = peer & 3;
  const bool isx = link < 3;
  const unsigned cnt0 = ld_agent(ws, CNT_OFF + blockIdx.x * 128 + w * 4);
  f32x4 v[16];
  int b = 0, j = 0;
  if (isx) { x_piece(t, b, j); x_load(v, mk(p.x, IN_BYTES), lane, b, j); }
  else w_load(v, mk(p.w, IN_BYTES), lane, jp, t);
  const unsigned call = __builtin_amdgcn_readfirstlane(cnt0) + 1;
  const int base = CTRL_BYTES + (call & 1) * BUF_BYTES;
  const rsrc_t rp = mk(p.ws[peer], WS_USED);
  const int slot = mate_slot((jme - jp - 1) & 3, b);
  const int o0 = isx ? base + slot * BLK + j * 8192 + lane * 16
                     : base + B_OFF + (4 + jme) * 4 * BLK + w_lane_off(lane) + t * 2048;
  unsigned sc = 0;
  for (;;) {
    u32x4 f[8];
    if (isx) x_frags(f, v, lds, lane); else w_frags(f, v);
#pragma unroll
    for (int i = 0; i < 8; ++i) {
      if (i > PWIN) asm volatile("s_waitcnt vmcnt(%0)" :: "i"(PWIN) : "memory");
      __builtin_amdgcn_raw_buffer_store_b128(f[i], rp, o0 + (isx ? i * 1024 : (i >> 2) * 1024 + (i & 3) * 128), 0, AUX_SYS);
    }
    __builtin_amdgcn_sched_barrier(0);
    if (sc != 0 || (sc = scale_piece(v)) == 0) break;
    DRAIN();
  }
  DRAIN();
  if (lane < (isx ? 2 : 1))
    st_sys(p.ws[peer], (call & 1) * FLG_PAR + (isx ? a_flag(slot, 2 * j + lane) : b_flag(4 + jme, t)), call << 4 | sc);
  if (lane == 0) st_agent(ws, CNT_OFF + blockIdx.x * 128 + w * 4, call);
  return call;
}

struct Piece {
  int src, ex, dst;
  int kind;
  int fa, fb;
};
__device__ __forceinline__ Piece piece_of(int rank, int peer, int t, unsigned desc, int base) {
  Piece c;
  const int jme = rank & 3, jp = peer & 3;
  const bool intra = (peer >> 2) == (rank >> 2);
  if (intra || (t >= T_AWAY && t < T_OUT)) {
    int b, j;
    c.ex = intra ? (t < N_HOME ? t : t + (E_ALL - N_HOME)) : t - T_AWAY + N_HOME;
    x_piece(c.ex, b, j);
    const int slot = intra ? mate_slot((jp - jme - 1) & 3, b) : away_slot(jp, b);
    c.src = base + b * BLK + j * 8192; c.dst = base + slot * BLK + j * 8192; c.kind = 0;
    c.fa = a_flag(slot, 2 * j); c.fb = a_flag(slot, 2 * j + 1);
  } else if (t < T_AWAY) {
    c.src = base + B_OFF + jme * 4 * BLK + t * 2048; c.ex = 512 + jme * 128 + t;
    c.dst = base + B_OFF + (4 + jp) * 4 * BLK + t * 2048; c.kind = 1;
    c.fa = c.fb = b_flag(4 + jp, t);
  } else {
    const int o = min((int)(desc & 127u), N_OUT - 1);
    c.src = base + OX_OFF + jme * (1 << 20) + o * 8192; c.ex = 0;
    c.kind = o < 100 ? 2 : 3;
    c.dst = (slot_row(peer, out_slot(jme, o >> 1)) * NS + jp * 128 + (o & 1) * (o < 100 ? 16 * NS : 64)) * 4;
    c.fa = c.fb = 0;
  }
  c.ex = base + EX_OFF + c.ex * EX_PIECE;
  return c;
}

constexpr int NSLOT = 8, SLOT_B = 8192 + 256, F_CTL = NSLOT * SLOT_B;
enum { C_QN = 0, C_FIN = 1, C_ISSUED = 2, C_LANDED = 3, C_DONE = 4, C_ENT = 16 };
constexpr int OOB = 0x7ff00000;
__device__ __forceinline__ unsigned lds_ld(const unsigned* a) {
  return __builtin_amdgcn_readfirstlane(__hip_atomic_load(a, __ATOMIC_ACQUIRE, __HIP_MEMORY_SCOPE_WORKGROUP));
}
__device__ __forceinline__ void lds_st(unsigned* a, unsigned v) {
  asm volatile("" ::: "memory");
  __hip_atomic_store(a, v, __ATOMIC_RELEASE, __HIP_MEMORY_SCOPE_WORKGROUP);
  asm volatile("" ::: "memory");
}

__device__ __forceinline__ void fetch_schedule(const Params& p, char* ws, unsigned* ctl, unsigned call, int link, int lane) {
  const int rank = __builtin_amdgcn_readfirstlane(p.rank);
  const int peer = rank ^ (link + 1);
  const int np = link < 3 ? NP_INTRA : NP_CROSS;
  const rsrc_t rl = mk(ws, WS_USED);
  const int base = CTRL_BYTES + (call & 1) * BUF_BYTES;
  const int sf = SF_OFF + ((call & 1) * 8 + peer) * SF_SRC;
  const int nsg = link < 3 ? NSI : NSINGLE, v2 = (np - NPUSH - nsg) >> 1, nv = v2 + nsg;
  const unsigned tkbase = (call - 1) * (unsigned)nv - (unsigned)NFW;
  int tko = TK_OFF + link * 128;
  asm volatile("" : "+v"(tko));
  unsigned tk = 0;
  int v = blockIdx.x & 7, n = 0;
  bool fail = false;
  while ((unsigned)v < (unsigned)nv && !fail) {
    const int k = v < v2 ? 2 : 1, tf = NPUSH + v + min(v, v2);
    const int sfo = sf + sf_pos(tf + (lane & (k - 1)));
    while ((int)lds_ld(ctl + C_ISSUED) + AHEAD <= n || (int)lds_ld(ctl + C_DONE + (n & 1)) + NSLOT <= n ||
           (int)lds_ld(ctl + C_DONE + (~n & 1)) + NSLOT <= n + 1) __builtin_amdgcn_s_sleep(1);
    if (lane == 0) tk = __builtin_amdgcn_raw_ptr_buffer_atomic_add_i32(1, rl, tko, 0, 0);
    unsigned dv = ld_sys(ws, sfo);
    for (int i = 0; i < k; ++i) {
      const int t = tf + i;
      unsigned dc = __builtin_amdgcn_readlane(dv, i);
      if (!desc_ok(dc, call)) {
        const unsigned long long ts = __builtin_amdgcn_s_memrealtime();
        for (unsigned it = 1;; ++it) {
          dv = ld_sys(ws, sfo);
          dc = __builtin_amdgcn_readlane(dv, i);
          if (desc_ok(dc, call)) break;
          if ((it & 15) == 0 && (__builtin_amdgcn_readfirstlane(ld_agent(ws, TMO_OFF)) == call ||
                                 __builtin_amdgcn_s_memrealtime() - ts > DEADLINE)) { fail = true; break; }
          __builtin_amdgcn_s_sleep(1);
        }
        if (fail) break;
      }
      if (link < 3 && t >= N_HOME - NRI && t < N_HOME) dc |= D_RAW;
      const Piece c = piece_of(rank, peer, t, dc, base);
      const bool wide = link >= 3 && t >= NP_CROSS - NWC;
      unsigned* e = ctl + C_ENT + (n & (NSLOT - 1)) * 8;
      *(u32x4*)e = u32x4{(unsigned)((dc & D_RAW) ? c.src : c.ex), dc, (unsigned)(c.kind | (wide ? 4 : 0)), (unsigned)c.dst};
      *(u32x4*)(e + 4) = u32x4{(unsigned)c.fa, (unsigned)c.fb, (unsigned)t, 0u};
      lds_st(ctl + C_QN, ++n);
    }
    v = (int)(__builtin_amdgcn_readfirstlane(tk) - tkbase);
  }
  lds_st(ctl + C_FIN, n + 1);
  if (fail) {
    st_agent(ws, TMO_OFF, call);
    if (lane == 0) p.out[rank * MS * NS] = __builtin_nanf("");
  }
}

#define DMA(W, op, voff, ldsaddr) do { unsigned keep_; \
  asm volatile("s_mov_b32 %0, m0\n\ts_mov_b32 m0, %2\n\ts_waitcnt vmcnt(%4)\n\t" op " %1, %3, 0 offen sc0 sc1 lds\n\t" \
               "s_mov_b32 m0, %0" : "=&s"(keep_) : "v"(voff), "s"(ldsaddr), "s"(rp), "i"((W) - 1) : "memory"); } while (0)
__device__ __forceinline__ void fetch_load(const Params& p, char* lds, unsigned* ctl, int link, int lane) {
  const rsrc_t rp = mk(p.ws[p.rank ^ (link + 1)], WS_USED);
  const unsigned ring = (unsigned)(unsigned long long)(__attribute__((address_space(3))) char*)lds;
  const int lx = lane * 16, lw = w_lane_off(lane);
  unsigned q = 0;
  u32x4 e = {0, 0, 0, 0};
  int pub = 0;
  for (int n = 0;; ++n) {
    if ((int)__builtin_amdgcn_readfirstlane(q) <= n) {
      if ((int)lds_ld(ctl + C_QN) <= n) {
        if (pub < n - 1) {
          asm volatile("s_waitcnt vmcnt(8)" ::: "memory");
          lds_st(ctl + C_LANDED, n - 1);
        }
        DRAIN();
        pub = n;
        lds_st(ctl + C_LANDED, n);
        bool end = false;
        while ((int)lds_ld(ctl + C_QN) <= n) {
          if ((int)lds_ld(ctl + C_FIN) == n + 1) { end = true; break; }
          __builtin_amdgcn_s_sleep(1);
        }
        if (end) break;
      }
      asm volatile("" ::: "memory");
      e = *(const u32x4*)(ctl + C_ENT + (n & (NSLOT - 1)) * 8);
    }
    lds_st(ctl + C_ISSUED, n + 1);
    const unsigned dc = __builtin_amdgcn_readfirstlane(e[1]);
    asm volatile("" :: "v"(e));
    const unsigned kind = __builtin_amdgcn_readfirstlane(e[2]);
    const bool raw = dc & D_RAW, wraw = raw && (kind & 3) == 1;
    const int src = __builtin_amdgcn_readfirstlane(e[0]);
    const int a0 = (wraw ? lw : lx) + src, st = wraw ? 128 : 1024, hb = wraw ? 1024 : 4096;
    const unsigned slot = ring + (n & (NSLOT - 1)) * SLOT_B;
#define PIECE(W) _Pragma("unroll") for (int j = 0; j < 8; ++j) { \
      if (j < 7 || raw) DMA(W, "buffer_load_dwordx4", a0 + (j >> 2) * hb + (j & 3) * st, slot + j * 1024); \
      else DMA(W, "buffer_load_dword", (lane == 0 || 2 * lane < (int)((dc >> 5) & 255u)) ? src + 7168 + lane * 4 : OOB, slot + 8192); \
      if (j == ((W) - 1) % 8) {                      \
        const int l = n - ((W) - 1) / 8; \
        if (l > pub) { pub = l; lds_st(ctl + C_LANDED, l); } \
      } \
      if (j == 4) {                                  \
        q = __hip_atomic_load(ctl + C_QN, __ATOMIC_RELAXED, __HIP_MEMORY_SCOPE_WORKGROUP); \
        asm volatile("" ::: "memory"); \
        e = *(const u32x4*)(ctl + C_ENT + ((n + 1) & (NSLOT - 1)) * 8); \
      } }
    if (kind & 4) { PIECE(WWIN) } else { PIECE(WIN) }
#undef PIECE
  }
}

__device__ __forceinline__ void fetch_work(const Params& p, char* ws, char* lds, unsigned* ctl, unsigned call, int link, int k, int lane) {
  const rsrc_t rl = mk(ws, WS_USED), ro = mk(p.out, OUT_BYTES);
  const int lx = lane * 16, lw = w_lane_off(lane), lo = (lane >> 5) * (NS * 4) + (lane & 31) * 16,
            lo3 = (lane >> 4) * (NS * 4) + (lane & 15) * 16;
  for (int n = k;; n += 2) {
    bool end = false;
    while ((int)lds_ld(ctl + C_LANDED) <= n) {
      const int fin = (int)lds_ld(ctl + C_FIN);
      if (fin != 0 && fin - 1 <= n) { end = true; break; }
      __builtin_amdgcn_s_sleep(1);
    }
    if (end) break;
    asm volatile("" ::: "memory");
    char* img = lds + (n & (NSLOT - 1)) * SLOT_B;
    const u32x4 e0 = *(const u32x4*)(ctl + C_ENT + (n & (NSLOT - 1)) * 8), e1 = *(const u32x4*)(ctl + C_ENT + (n & (NSLOT - 1)) * 8 + 4);
    u32x4 r[8];
#pragma unroll
    for (int i = 0; i < 8; ++i) r[i] = *(const u32x4*)(img + i * 1024 + lane * 16);
    const unsigned pt = *(const unsigned*)(img + 8192 + lane * 4);
    asm volatile("" ::: "memory");
    const unsigned dc = __builtin_amdgcn_readfirstlane(e0[1]);
    const int kind = __builtin_amdgcn_readfirstlane(e0[2]) & 3;
    if (!(dc & D_RAW)) unpack_piece(r, dc, 2 * lane < (int)((dc >> 5) & 255u) ? pt : 0u, img, lane);
    const int d0 = (kind == 0 ? lx : kind == 1 ? lw : kind == 2 ? lo : lo3) + (int)e0[3];
    const int dst_st = kind == 0 ? 1024 : kind == 1 ? 128 : kind == 2 ? 2 * NS * 4 : 4 * NS * 4, dst_hb = kind == 1 ? 1024 : 4 * dst_st;
    const rsrc_t rd = kind >= 2 ? ro : rl;
#pragma unroll
    for (int i = 0; i < 8; ++i)
      __builtin_amdgcn_raw_buffer_store_b128(r[i], rd, d0 + (i >> 2) * dst_hb + (i & 3) * dst_st, 0, AUX_SYS);
    lds_st(ctl + C_DONE + k, n + 1);
    DRAIN();
    if (lane < 2 && kind < 2) st_sys(ws, (call & 1) * FLG_PAR + (lane ? e1[1] : e1[0]), call << 4 | ((dc >> 14) & 15u));
  }
}

struct Ready { int na, nb; unsigned f; };
template <int SLEEP, bool ISA, int SLEEP0 = 16>
__device__ __forceinline__ void wait_ready(void* ws, int off, unsigned call, int need, Ready& r) {
  unsigned long long t0 = __builtin_amdgcn_s_memrealtime();
  for (unsigned it = 1;; ++it) {
    r.f = ld_sys(ws, off);
    const unsigned long long m = __ballot(r.f >= call << 4);
    r.na = __builtin_ctz(~((unsigned)(m & (m >> 16)) & 0xffffu));
    r.nb = __builtin_ctz(~((unsigned)((m >> 32) & (m >> 48)) & 0xffffu));
    if ((ISA ? r.na : r.nb) > need) break;
    if ((it & 15) == 0 && (ld_agent(ws, TMO_OFF) == call || __builtin_amdgcn_s_memrealtime() - t0 > DEADLINE)) {
      st_agent(ws, TMO_OFF, call);
      r.na = r.nb = 17;
      break;
    }
    if (need == 0) __builtin_amdgcn_s_sleep(SLEEP0); else if (need < 12) __builtin_amdgcn_s_sleep(SLEEP); else __builtin_amdgcn_s_sleep(SL2);
  }
}
__device__ __forceinline__ int scale_of(unsigned f, int lane) { return (int)((unsigned)__builtin_amdgcn_readlane(f, lane) << 28) >> 24; }

#define MMA(B, A, C) __builtin_amdgcn_mfma_f32_32x32x16_f16(__builtin_bit_cast(f16x8, B), __builtin_bit_cast(f16x8, A), C, 0, 0, 0)
template <int NB>
__device__ __forceinline__ void unscale_partials(rsrc_t rs, unsigned f, float* dst, int mb, int a0, int a1, int b0, int lane) {
  constexpr int LD = NB * 32 + 4;
  const unsigned lead = __shfl(f, lane < 32 ? lane & 16 : 32);
  const bool uniform = __ballot(((f ^ lead) & 15u) != 0) == 0;
#pragma clang loop unroll(disable)
  for (int mn = 0; mn < mb * NB; ++mn) {
    {
      const int m = mn / NB, n = mn % NB;
      float* d = dst + m * 32 * LD + n * 32;
      if (uniform) {
        const int sh = -scale_of(f, 16 * m) - scale_of(f, 32);
        for (int i = 0; i < 16; ++i) d[i] = __builtin_amdgcn_ldexpf(d[i], sh);
      } else {
        f32x16 acc = {0, 0, 0, 0, 0, 0, 0, 0, 0, 0, 0, 0, 0, 0, 0, 0};
#pragma clang loop unroll(disable)
        for (int kc = 0; kc < 16; ++kc) {
          u32x4 a[4], b[4];
#pragma unroll
          for (int s = 0; s < 4; ++s) {
            a[s] = __builtin_amdgcn_raw_buffer_load_b128(rs, lane * 16 + s * 1024 + (m ? a1 : a0) + kc * 16384, 0, AUX_SYS);
            b[s] = __builtin_amdgcn_raw_buffer_load_b128(rs, lane * 16 + s * 1024 + b0 + n * BLK + kc * 16384, 0, AUX_SYS);
          }
#pragma unroll
          for (int hh = 0; hh < 2; ++hh) {
            const f32x16 z = {0, 0, 0, 0, 0, 0, 0, 0, 0, 0, 0, 0, 0, 0, 0, 0};
            const f32x16 t = MMA(b[2 * hh + 1], a[2 * hh + 1], MMA(b[2 * hh], a[2 * hh], z));
            const int sh = -scale_of(f, 16 * m + kc) - scale_of(f, 32 + 16 * hh + kc);
#pragma unroll
            for (int r = 0; r < 16; ++r) acc[r] += __builtin_amdgcn_ldexpf(t[r], sh);
          }
        }
#pragma unroll
        for (int r = 0; r < 16; ++r) d[(r & 3) * 4 + (r >> 2)] = acc[r];
      }
    }
  }
}

template <int MB, int NB, int D, int PRE>
__device__ __forceinline__ void gemm_tile(const Params& p, char* ws, unsigned call, float* lds, int slot0, int slot1, int qslot,
                                          int nt, int ob) {
  const int lane = threadIdx.x & 63;
  const int w = __builtin_amdgcn_readfirstlane(threadIdx.x >> 6);
  const rsrc_t rs = mk(ws, WS_USED);
  const int base = CTRL_BYTES + (call & 1) * BUF_BYTES;
  const int abase[2] = {base + slot0 * BLK + w * 4096, base + slot1 * BLK + w * 4096};
  const int bbase = base + B_OFF + (qslot * 4 + nt * NB) * BLK + w * 4096;
  const int foff = (call & 1) * FLG_PAR + (lane < 32 ? a_flag((lane & 16) ? slot1 : slot0, w) + (lane & 15) * 4
                             : FB_OFF + qslot * 512 + w * 128 + (lane & 31) * 4);
  const int voff = lane * 16;

  f32x16 acc[MB][NB];
#pragma unroll
  for (int m = 0; m < MB; ++m)
#pragma unroll
    for (int n = 0; n < NB; ++n)
#pragma unroll
      for (int r = 0; r < 16; ++r) acc[m][n][r] = 0.f;
  u32x4 fa[D][4][MB], fb[D][4][NB];

#define ISSUE_B(FB, kc, FROM_LDS) do { \
    if (FROM_LDS) { \
      _Pragma("unroll") for (int s = 0; s < 4; ++s) \
        _Pragma("unroll") for (int n = 0; n < NB; ++n) \
          FB[s][n] = *(const u32x4*)(pre + (((kc) - (16 - PRE)) * 4 * NB + s * NB + n) * 1024); \
    } else { \
      if (!PRE && rd.nb <= (kc)) wait_ready<(MB == 2 ? 8 : SL1), false>(ws, foff, call, (kc), rd); \
      asm volatile("" ::: "memory"); \
      _Pragma("unroll") for (int s = 0; s < 4; ++s) \
        _Pragma("unroll") for (int n = 0; n < NB; ++n) \
          FB[s][n] = __builtin_amdgcn_raw_buffer_load_b128(rs, voff + s * 1024, bbase + n * BLK + (kc) * 16384, AUX_SYS); \
    } \
    __builtin_amdgcn_sched_barrier(0); } while (0)
#define ISSUE_A(FA, kc) do { \
    if (rd.na <= (kc)) wait_ready<(MB == 2 ? 8 : SL1), true, (PRE ? 4 : 16)>(ws, foff, call, (kc), rd); \
    asm volatile("" ::: "memory"); \
    _Pragma("unroll") for (int s = 0; s < 4; ++s) \
      _Pragma("unroll") for (int m = 0; m < MB; ++m) \
        FA[s][m] = __builtin_amdgcn_raw_buffer_load_b128(rs, voff + s * 1024, abase[m] + (kc) * 16384, AUX_SYS); \
    __builtin_amdgcn_sched_barrier(0); } while (0)
#define ISSUE(FA, FB, kc, FROM_LDS) do { ISSUE_B(FB, kc, FROM_LDS); ISSUE_A(FA, kc); } while (0)
#define COMPUTE(FA, FB) do { \
    _Pragma("unroll") for (int s = 0; s < 4; ++s) \
      _Pragma("unroll") for (int m = 0; m < MB; ++m) \
        _Pragma("unroll") for (int n = 0; n < NB; ++n) acc[m][n] = MMA(FB[s][n], FA[s][m], acc[m][n]); \
    __builtin_amdgcn_sched_barrier(0); } while (0)

  Ready rd = {0, 0, 0u};
  constexpr int PSTR = PRE ? LDS_RED / 16 : MB * 32 * (NB * 32 + 4);
  static_assert(PRE % D == 0 && PRE * 4 * NB * 1024 <= LDS_RED / 4, "LDS region of a wave");
  char* const pre = (char*)(lds + w * PSTR) + lane * 16;
  if (PRE) {
    wait_ready<8, false>(ws, foff, call, 15, rd);
    asm volatile("" ::: "memory");
#pragma clang loop unroll(disable)
    for (int kc = 16 - PRE; kc < 16; ++kc) {
#pragma unroll
      for (int i = 0; i < 4 * NB; ++i)
        *(u32x4*)(pre + ((kc - (16 - PRE)) * 4 * NB + i) * 1024) = __builtin_amdgcn_raw_buffer_load_b128(
            rs, voff + (i / NB) * 1024, bbase + (i % NB) * BLK + kc * 16384, AUX_SYS);
    }
  }
#pragma unroll
  for (int d = 0; d < D; ++d) { ISSUE_B(fb[d], d, false); if (!PRE) ISSUE_A(fa[d], d); }
  if (PRE) {
#pragma unroll
    for (int d = 0; d < D; ++d) ISSUE_A(fa[d], d);
  }
#pragma clang loop unroll(disable)
  for (int kc = D; kc < 16 - PRE; kc += D) {
#pragma unroll
    for (int d = 0; d < D; ++d) {
      COMPUTE(fa[d], fb[d]);
      ISSUE(fa[d], fb[d], kc + d, false);
    }
  }
  if (PRE) {
#pragma clang loop unroll(disable)
    for (int kc = 16 - PRE; kc < 16; kc += D) {
#pragma unroll
      for (int d = 0; d < D; ++d) {
        COMPUTE(fa[d], fb[d]);
        ISSUE(fa[d], fb[d], kc + d, true);
      }
    }
  }
  const unsigned tmo = ld_agent(ws, TMO_OFF);
#pragma unroll
  for (int d = 0; d < D; ++d) COMPUTE(fa[d], fb[d]);
#undef ISSUE
#undef ISSUE_A
#undef ISSUE_B
#undef COMPUTE

  constexpr int LD = NB * 32 + 4;
  const bool remote = qslot >= 4;
  unsigned oq = 0;
  if (remote && threadIdx.x == 0) {
    int oo = OQ_OFF + (qslot - 4) * 128;
    asm volatile("" : "+v"(oo));
    oq = __builtin_amdgcn_raw_ptr_buffer_atomic_add_i32(MB * NB / 2, rs, oo, 0, 0);
  }
  {
    const bool bad = rd.na > 16 || tmo == call;
    float* dst = lds + w * PSTR + (lane & 31) * LD + (lane >> 5) * 16;
#pragma unroll
    for (int m = 0; m < MB; ++m)
#pragma unroll
      for (int n = 0; n < NB; ++n)
#pragma unroll
        for (int t = 0; t < 4; ++t) {
          f32x4 o = {acc[m][n][t], acc[m][n][4 + t], acc[m][n][8 + t], acc[m][n][12 + t]};
          *(f32x4*)(dst + m * 32 * LD + n * 32 + t * 4) = o;
        }
    if (__ballot((rd.f & 15u) != 0) != 0) unscale_partials<NB>(rs, rd.f, dst, MB, abase[0], abase[1], bbase, lane);
    if (bad) {
      const float q = __builtin_nanf("");
      for (int m = 0; m < MB; ++m)
        for (int i = 0; i < NB * 4; ++i) *(f32x4*)(dst + m * 32 * LD + (i >> 2) * 32 + (i & 3) * 4) = f32x4{q, q, q, q};
    }
  }
  __syncthreads();
  const rsrc_t ro = remote ? rs : mk(p.out, OUT_BYTES);
  constexpr int LPR = NB * 8;
  constexpr int RPI = 64 / LPR;
  constexpr int RPW = MB * 8;
  const int grow = slot_row(p.rank, (w * RPW) >> 5 ? slot1 : slot0) + ((w * RPW) & 31);
  const int o0 = remote ? base + OX_OFF + (qslot - 4) * (1 << 20) + (2 * ob + nt) * 8192 + w * (RPW * NB * 128) + lane * 16
                        : ((grow + lane / LPR) * NS + qslot * 128 + nt * NB * 32) * 4 + (lane % LPR) * 16;
  const int ost = remote ? 1024 : RPI * NS * 4;
#pragma unroll
  for (int i = 0; i < RPW / RPI; ++i) {
    const int row = w * RPW + i * RPI + lane / LPR, c4 = (lane % LPR) * 4;
    f32x4 s = *(const f32x4*)(lds + row * LD + c4);
#pragma unroll
    for (int k = 1; k < 4; ++k) s += *(const f32x4*)(lds + k * PSTR + row * LD + c4);
    f32x4 o;
#pragma unroll
    for (int e = 0; e < 4; ++e) o[e] = silu(s[e]);
    __builtin_amdgcn_raw_buffer_store_b128(__builtin_bit_cast(u32x4, o), ro, o0 + i * ost, 0, AUX_SYS);
  }
  DRAIN();
  __syncthreads();
  if (remote && w == 0) {
    const unsigned at = __builtin_amdgcn_readfirstlane(oq) - (call - 1) * (unsigned)N_OUT + lane;
    if (lane < MB * NB / 2 && at < (unsigned)N_OUT)
      st_sys(p.ws[((p.rank >> 2) ^ 1) * 4 + (qslot - 4)], SF_OFF + ((call & 1) * 8 + p.rank) * SF_SRC + sf_pos(T_OUT + at),
             call << 18 | D_RAW | (2 * ob + nt + lane));
  }
}

__global__ void __launch_bounds__(256, 1) ag_gemm(Params p) {
  extern __shared__ __attribute__((aligned(16))) float lds[];
  const int lane = threadIdx.x & 63;
  const int w = __builtin_amdgcn_readfirstlane(threadIdx.x >> 6);
  const int xcd = blockIdx.x & 7, idx = blockIdx.x >> 3;
  char* const ws = uni(p.ws[p.rank]);

  if (idx < 7) {
    const int link = (idx + xcd) % 7;
    unsigned* ctl = (unsigned*)((char*)lds + F_CTL);
    unsigned call = 0;
    if (w == 0) {
      if (lane < 16) ctl[lane] = 0;
      asm volatile("s_waitcnt lgkmcnt(0)\n\ts_barrier" ::: "memory");
      call = __builtin_amdgcn_readfirstlane(ld_agent(ws, CNT_OFF + blockIdx.x * 128)) + 1;
      if (lane == 0) st_agent(ws, CNT_OFF + blockIdx.x * 128, call);
      fetch_schedule(p, ws, ctl, call, link, lane);
    } else if (w == 1) {
      asm volatile("s_barrier" ::: "memory");
      fetch_load(p, (char*)lds, ctl, link, lane);
    } else {
      asm volatile("s_barrier" ::: "memory");
      call = __builtin_amdgcn_readfirstlane(ld_agent(ws, CNT_OFF + blockIdx.x * 128 + w * 4)) + 1;
      if (lane == 0) st_agent(ws, CNT_OFF + blockIdx.x * 128 + w * 4, call);
      fetch_work(p, ws, (char*)lds, ctl, call, link, w - 2, lane);
    }
  } else {
    unsigned call = 0;
    {
      const int g = (idx - 7) * 8 + xcd;
      constexpr int A0 = NPUSH / 4, A1 = (NPUSH + NE) / 4, NG = 32 - NE / 4, NTASK = 5 * NG + 96;
      auto task_piece = [](int o) {
        if (o < 5 * NG) {
          int a = o / 5; const int r = o - a * 5;
          a = a < 8 ? a + A1 : a < 8 + A0 ? a - 8 : a + NE / 4;
          return r == 0 ? a * 4 : 384 + r * 128 + a * 4;
        }
        const int r = o - 5 * NG;
        return r < 80 ? ((r & 1) ? N_HOME + (r >> 1) * 4 : (32 + (r >> 1)) * 4) : (r - 8) * 4;
      };
      const int ei = g < 80 ? (w == 0 ? g : w == 2 ? g + 80 : -1) : (w == 0 && g + 80 < 5 * NE ? g + 80 : -1);
      const int pk = g - (200 - 7 * NPUSH);
      if (w == 1 && pk >= 0) call = push_first(p, ws, (char*)lds + LDS_WAVE, pk % 7, pk / 7, w, lane);
      if (ei < 0) for (int i = call ? 64 : -P0; i < g; ++i) __builtin_amdgcn_s_sleep(1);
      else for (int i = EP0; i < ei; i += EPS) __builtin_amdgcn_s_sleep(1);
      const int r1 = g >= NTASK ? 1 : g >= 400 - NTASK ? 3 : 2;
#pragma clang loop unroll(disable)
      for (int r = ei >= 0 ? 0 : 1; r < r1; ++r) {
        const int s5 = ei % 5;
        const int e = r == 0 ? (s5 == 0 ? 0 : 384 + s5 * 128) + NPUSH + ei / 5 : task_piece(r == 1 ? g : g + NTASK - 200) + w;
        convert(p, ws, (char*)lds, e, r != 0, w, lane, call, r);
      }
      if (call == 0) {
        call = __builtin_amdgcn_readfirstlane(ld_agent(ws, CNT_OFF + blockIdx.x * 128 + w * 4)) + 1;
        if (lane == 0) st_agent(ws, CNT_OFF + blockIdx.x * 128 + w * 4, call);
      }
    }
    {
      const int gi = idx - 7;
      const bool own = xcd < 4;
      __syncthreads();
#pragma clang loop unroll(disable)
      for (int ob = 2 * gi; ob < (own ? 58 : 50); ob += 50) {
        gemm_tile<2, 4, 2, 0>(p, ws, call, lds, own ? own_slot(ob) : out_slot(xcd - 4, ob), own ? own_slot(ob + 1) : out_slot(xcd - 4, ob + 1),
                        xcd, 0, ob);
      }
      const int k2 = own ? gi - 9 : gi;
      if (own && gi >= 4 && gi < 9) {
        const int s0 = own_slot(54 + gi);
        gemm_tile<1, 4, 2, 0>(p, ws, call, lds, s0, s0, xcd, 0, 0);
      } else if (k2 >= 0 && k2 < 12) {
        const int s0 = out_slot(0, 50 + (k2 >> 1));
        gemm_tile<1, 2, D2, PR2>(p, ws, call, lds, s0, s0, xcd, k2 & 1, 50 + (k2 >> 1));
      } else if (!own && gi < 24) {
        const int s0 = mate_slot((gi - 12) >> 2, 15);
        gemm_tile<1, 1, D1, PR1>(p, ws, call, lds, s0, s0, xcd & 3, gi & 3, 0);
      }
    }
  }
}

}

extern "C" void kernel_launch(int rank, int nranks, ncclComm_t comm,
                              hipStream_t stream,
                              void* const* d_in, const int* in_sizes, int n_in,
                              void* d_out, int out_size,
                              const int* in_dtypes, int out_dtype,
                              void* const* d_out_all,
                              void* const* d_ws_all, size_t ws_bytes) {
  using namespace agk;
  if (nranks != NR || (unsigned)rank >= (unsigned)NR || n_in < 2 || in_sizes[0] != MS * KD || in_sizes[1] != KD * NS ||
      out_size != 4096 * NS || ws_bytes < (size_t)WS_USED) {
    fprintf(stderr, "kernel_launch: unexpected problem shape\n");
    return;
  }
  HIP_CHECK(hipFuncSetAttribute((const void*)ag_gemm, hipFuncAttributeMaxDynamicSharedMemorySize, LDS_BYTES));
  Params p;
  p.x = (const float*)d_in[0];
  p.w = (const float*)d_in[1];
  p.out = (float*)d_out;
  for (int i = 0; i < NR; ++i) p.ws[i] = d_ws_all[i];
  p.rank = rank;
  p.pad = 0;
  hipLaunchKernelGGL(ag_gemm, dim3(256), dim3(256), LDS_BYTES, stream, p);
  HIP_CHECK(hipGetLastError());
}
```

```cpp
#include <hip/hip_runtime.h>
#include <rccl/rccl.h>
#include <cstdio>

#define HIP_CHECK(expr) do { hipError_t _e = (expr); if (_e != hipSuccess) { \
    fprintf(stderr, "HIP error %s at %s:%d\n", hipGetErrorString(_e), __FILE__, __LINE__); \
    return; } } while (0)

namespace agk {

typedef unsigned u32x4 __attribute__((ext_vector_type(4)));
typedef float f32x4 __attribute__((ext_vector_type(4)));
typedef float f32x16 __attribute__((ext_vector_type(16)));
typedef _Float16 f16x8 __attribute__((ext_vector_type(8)));
typedef __attribute__((address_space(1))) unsigned gu32;
typedef __amdgpu_buffer_rsrc_t rsrc_t;

constexpr int NR = 8, MS = 512, NS = 512, KD = 4096;
constexpr int BLK = 32 * KD * 2;
constexpr int A_SLOTS = 72;
constexpr int A_BYTES = A_SLOTS * BLK, B_BYTES = 32 * BLK, OX_BYTES = 4 << 20;
constexpr int EX_PIECE = 7168 + 256, EX_BYTES = 8 << 20;
constexpr int B_OFF = A_BYTES, OX_OFF = B_OFF + B_BYTES, EX_OFF = OX_OFF + OX_BYTES, BUF_BYTES = EX_OFF + EX_BYTES;
static_assert(1024 * EX_PIECE <= EX_BYTES, "export buffer");
constexpr int CNT_OFF = 0;
constexpr int TMO_OFF = 32768;
constexpr int TK_OFF = 33024;
constexpr int OQ_OFF = 34816;
constexpr int FA_OFF = 65536, FLG_PAR = 65536;
constexpr int FB_OFF = FA_OFF + A_SLOTS * 512;
constexpr int SF_OFF = 262144, SF_SRC = 32768;
constexpr int CTRL_BYTES = 1 << 20;
static_assert(FB_OFF + 4096 <= FA_OFF + FLG_PAR && FA_OFF + 2 * FLG_PAR <= SF_OFF && SF_OFF + 16 * SF_SRC <= CTRL_BYTES, "control block");
constexpr int WS_USED = CTRL_BYTES + 2 * BUF_BYTES;
constexpr int OUT_BYTES = 4096 * NS * 4;
constexpr int IN_BYTES = MS * KD * 4;
constexpr int AUX_SYS = 17;
constexpr int RSRC_W3 = 0x00020000;
constexpr unsigned long long DEADLINE = 30000000ull;
constexpr int LDS_RED = 4 * 128 * 68 * 4, LDS_BYTES = LDS_RED + 64;
constexpr int LDS_WAVE = 16 * 528 + 512;

constexpr int NPUSH = 16;
constexpr int NR0 = 24;
constexpr int PWIN = 2;
constexpr int WIN = 6;
constexpr int NSINGLE = 12;
constexpr int AHEAD = 2;
constexpr int NE = 56;
constexpr int P0 = 40;
constexpr int EP0 = 60;
constexpr int EPS = 8;
constexpr int NRI = 8;
constexpr int NSI = 24;
constexpr int D2 = 4;
constexpr int D1 = 4;
constexpr int PR2 = 4;
constexpr int PR1 = 8;
constexpr int SL1 = 2;
constexpr int SL2 = 1;
constexpr int NWC = 12;
constexpr int WWIN = 16;
constexpr int NFW = 8;
constexpr int N_HOME = 352, E_ALL = 480, NP_INTRA = 384, T_AWAY = 128, T_OUT = 288, N_OUT = 112, NP_CROSS = T_OUT + N_OUT;
constexpr unsigned D_RAW = 1u << 13;
__device__ __forceinline__ bool desc_ok(unsigned d, unsigned call) { return (d >> 18) == (call & 0x3fffu); }

struct Params {
  const float* x;
  const float* w;
  float* out;
  void* me;
  void* ws[NR];
  int rank;
  int pad;
};


__device__ __forceinline__ char* uni(const void* p) {
  const unsigned long long a = (unsigned long long)p;
  const unsigned lo = __builtin_amdgcn_readfirstlane((unsigned)a), hi = __builtin_amdgcn_readfirstlane((unsigned)(a >> 32));
  return (char*)(((unsigned long long)hi << 32) | lo);
}
__device__ __forceinline__ char* lane_ws(const Params& p, int r) {
  char* a = uni(p.ws[0]);
#pragma unroll
  for (int i = 1; i < NR; ++i) { char* b = uni(p.ws[i]); a = r == i ? b : a; }
  return a;
}
__device__ __forceinline__ rsrc_t mk(const void* p, int bytes) {
  return __builtin_amdgcn_make_buffer_rsrc((void*)uni(p), 0, bytes, RSRC_W3);
}
__device__ __forceinline__ unsigned ld_sys(const void* base, int off) {
  return __hip_atomic_load((gu32*)((char*)base + off), __ATOMIC_RELAXED, __HIP_MEMORY_SCOPE_SYSTEM);
}
__device__ __forceinline__ void st_sys(void* base, int off, unsigned v) {
  __hip_atomic_store((gu32*)((char*)base + off), v, __ATOMIC_RELAXED, __HIP_MEMORY_SCOPE_SYSTEM);
}
__device__ __forceinline__ unsigned ld_agent(const void* base, int off) {
  return __hip_atomic_load((gu32*)((char*)base + off), __ATOMIC_RELAXED, __HIP_MEMORY_SCOPE_AGENT);
}
__device__ __forceinline__ void st_agent(void* base, int off, unsigned v) {
  __hip_atomic_store((gu32*)((char*)base + off), v, __ATOMIC_RELAXED, __HIP_MEMORY_SCOPE_AGENT);
}
#define DRAIN() asm volatile("s_waitcnt vmcnt(0)" ::: "memory")

__device__ __forceinline__ u32x4 cvt8(f32x4 a, f32x4 b) {
  f16x8 h;
  h[0] = (_Float16)a[0]; h[1] = (_Float16)a[1]; h[2] = (_Float16)a[2]; h[3] = (_Float16)a[3];
  h[4] = (_Float16)b[0]; h[5] = (_Float16)b[1]; h[6] = (_Float16)b[2]; h[7] = (_Float16)b[3];
  return __builtin_bit_cast(u32x4, h);
}

__device__ __forceinline__ float silu(float y) {
  const float t = fminf(fmaxf(y, -60.f), 60.f);
  return y * __builtin_amdgcn_rcpf(1.f + __expf(-t));
}

__device__ __forceinline__ int slot_row(int rank, int slot) {
  const int grp = rank >> 2, j = rank & 3;
  int src, b;
  if (slot < 16) { src = rank; b = slot; }
  else if (slot < 52) { const int r = (slot - 16) / 12; src = grp * 4 + ((j + 1 + r) & 3); b = slot - 16 - r * 12; b = b == 11 ? 15 : b; }
  else { const int i = (slot - 52) / 5; src = (grp ^ 1) * 4 + i; b = 11 + slot - 52 - i * 5; }
  return src * MS + b * 32;
}
__device__ __forceinline__ int mate_slot(int r, int b) { return 16 + r * 12 + (b == 15 ? 11 : b); }
__device__ __forceinline__ int away_slot(int i, int b) { return 52 + i * 5 + (b - 11); }
__device__ __forceinline__ int out_slot(int ow, int ob) {
  if (ob < 11) return ob;
  if (ob < 26) { const int r = (ob - 11) / 5; return mate_slot(r, ob - 11 - r * 5); }
  if (ob < 38) return away_slot((ow + 1 + ((ob - 26) >> 2)) & 3, 11 + ((ob - 26) & 3));
  if (ob < 50) return mate_slot((ob - 38) >> 2, 5 + ((ob - 38) & 3));
  return mate_slot((ob - 50) >> 1, 9 + (ob & 1));
}
__device__ __forceinline__ int own_slot(int i) {
  if (i < 16) return i;
  if (i < 31) return out_slot(0, i - 5);
  if (i < 47) return away_slot((i - 31) >> 2, 11 + ((i - 31) & 3));
  if (i < 59) return out_slot(0, i - 9);
  if (i < 63) return away_slot(i - 59, 15);
  if (i < 69) return out_slot(0, i - 13);
  return mate_slot(i - 69, 15);
}
__device__ __forceinline__ int sf_pos(int t) {
  return t < NPUSH + NE ? 16384 + (t >> 1) * 128 + (t & 1) * 4 : (t >> 2) * 128 + (t & 3) * 4;
}
__device__ __forceinline__ int a_flag(int slot, int u) { return FA_OFF + slot * 512 + (u & 3) * 128 + (u >> 2) * 4; }
__device__ __forceinline__ int b_flag(int qslot, int h) { return FB_OFF + qslot * 512 + ((h >> 1) & 3) * 128 + (h & 1) * 64 + (h >> 3) * 4; }

__device__ __forceinline__ void x_piece(int e, int& b, int& j) {
  if (e < 160) { j = e / 5; b = e - j * 5; }
  else if (e < 288) { b = 5 + (e & 3); j = (e - 160) >> 2; }
  else if (e < N_HOME) { b = 9 + (e & 1); j = (e - 288) >> 1; }
  else if (e < E_ALL) { b = 11 + (e & 3); j = (e - N_HOME) >> 2; }
  else { b = 15; j = e - E_ALL; }
}
__device__ __forceinline__ void piece_load(f32x4 (&v)[16], const Params& p, bool isx, int lane, int b, int j, int q, int h) {
  static_assert(KD * 4 == 8 * NS * 4, "same lane offsets");
  const rsrc_t r = mk(isx ? p.x : p.w, IN_BYTES);
  const int voff = (lane >> 5) * (KD * 4) + (lane & 31) * 16;
  const int soff = isx ? b * (32 * KD * 4) + j * 512 : h * (32 * NS * 4) + q * 512;
  const int s1 = isx ? 2 * KD * 4 : NS * 4, s8 = isx ? 16 * KD * 4 : 16 * NS * 4;
#pragma unroll
  for (int i = 0; i < 16; ++i)
    v[i] = __builtin_bit_cast(f32x4, __builtin_amdgcn_raw_buffer_load_b128(r, voff, soff + (i >> 3) * s8 + (i & 7) * s1, 0));
  __builtin_amdgcn_sched_barrier(0);
}
__device__ __forceinline__ unsigned scale_piece(f32x4 (&v)[16]) {
  float m = 0.f;
#pragma unroll
  for (int i = 0; i < 16; ++i)
    m = fmaxf(fmaxf(m, fmaxf(fabsf(v[i][0]), fabsf(v[i][1]))), fmaxf(fabsf(v[i][2]), fabsf(v[i][3])));
  unsigned u = __builtin_bit_cast(unsigned, m);
  u = max(u, (unsigned)__builtin_amdgcn_update_dpp(0u, u, 0x111, 0xf, 0xf, false));
  u = max(u, (unsigned)__builtin_amdgcn_update_dpp(0u, u, 0x112, 0xf, 0xf, false));
  u = max(u, (unsigned)__builtin_amdgcn_update_dpp(0u, u, 0x114, 0xf, 0xf, false));
  u = max(u, (unsigned)__builtin_amdgcn_update_dpp(0u, u, 0x118, 0xf, 0xf, false));
  u = max(u, (unsigned)__builtin_amdgcn_update_dpp(0u, u, 0x142, 0xa, 0xf, false));
  u = max(u, (unsigned)__builtin_amdgcn_update_dpp(0u, u, 0x143, 0xc, 0xf, false));
  const unsigned top = __builtin_amdgcn_readlane(u, 63);
  const int e = top >> 23;
  if (top == 0 || e == 255 || (e >= 117 && e <= 141)) return 0;
  const int z = min((141 - e) >> 4, 7);
#pragma unroll
  for (int i = 0; i < 16; ++i)
#pragma unroll
    for (int k = 0; k < 4; ++k) v[i][k] = __builtin_amdgcn_ldexpf(v[i][k], 16 * z);
  return z & 15;
}
__device__ __forceinline__ void x_frags(u32x4 (&f)[8], const f32x4 (&v)[16], char* lw, int lane) {
  typedef _Float16 f16x4 __attribute__((ext_vector_type(4)));
  char* wp = lw + ((lane & 31) >> 1) * 528 + (lane >> 5) * 16 + (lane & 1) * 8;
#pragma unroll
  for (int i = 0; i < 16; ++i) {
    f16x4 h = {(_Float16)v[i][0], (_Float16)v[i][1], (_Float16)v[i][2], (_Float16)v[i][3]};
    *(f16x4*)(wp + i * 32) = h;
  }
  asm volatile("" ::: "memory");
  const char* rp = lw + (lane >> 5) * 528 + (lane & 31) * 16;
#pragma unroll
  for (int s = 0; s < 8; ++s) f[s] = *(const u32x4*)(rp + s * 1056);
  asm volatile("" ::: "memory");
}
__device__ __forceinline__ void w_frags(u32x4 (&f)[8], const f32x4 (&v)[16]) {
#pragma unroll
  for (int s = 0; s < 2; ++s)
#pragma unroll
    for (int e = 0; e < 4; ++e) {
      f32x4 a = {v[s * 8][e], v[s * 8 + 1][e], v[s * 8 + 2][e], v[s * 8 + 3][e]};
      f32x4 c = {v[s * 8 + 4][e], v[s * 8 + 5][e], v[s * 8 + 6][e], v[s * 8 + 7][e]};
      f[s * 4 + e] = cvt8(a, c);
    }
}
__device__ __forceinline__ int w_lane_off(int lane) { return ((lane & 31) >> 3) * BLK + ((lane >> 5) * 32 + (lane & 7)) * 16; }

__device__ __forceinline__ unsigned pack_piece(const u32x4 (&f)[8], rsrc_t rl, int exoff, char* lw, int lane) {
  typedef unsigned short u16x2 __attribute__((ext_vector_type(2)));
  unsigned* list = (unsigned*)(lw + 16 * 528);
  list[lane] = 0;
  unsigned se = 0;
#pragma unroll
  for (int d = 0; d < 4; ++d) se += ((f[0][d] >> 10) & 0x001f001fu) + ((f[4][d] >> 10) & 0x001f001fu);
  se = (se & 0xffffu) + (se >> 16);
  se += __builtin_amdgcn_update_dpp(0u, se, 0x111, 0xf, 0xf, false);
  se += __builtin_amdgcn_update_dpp(0u, se, 0x112, 0xf, 0xf, false);
  se += __builtin_amdgcn_update_dpp(0u, se, 0x114, 0xf, 0xf, false);
  se += __builtin_amdgcn_update_dpp(0u, se, 0x118, 0xf, 0xf, false);
  se += __builtin_amdgcn_update_dpp(0u, se, 0x142, 0xa, 0xf, false);
  se += __builtin_amdgcn_update_dpp(0u, se, 0x143, 0xc, 0xf, false);
  int base = (int)(((unsigned)__builtin_amdgcn_readlane(se, 63) + 1843 + 1023) >> 10) - 7;
  base = base < 0 ? 0 : base > 24 ? 24 : base;
  const unsigned b2 = (unsigned)base * 0x04000400u;
  unsigned pk[28], esc[8];
#pragma unroll
  for (int g = 0; g < 4; ++g) {
    unsigned t[8], wd[8];
#pragma unroll
    for (int i = 0; i < 8; ++i) {
      wd[i] = f[2 * g + (i >> 2)][i & 3];
      t[i] = __builtin_bit_cast(unsigned, (u16x2)(__builtin_bit_cast(u16x2, wd[i]) - __builtin_bit_cast(u16x2, b2)));
    }
    const unsigned z = (t[7] & 0x1fff1fffu) | ((wd[7] >> 2) & 0x20002000u);
#pragma unroll
    for (int i = 0; i < 7; ++i) pk[7 * g + i] = (t[i] & 0x1fff1fffu) | (wd[i] & 0x80008000u) | ((z << (13 - 2 * i)) & 0x60006000u);
#pragma unroll
    for (int u = 0; u < 2; ++u) {
      unsigned a = 0;
#pragma unroll
      for (int d = 0; d < 4; ++d) a |= ((t[4 * u + d] ^ (wd[4 * u + d] & 0x80008000u)) >> (4 * d)) & (0xe000e000u >> (4 * d));
      esc[2 * g + u] = a;
    }
  }
#pragma unroll
  for (int k = 0; k < 7; ++k)
    __builtin_amdgcn_raw_buffer_store_b128(u32x4{pk[4 * k], pk[4 * k + 1], pk[4 * k + 2], pk[4 * k + 3]}, rl,
                                           exoff + k * 1024 + lane * 16, 0, AUX_SYS);
  int cnt = 0;
  asm volatile("" ::: "memory");
#pragma unroll
  for (int u = 0; u < 8; ++u) {
    unsigned a = esc[u];
    unsigned long long m = __ballot(a != 0);
    do {
      const unsigned sh = __builtin_ctz(a | 0x80000000u) & 28u;
      const unsigned q = (a >> (sh + 1)) & 7u;
      const unsigned e = ((u * 512 + lane * 8 + 6) << 4) + (sh & 16u) - ((sh & 12u) << 3) + q;
      unsigned at = __builtin_amdgcn_mbcnt_hi((unsigned)(m >> 32), __builtin_amdgcn_mbcnt_lo((unsigned)m, cnt));
      at = (a != 0 && at < 128u) ? at : 128u + lane;
      ((unsigned short*)list)[at] = (unsigned short)e;
      a &= ~(0xfu << sh);
      cnt += __builtin_popcountll(m);
      m = __ballot(a != 0);
    } while (m != 0);
  }
  asm volatile("" ::: "memory");
  __builtin_amdgcn_raw_buffer_store_b32(list[lane], rl, exoff + 7168 + lane * 4, 0, AUX_SYS);
  asm volatile("" ::: "memory");
  return cnt > 128 ? D_RAW : (unsigned)(cnt << 5 | base);
}
__device__ __forceinline__ void unpack_piece(u32x4 (&r)[8], unsigned desc, unsigned patch, char* img, int lane) {
  const unsigned b2 = (desc & 31u) * 0x04000400u;
  unsigned pk[28];
#pragma unroll
  for (int i = 0; i < 28; ++i) pk[i] = r[i >> 2][i & 3];
#pragma unroll
  for (int g = 0; g < 4; ++g) {
    unsigned o[8], z = 0;
#pragma unroll
    for (int i = 0; i < 7; ++i) {
      o[i] = (pk[7 * g + i] & 0x9fff9fffu) + b2;
      z |= (pk[7 * g + i] >> (13 - 2 * i)) & (0x00030003u << (2 * i));
    }
    o[7] = ((z & 0x1fff1fffu) | ((z << 2) & 0x80008000u)) + b2;
    *(u32x4*)(img + g * 2048 + lane * 16) = u32x4{o[0], o[1], o[2], o[3]};
    *(u32x4*)(img + g * 2048 + 1024 + lane * 16) = u32x4{o[4], o[5], o[6], o[7]};
  }
  asm volatile("" ::: "memory");
#pragma unroll
  for (int k = 0; k < 2; ++k) {
    const unsigned e = k ? patch >> 16 : patch & 0xffffu;
    const unsigned hi = e << 29;
    if ((e & 7u) != 0)
      __hip_atomic_fetch_add((unsigned*)(img + (e >> 5) * 4), (e & 16u) ? hi : (unsigned)((int)hi >> 16),
                             __ATOMIC_RELAXED, __HIP_MEMORY_SCOPE_WORKGROUP);
  }
  asm volatile("" ::: "memory");
#pragma unroll
  for (int i = 0; i < 8; ++i) r[i] = *(const u32x4*)(img + i * 1024 + lane * 16);
  asm volatile("" ::: "memory");
}

__device__ __forceinline__ void convert(const Params& p, char* ws, char* lds, int e, bool agg, int w, int lane, unsigned& call, int round) {
  const int grp = p.rank >> 2, jme = p.rank & 3;
  const bool isx = e < 512, home = e < N_HOME;
  f32x4 v[16];
  int b = 0, j = 0;
  const int q = (e >> 7) & 3, h = e & 127;
  const unsigned cnt0 = ld_agent(ws, CNT_OFF + blockIdx.x * 128 + w * 4);
  if (isx) x_piece(e, b, j);
  piece_load(v, p, isx, lane, b, j, q, h);
  call = __builtin_amdgcn_readfirstlane(cnt0) + 1;
  const int t = home ? e : isx ? e - N_HOME + T_AWAY : h;
  const int d = lane >> 2, nd = home ? 3 : !isx ? 1 : e < E_ALL ? 4 : 7;
  const bool tomate = home || d >= 4;
  char* const sfp = lane_ws(p, tomate ? grp * 4 + ((jme + 1 + d) & 3) : (grp ^ 1) * 4 + (isx ? d : q)) +
                    SF_OFF + ((call & 1) * 8 + p.rank) * SF_SRC + sf_pos((tomate && !home ? t + 96 : t) + (lane & 3));
  const int base = CTRL_BYTES + (call & 1) * BUF_BYTES;
  const rsrc_t rl = mk(ws, WS_USED);
  char* lw = lds + w * LDS_WAVE;
  unsigned sc = 0;
  u32x4 f[8];
  for (;;) {
    if (isx) {
      x_frags(f, v, lw, lane);
      const int o = base + b * BLK + j * 8192 + lane * 16;
#pragma unroll
      for (int i = 0; i < 8; ++i) __builtin_amdgcn_raw_buffer_store_b128(f[i], rl, o + i * 1024, 0, AUX_SYS);
    } else {
      w_frags(f, v);
      const int o = base + B_OFF + q * 4 * BLK + w_lane_off(lane) + h * 2048;
#pragma unroll
      for (int i = 0; i < 8; ++i) __builtin_amdgcn_raw_buffer_store_b128(f[i], rl, o + (i >> 2) * 1024 + (i & 3) * 128, 0, AUX_SYS);
    }
    __builtin_amdgcn_sched_barrier(0);
    if (sc != 0 || (sc = scale_piece(v)) == 0) break;
    DRAIN();
  }
  unsigned desc = D_RAW;
  if (t >= NPUSH + NR0) desc = pack_piece(f, rl, base + EX_OFF + e * EX_PIECE, lw, lane);
  DRAIN();
  if (!agg) {
    if (d < nd && (lane & 3) == 0) st_sys(sfp, 0, call << 18 | sc << 14 | desc);
  } else if (t >= NPUSH) {
    unsigned* dl = (unsigned*)(lds + LDS_RED) + round * 4;
    if (lane == 0) dl[w] = call << 18 | sc << 14 | desc;
    __syncthreads();
    if (w == 0 && d < nd) st_sys(sfp, 0, dl[lane & 3]);
  }
  if (lane < (isx ? 2 : 1)) st_sys(ws, (call & 1) * FLG_PAR + (isx ? a_flag(b, 2 * j + lane) : b_flag(q, h)), call << 4 | sc);
}

__device__ __forceinline__ unsigned push_first(const Params& p, char* ws, char* lds, int link, int t, int w, int lane) {
  const int peer = p.rank ^ (link + 1), jme = p.rank & 3, jp = peer & 3;
  const bool isx = link < 3;
  const unsigned cnt0 = ld_agent(ws, CNT_OFF + blockIdx.x * 128 + w * 4);
  f32x4 v[16];
  int b = 0, j = 0;
  if (isx) { j = t / 5; b = t - j * 5; }
  piece_load(v, p, isx, lane, b, j, jp, t);
  const unsigned call = __builtin_amdgcn_readfirstlane(cnt0) + 1;
  const int base = CTRL_BYTES + (call & 1) * BUF_BYTES;
  const rsrc_t rp = mk(p.ws[peer], WS_USED);
  const int slot = mate_slot((jme - jp - 1) & 3, b);
  const int o0 = isx ? base + slot * BLK + j * 8192 + lane * 16
                     : base + B_OFF + (4 + jme) * 4 * BLK + w_lane_off(lane) + t * 2048;
  unsigned sc = 0;
  for (;;) {
    u32x4 f[8];
    if (isx) x_frags(f, v, lds, lane); else w_frags(f, v);
#pragma unroll
    for (int i = 0; i < 8; ++i) {
      if (i > PWIN) asm volatile("s_waitcnt vmcnt(%0)" :: "i"(PWIN) : "memory");
      __builtin_amdgcn_raw_buffer_store_b128(f[i], rp, o0 + (isx ? i * 1024 : (i >> 2) * 1024 + (i & 3) * 128), 0, AUX_SYS);
    }
    __builtin_amdgcn_sched_barrier(0);
    if (sc != 0 || (sc = scale_piece(v)) == 0) break;
    DRAIN();
  }
  DRAIN();
  if (lane < (isx ? 2 : 1))
    st_sys(p.ws[peer], (call & 1) * FLG_PAR + (isx ? a_flag(slot, 2 * j + lane) : b_flag(4 + jme, t)), call << 4 | sc);
  return call;
}

struct Piece {
  int src, ex, dst;
  int kind;
  int fa, fb;
};
__device__ __forceinline__ Piece in_piece(int rank, int peer, int t, int base) {
  Piece c;
  const int jme = rank & 3, jp = peer & 3;
  const bool intra = (peer >> 2) == (rank >> 2);
  if (intra || t >= T_AWAY) {
    int b, j;
    c.ex = intra ? (t < N_HOME ? t : t + (E_ALL - N_HOME)) : t - T_AWAY + N_HOME;
    x_piece(c.ex, b, j);
    const int slot = intra ? mate_slot((jp - jme - 1) & 3, b) : away_slot(jp, b);
    c.src = base + b * BLK + j * 8192; c.dst = base + slot * BLK + j * 8192; c.kind = 0;
    c.fa = a_flag(slot, 2 * j); c.fb = a_flag(slot, 2 * j + 1);
  } else {
    c.src = base + B_OFF + jme * 4 * BLK + t * 2048; c.ex = 512 + jme * 128 + t;
    c.dst = base + B_OFF + (4 + jp) * 4 * BLK + t * 2048; c.kind = 1;
    c.fa = c.fb = b_flag(4 + jp, t);
  }
  c.ex = base + EX_OFF + c.ex * EX_PIECE;
  return c;
}
__device__ __forceinline__ Piece out_piece(int rank, int peer, unsigned desc, int base) {
  Piece c;
  const int jme = rank & 3, jp = peer & 3;
  const int o = min((int)(desc & 127u), N_OUT - 1);
  c.ex = c.src = base + OX_OFF + jme * (1 << 20) + o * 8192;
  c.kind = o < 100 ? 2 : 3;
  c.dst = (slot_row(peer, out_slot(jme, o >> 1)) * NS + jp * 128 + (o & 1) * (o < 100 ? 16 * NS : 64)) * 4;
  c.fa = c.fb = 0;
  return c;
}

constexpr int NSLOT = 8, SLOT_B = 8192 + 256, F_CTL = NSLOT * SLOT_B;
enum { C_QN = 0, C_FIN = 1, C_ISSUED = 2, C_LANDED = 3, C_DONE = 4, C_ENT = 16 };
constexpr int OOB = 0x7ff00000;
__device__ __forceinline__ unsigned lds_ld(const unsigned* a) {
  return __builtin_amdgcn_readfirstlane(__hip_atomic_load(a, __ATOMIC_ACQUIRE, __HIP_MEMORY_SCOPE_WORKGROUP));
}
__device__ __forceinline__ void lds_st(unsigned* a, unsigned v) {
  asm volatile("" ::: "memory");
  __hip_atomic_store(a, v, __ATOMIC_RELEASE, __HIP_MEMORY_SCOPE_WORKGROUP);
  asm volatile("" ::: "memory");
}

__device__ __forceinline__ void fetch_schedule(const Params& p, char* ws, unsigned* ctl, unsigned call, int link, int lane) {
  const int rank = __builtin_amdgcn_readfirstlane(p.rank);
  const int peer = rank ^ (link + 1);
  const int np = link < 3 ? NP_INTRA : NP_CROSS;
  const rsrc_t rl = mk(ws, WS_USED);
  const int base = CTRL_BYTES + (call & 1) * BUF_BYTES;
  const int sf = SF_OFF + ((call & 1) * 8 + peer) * SF_SRC;
  const int nsg = link < 3 ? NSI : NSINGLE, v2 = (np - NPUSH - nsg) >> 1, nv = v2 + nsg;
  const unsigned tkbase = (call - 1) * (unsigned)nv - (unsigned)NFW;
  int tko = TK_OFF + link * 128;
  asm volatile("" : "+v"(tko));
  unsigned tk = 0;
  int v = blockIdx.x & 7, n = 0;
  bool fail = false;
  while ((unsigned)v < (unsigned)nv && !fail) {
    const int k = v < v2 ? 2 : 1, tf = NPUSH + v + min(v, v2);
    const int sfo = sf + sf_pos(tf + (lane & (k - 1)));
    while ((int)lds_ld(ctl + C_ISSUED) + AHEAD <= n || (int)lds_ld(ctl + C_DONE + (n & 1)) + NSLOT <= n ||
           (int)lds_ld(ctl + C_DONE + (~n & 1)) + NSLOT <= n + 1) __builtin_amdgcn_s_sleep(1);
    if (lane == 0) tk = __builtin_amdgcn_raw_ptr_buffer_atomic_add_i32(1, rl, tko, 0, 0);
    unsigned dv = ld_sys(ws, sfo);
    for (int i = 0; i < k; ++i) {
      const int t = tf + i;
      const bool isout = link >= 3 && t >= T_OUT;
      Piece c = in_piece(rank, peer, isout ? 0 : t, base);
      asm volatile("" :: "s"(c.src), "s"(c.ex), "s"(c.dst), "s"(c.fa), "s"(c.fb));
      unsigned dc = __builtin_amdgcn_readlane(dv, i);
      if (!desc_ok(dc, call)) {
        const unsigned long long ts = __builtin_amdgcn_s_memrealtime();
        for (unsigned it = 1;; ++it) {
          dv = ld_sys(ws, sfo);
          dc = __builtin_amdgcn_readlane(dv, i);
          if (desc_ok(dc, call)) break;
          if ((it & 15) == 0 && (__builtin_amdgcn_readfirstlane(ld_agent(ws, TMO_OFF)) == call ||
                                 __builtin_amdgcn_s_memrealtime() - ts > DEADLINE)) { fail = true; break; }
          __builtin_amdgcn_s_sleep(1);
        }
        if (fail) break;
      }
      if (link < 3 && t >= N_HOME - NRI && t < N_HOME) dc |= D_RAW;
      if (isout) c = out_piece(rank, peer, dc, base);
      const bool wide = link >= 3 && t >= NP_CROSS - NWC;
      unsigned* e = ctl + C_ENT + (n & (NSLOT - 1)) * 8;
      *(u32x4*)e = u32x4{(unsigned)((dc & D_RAW) ? c.src : c.ex), dc, (unsigned)(c.kind | (wide ? 4 : 0)), (unsigned)c.dst};
      *(u32x4*)(e + 4) = u32x4{(unsigned)c.fa, (unsigned)c.fb, (unsigned)t, 0u};
      lds_st(ctl + C_QN, ++n);
    }
    v = (int)(__builtin_amdgcn_readfirstlane(tk) - tkbase);
  }
  lds_st(ctl + C_FIN, n + 1);
  if (fail) {
    st_agent(ws, TMO_OFF, call);
    if (lane == 0) p.out[rank * MS * NS] = __builtin_nanf("");
  }
}

#define DMA(W, op, voff, ldsaddr) do { unsigned keep_; \
  asm volatile("s_mov_b32 %0, m0\n\ts_mov_b32 m0, %2\n\ts_waitcnt vmcnt(%4)\n\t" op " %1, %3, 0 offen sc0 sc1 lds\n\t" \
               "s_mov_b32 m0, %0" : "=&s"(keep_) : "v"(voff), "s"(ldsaddr), "s"(rp), "i"((W) - 1) : "memory"); } while (0)
__device__ __forceinline__ void fetch_load(const Params& p, char* lds, unsigned* ctl, int link, int lane) {
  const rsrc_t rp = mk(p.ws[p.rank ^ (link + 1)], WS_USED);
  const unsigned ring = (unsigned)(unsigned long long)(__attribute__((address_space(3))) char*)lds;
  const int lx = lane * 16, lw = w_lane_off(lane);
  unsigned q = 0;
  u32x4 e = {0, 0, 0, 0};
  int pub = 0;
  for (int n = 0;; ++n) {
    if ((int)__builtin_amdgcn_readfirstlane(q) <= n) {
      if ((int)lds_ld(ctl + C_QN) <= n) {
        if (pub < n - 1) {
          asm volatile("s_waitcnt vmcnt(8)" ::: "memory");
          lds_st(ctl + C_LANDED, n - 1);
        }
        DRAIN();
        pub = n;
        lds_st(ctl + C_LANDED, n);
        bool end = false;
        while ((int)lds_ld(ctl + C_QN) <= n) {
          if ((int)lds_ld(ctl + C_FIN) == n + 1) { end = true; break; }
          __builtin_amdgcn_s_sleep(1);
        }
        if (end) break;
      }
      asm volatile("" ::: "memory");
      e = *(const u32x4*)(ctl + C_ENT + (n & (NSLOT - 1)) * 8);
    }
    lds_st(ctl + C_ISSUED, n + 1);
    const unsigned dc = __builtin_amdgcn_readfirstlane(e[1]);
    asm volatile("" :: "v"(e));
    const unsigned kind = __builtin_amdgcn_readfirstlane(e[2]);
    const bool raw = dc & D_RAW, wraw = raw && (kind & 3) == 1;
    const int src = __builtin_amdgcn_readfirstlane(e[0]);
    const int a0 = (wraw ? lw : lx) + src, st = wraw ? 128 : 1024, hb = wraw ? 1024 : 4096;
    const unsigned slot = ring + (n & (NSLOT - 1)) * SLOT_B;
#define PIECE(W) _Pragma("unroll") for (int j = 0; j < 8; ++j) { \
      if (j < 7 || raw) DMA(W, "buffer_load_dwordx4", a0 + (j >> 2) * hb + (j & 3) * st, slot + j * 1024); \
      else DMA(W, "buffer_load_dword", (lane == 0 || 2 * lane < (int)((dc >> 5) & 255u)) ? src + 7168 + lane * 4 : OOB, slot + 8192); \
      if (j == ((W) - 1) % 8) {                      \
        const int l = n - ((W) - 1) / 8; \
        if (l > pub) { pub = l; lds_st(ctl + C_LANDED, l); } \
      } \
      if (j == 4) {                                  \
        q = __hip_atomic_load(ctl + C_QN, __ATOMIC_RELAXED, __HIP_MEMORY_SCOPE_WORKGROUP); \
        asm volatile("" ::: "memory"); \
        e = *(const u32x4*)(ctl + C_ENT + ((n + 1) & (NSLOT - 1)) * 8); \
      } }
    if (kind & 4) { PIECE(WWIN) } else { PIECE(WIN) }
#undef PIECE
  }
}

__device__ __forceinline__ void fetch_work(const Params& p, char* ws, char* lds, unsigned* ctl, unsigned call, int link, int k, int lane) {
  const rsrc_t rl = mk(ws, WS_USED), ro = mk(p.out, OUT_BYTES);
  const int lx = lane * 16, lw = w_lane_off(lane), lo = (lane >> 5) * (NS * 4) + (lane & 31) * 16,
            lo3 = (lane >> 4) * (NS * 4) + (lane & 15) * 16;
  for (int n = k;; n += 2) {
    bool end = false;
    while ((int)lds_ld(ctl + C_LANDED) <= n) {
      const int fin = (int)lds_ld(ctl + C_FIN);
      if (fin != 0 && fin - 1 <= n) { end = true; break; }
      __builtin_amdgcn_s_sleep(1);
    }
    if (end) break;
    asm volatile("" ::: "memory");
    char* img = lds + (n & (NSLOT - 1)) * SLOT_B;
    const u32x4 e0 = *(const u32x4*)(ctl + C_ENT + (n & (NSLOT - 1)) * 8), e1 = *(const u32x4*)(ctl + C_ENT + (n & (NSLOT - 1)) * 8 + 4);
    u32x4 r[8];
#pragma unroll
    for (int i = 0; i < 8; ++i) r[i] = *(const u32x4*)(img + i * 1024 + lane * 16);
    const unsigned pt = *(const unsigned*)(img + 8192 + lane * 4);
    asm volatile("" ::: "memory");
    const unsigned dc = __builtin_amdgcn_readfirstlane(e0[1]);
    const int kind = __builtin_amdgcn_readfirstlane(e0[2]) & 3;
    if (!(dc & D_RAW)) unpack_piece(r, dc, 2 * lane < (int)((dc >> 5) & 255u) ? pt : 0u, img, lane);
    const int d0 = (kind == 0 ? lx : kind == 1 ? lw : kind == 2 ? lo : lo3) + (int)e0[3];
    const int dst_st = kind == 0 ? 1024 : kind == 1 ? 128 : kind == 2 ? 2 * NS * 4 : 4 * NS * 4, dst_hb = kind == 1 ? 1024 : 4 * dst_st;
    const rsrc_t rd = kind >= 2 ? ro : rl;
#pragma unroll
    for (int i = 0; i < 8; ++i)
      __builtin_amdgcn_raw_buffer_store_b128(r[i], rd, d0 + (i >> 2) * dst_hb + (i & 3) * dst_st, 0, AUX_SYS);
    lds_st(ctl + C_DONE + k, n + 1);
    DRAIN();
    if (lane < 2 && kind < 2) st_sys(ws, (call & 1) * FLG_PAR + (lane ? e1[1] : e1[0]), call << 4 | ((dc >> 14) & 15u));
  }
}

struct Ready { int na, nb; unsigned f; };
template <int SLEEP, bool ISA, int SLEEP0 = 16>
__device__ __forceinline__ void wait_ready(void* ws, int off, unsigned call, int need, Ready& r) {
  unsigned long long t0 = __builtin_amdgcn_s_memrealtime();
  for (unsigned it = 1;; ++it) {
    r.f = ld_sys(ws, off);
    const unsigned long long m = __ballot(r.f >= call << 4);
    r.na = __builtin_ctz(~((unsigned)(m & (m >> 16)) & 0xffffu));
    r.nb = __builtin_ctz(~((unsigned)((m >> 32) & (m >> 48)) & 0xffffu));
    if ((ISA ? r.na : r.nb) > need) break;
    if ((it & 15) == 0 && (ld_agent(ws, TMO_OFF) == call || __builtin_amdgcn_s_memrealtime() - t0 > DEADLINE)) {
      st_agent(ws, TMO_OFF, call);
      r.na = r.nb = 17;
      break;
    }
    if (need == 0) __builtin_amdgcn_s_sleep(SLEEP0); else if (need < 12) __builtin_amdgcn_s_sleep(SLEEP); else __builtin_amdgcn_s_sleep(SL2);
  }
}
__device__ __forceinline__ int scale_of(unsigned f, int lane) { return (int)((unsigned)__builtin_amdgcn_readlane(f, lane) << 28) >> 24; }

#define MMA(B, A, C) __builtin_amdgcn_mfma_f32_32x32x16_f16(__builtin_bit_cast(f16x8, B), __builtin_bit_cast(f16x8, A), C, 0, 0, 0)
template <int NB>
__device__ __forceinline__ void unscale_partials(rsrc_t rs, unsigned f, float* dst, int mb, int a0, int a1, int b0, int lane) {
  constexpr int LD = NB * 32 + 4;
  const unsigned lead = __shfl(f, lane < 32 ? lane & 16 : 32);
  const bool uniform = __ballot(((f ^ lead) & 15u) != 0) == 0;
#pragma clang loop unroll(disable)
  for (int mn = 0; mn < mb * NB; ++mn) {
    {
      const int m = mn / NB, n = mn % NB;
      float* d = dst + m * 32 * LD + n * 32;
      if (uniform) {
        const int sh = -scale_of(f, 16 * m) - scale_of(f, 32);
        for (int i = 0; i < 16; ++i) d[i] = __builtin_amdgcn_ldexpf(d[i], sh);
      } else {
        f32x16 acc = {0, 0, 0, 0, 0, 0, 0, 0, 0, 0, 0, 0, 0, 0, 0, 0};
#pragma clang loop unroll(disable)
        for (int kc = 0; kc < 16; ++kc) {
          u32x4 a[4], b[4];
#pragma unroll
          for (int s = 0; s < 4; ++s) {
            a[s] = __builtin_amdgcn_raw_buffer_load_b128(rs, lane * 16 + s * 1024 + (m ? a1 : a0) + kc * 16384, 0, AUX_SYS);
            b[s] = __builtin_amdgcn_raw_buffer_load_b128(rs, lane * 16 + s * 1024 + b0 + n * BLK + kc * 16384, 0, AUX_SYS);
          }
#pragma unroll
          for (int hh = 0; hh < 2; ++hh) {
            const f32x16 z = {0, 0, 0, 0, 0, 0, 0, 0, 0, 0, 0, 0, 0, 0, 0, 0};
            const f32x16 t = MMA(b[2 * hh + 1], a[2 * hh + 1], MMA(b[2 * hh], a[2 * hh], z));
            const int sh = -scale_of(f, 16 * m + kc) - scale_of(f, 32 + 16 * hh + kc);
#pragma unroll
            for (int r = 0; r < 16; ++r) acc[r] += __builtin_amdgcn_ldexpf(t[r], sh);
          }
        }
#pragma unroll
        for (int r = 0; r < 16; ++r) d[(r & 3) * 4 + (r >> 2)] = acc[r];
      }
    }
  }
}

template <int MB, int NB, int D, int PRE>
__device__ __forceinline__ void gemm_tile(const Params& p, char* ws, unsigned call, float* lds, int slot0, int slot1, int qslot,
                                          int nt, int ob) {
  const int lane = threadIdx.x & 63;
  const int w = __builtin_amdgcn_readfirstlane(threadIdx.x >> 6);
  const rsrc_t rs = mk(ws, WS_USED);
  const int base = CTRL_BYTES + (call & 1) * BUF_BYTES;
  const int abase[2] = {base + slot0 * BLK + w * 4096, base + slot1 * BLK + w * 4096};
  const int bbase = base + B_OFF + (qslot * 4 + nt * NB) * BLK + w * 4096;
  const int foff = (call & 1) * FLG_PAR + (lane < 32 ? a_flag((lane & 16) ? slot1 : slot0, w) + (lane & 15) * 4
                             : FB_OFF + qslot * 512 + w * 128 + (lane & 31) * 4);
  const int voff = lane * 16;

  f32x16 acc[MB][NB];
#pragma unroll
  for (int m = 0; m < MB; ++m)
#pragma unroll
    for (int n = 0; n < NB; ++n)
#pragma unroll
      for (int r = 0; r < 16; ++r) acc[m][n][r] = 0.f;
  u32x4 fa[D][4][MB], fb[D][4][NB];

#define ISSUE_B(FB, kc, FROM_LDS) do { \
    if (FROM_LDS) { \
      _Pragma("unroll") for (int s = 0; s < 4; ++s) \
        _Pragma("unroll") for (int n = 0; n < NB; ++n) \
          FB[s][n] = *(const u32x4*)(pre + (((kc) - (16 - PRE)) * 4 * NB + s * NB + n) * 1024); \
    } else { \
      if (!PRE && rd.nb <= (kc)) wait_ready<(MB == 2 ? 8 : SL1), false>(ws, foff, call, (kc), rd); \
      asm volatile("" ::: "memory"); \
      _Pragma("unroll") for (int s = 0; s < 4; ++s) \
        _Pragma("unroll") for (int n = 0; n < NB; ++n) \
          FB[s][n] = __builtin_amdgcn_raw_buffer_load_b128(rs, voff + s * 1024, bbase + n * BLK + (kc) * 16384, AUX_SYS); \
    } \
    __builtin_amdgcn_sched_barrier(0); } while (0)
#define ISSUE_A(FA, kc) do { \
    if (rd.na <= (kc)) wait_ready<(MB == 2 ? 8 : SL1), true, (PRE ? 4 : 16)>(ws, foff, call, (kc), rd); \
    asm volatile("" ::: "memory"); \
    _Pragma("unroll") for (int s = 0; s < 4; ++s) \
      _Pragma("unroll") for (int m = 0; m < MB; ++m) \
        FA[s][m] = __builtin_amdgcn_raw_buffer_load_b128(rs, voff + s * 1024, abase[m] + (kc) * 16384, AUX_SYS); \
    __builtin_amdgcn_sched_barrier(0); } while (0)
#define ISSUE(FA, FB, kc, FROM_LDS) do { ISSUE_B(FB, kc, FROM_LDS); ISSUE_A(FA, kc); } while (0)
#define COMPUTE(FA, FB) do { \
    _Pragma("unroll") for (int s = 0; s < 4; ++s) \
      _Pragma("unroll") for (int m = 0; m < MB; ++m) \
        _Pragma("unroll") for (int n = 0; n < NB; ++n) acc[m][n] = MMA(FB[s][n], FA[s][m], acc[m][n]); \
    __builtin_amdgcn_sched_barrier(0); } while (0)

  Ready rd = {0, 0, 0u};
  constexpr int PSTR = PRE ? LDS_RED / 16 : MB * 32 * (NB * 32 + 4);
  static_assert(PRE % D == 0 && PRE * 4 * NB * 1024 <= LDS_RED / 4, "LDS region of a wave");
  char* const pre = (char*)(lds + w * PSTR) + lane * 16;
  if (PRE) {
    wait_ready<8, false>(ws, foff, call, 15, rd);
    asm volatile("" ::: "memory");
#pragma clang loop unroll(disable)
    for (int kc = 16 - PRE; kc < 16; ++kc) {
#pragma unroll
      for (int i = 0; i < 4 * NB; ++i)
        *(u32x4*)(pre + ((kc - (16 - PRE)) * 4 * NB + i) * 1024) = __builtin_amdgcn_raw_buffer_load_b128(
            rs, voff + (i / NB) * 1024, bbase + (i % NB) * BLK + kc * 16384, AUX_SYS);
    }
  }
#pragma unroll
  for (int d = 0; d < D; ++d) { ISSUE_B(fb[d], d, false); if (!PRE) ISSUE_A(fa[d], d); }
  if (PRE) {
#pragma unroll
    for (int d = 0; d < D; ++d) ISSUE_A(fa[d], d);
  }
#pragma clang loop unroll(disable)
  for (int kc = D; kc < 16 - PRE; kc += D) {
#pragma unroll
    for (int d = 0; d < D; ++d) {
      COMPUTE(fa[d], fb[d]);
      ISSUE(fa[d], fb[d], kc + d, false);
    }
  }
  if (PRE) {
#pragma clang loop unroll(disable)
    for (int kc = 16 - PRE; kc < 16; kc += D) {
#pragma unroll
      for (int d = 0; d < D; ++d) {
        COMPUTE(fa[d], fb[d]);
        ISSUE(fa[d], fb[d], kc + d, true);
      }
    }
  }
  const unsigned tmo = ld_agent(ws, TMO_OFF);
#pragma unroll
  for (int d = 0; d < D; ++d) COMPUTE(fa[d], fb[d]);
#undef ISSUE
#undef ISSUE_A
#undef ISSUE_B
#undef COMPUTE

  constexpr int LD = NB * 32 + 4;
  const bool remote = qslot >= 4;
  unsigned oq = 0;
  if (remote && threadIdx.x == 0) {
    int oo = OQ_OFF + (qslot - 4) * 128;
    asm volatile("" : "+v"(oo));
    oq = __builtin_amdgcn_raw_ptr_buffer_atomic_add_i32(MB * NB / 2, rs, oo, 0, 0);
  }
  {
    const bool bad = rd.na > 16 || tmo == call;
    float* dst = lds + w * PSTR + (lane & 31) * LD + (lane >> 5) * 16;
#pragma unroll
    for (int m = 0; m < MB; ++m)
#pragma unroll
      for (int n = 0; n < NB; ++n)
#pragma unroll
        for (int t = 0; t < 4; ++t) {
          f32x4 o = {acc[m][n][t], acc[m][n][4 + t], acc[m][n][8 + t], acc[m][n][12 + t]};
          *(f32x4*)(dst + m * 32 * LD + n * 32 + t * 4) = o;
        }
    if (__ballot((rd.f & 15u) != 0) != 0) unscale_partials<NB>(rs, rd.f, dst, MB, abase[0], abase[1], bbase, lane);
    if (bad) {
      const float q = __builtin_nanf("");
      for (int m = 0; m < MB; ++m)
        for (int i = 0; i < NB * 4; ++i) *(f32x4*)(dst + m * 32 * LD + (i >> 2) * 32 + (i & 3) * 4) = f32x4{q, q, q, q};
    }
  }
  __syncthreads();
  const rsrc_t ro = remote ? rs : mk(p.out, OUT_BYTES);
  constexpr int LPR = NB * 8;
  constexpr int RPI = 64 / LPR;
  constexpr int RPW = MB * 8;
  const int grow = slot_row(p.rank, (w * RPW) >> 5 ? slot1 : slot0) + ((w * RPW) & 31);
  const int o0 = remote ? base + OX_OFF + (qslot - 4) * (1 << 20) + (2 * ob + nt) * 8192 + w * (RPW * NB * 128) + lane * 16
                        : ((grow + lane / LPR) * NS + qslot * 128 + nt * NB * 32) * 4 + (lane % LPR) * 16;
  const int ost = remote ? 1024 : RPI * NS * 4;
#pragma unroll
  for (int i = 0; i < RPW / RPI; ++i) {
    const int row = w * RPW + i * RPI + lane / LPR, c4 = (lane % LPR) * 4;
    f32x4 s = *(const f32x4*)(lds + row * LD + c4);
#pragma unroll
    for (int k = 1; k < 4; ++k) s += *(const f32x4*)(lds + k * PSTR + row * LD + c4);
    f32x4 o;
#pragma unroll
    for (int e = 0; e < 4; ++e) o[e] = silu(s[e]);
    __builtin_amdgcn_raw_buffer_store_b128(__builtin_bit_cast(u32x4, o), ro, o0 + i * ost, 0, AUX_SYS);
  }
  DRAIN();
  __syncthreads();
  if (remote && w == 0) {
    const unsigned at = __builtin_amdgcn_readfirstlane(oq) - (call - 1) * (unsigned)N_OUT + lane;
    if (lane < MB * NB / 2 && at < (unsigned)N_OUT)
      st_sys(p.ws[((p.rank >> 2) ^ 1) * 4 + (qslot - 4)], SF_OFF + ((call & 1) * 8 + p.rank) * SF_SRC + sf_pos(T_OUT + at),
             call << 18 | D_RAW | (2 * ob + nt + lane));
  }
}

__global__ void __launch_bounds__(256, 1) ag_gemm(Params p) {
  extern __shared__ __attribute__((aligned(16))) float lds[];
  const int lane = threadIdx.x & 63;
  const int w = __builtin_amdgcn_readfirstlane(threadIdx.x >> 6);
  const int xcd = blockIdx.x & 7, idx = blockIdx.x >> 3;
  asm volatile("" :: "s"(p.x), "s"(p.w), "s"(p.me), "s"(p.rank));
  char* const ws = uni(p.me);

  if (idx < 7) {
    const int link = (idx + xcd) % 7;
    unsigned* ctl = (unsigned*)((char*)lds + F_CTL);
    unsigned call = 0;
    if (w == 0) {
      if (lane < 16) ctl[lane] = 0;
      asm volatile("s_waitcnt lgkmcnt(0)\n\ts_barrier" ::: "memory");
      call = __builtin_amdgcn_readfirstlane(ld_agent(ws, CNT_OFF + blockIdx.x * 128)) + 1;
      fetch_schedule(p, ws, ctl, call, link, lane);
      if (lane == 0) st_agent(ws, CNT_OFF + blockIdx.x * 128, call);
    } else if (w == 1) {
      asm volatile("s_barrier" ::: "memory");
      fetch_load(p, (char*)lds, ctl, link, lane);
    } else {
      asm volatile("s_barrier" ::: "memory");
      call = __builtin_amdgcn_readfirstlane(ld_agent(ws, CNT_OFF + blockIdx.x * 128 + w * 4)) + 1;
      if (lane == 0) st_agent(ws, CNT_OFF + blockIdx.x * 128 + w * 4, call);
      fetch_work(p, ws, (char*)lds, ctl, call, link, w - 2, lane);
    }
  } else {
    unsigned call = 0;
    {
      const int g = (idx - 7) * 8 + xcd;
      constexpr int A0 = NPUSH / 4, A1 = (NPUSH + NE) / 4, NG = 32 - NE / 4, NTASK = 5 * NG + 96;
      auto task_piece = [](int o) {
        if (o < 5 * NG) {
          int a = o / 5; const int r = o - a * 5;
          a = a < 8 ? a + A1 : a < 8 + A0 ? a - 8 : a + NE / 4;
          return r == 0 ? a * 4 : 384 + r * 128 + a * 4;
        }
        const int r = o - 5 * NG;
        return r < 80 ? ((r & 1) ? N_HOME + (r >> 1) * 4 : (32 + (r >> 1)) * 4) : (r - 8) * 4;
      };
      const int ei = g < 80 ? (w == 0 ? g : w == 2 ? g + 80 : -1) : (w == 0 && g + 80 < 5 * NE ? g + 80 : -1);
      const int pk = g - (200 - 7 * NPUSH);
      if (w == 1 && pk >= 0) call = push_first(p, ws, (char*)lds + LDS_WAVE, pk % 7, pk / 7, w, lane);
      if (ei < 0) for (int i = call ? 64 : -P0; i < g; ++i) __builtin_amdgcn_s_sleep(1);
      else for (int i = EP0; i < ei; i += EPS) __builtin_amdgcn_s_sleep(1);
      const int r1 = g >= NTASK ? 1 : g >= 400 - NTASK ? 3 : 2;
#pragma clang loop unroll(disable)
      for (int r = ei >= 0 ? 0 : 1; r < r1; ++r) {
        const int s5 = ei % 5;
        const int e = r == 0 ? (s5 == 0 ? 0 : 384 + s5 * 128) + NPUSH + ei / 5 : task_piece(r == 1 ? g : g + NTASK - 200) + w;
        convert(p, ws, (char*)lds, e, r != 0, w, lane, call, r);
      }
      if (call == 0) call = __builtin_amdgcn_readfirstlane(ld_agent(ws, CNT_OFF + blockIdx.x * 128 + w * 4)) + 1;
      if (lane == 0) st_agent(ws, CNT_OFF + blockIdx.x * 128 + w * 4, call);
    }
    {
      const int gi = idx - 7;
      const bool own = xcd < 4;
      __syncthreads();
#pragma clang loop unroll(disable)
      for (int ob = 2 * gi; ob < (own ? 58 : 50); ob += 50) {
        gemm_tile<2, 4, 2, 0>(p, ws, call, lds, own ? own_slot(ob) : out_slot(xcd - 4, ob), own ? own_slot(ob + 1) : out_slot(xcd - 4, ob + 1),
                        xcd, 0, ob);
      }
      const int k2 = own ? gi - 9 : gi;
      if (own && gi >= 4 && gi < 9) {
        const int s0 = own_slot(54 + gi);
        gemm_tile<1, 4, 2, 0>(p, ws, call, lds, s0, s0, xcd, 0, 0);
      } else if (k2 >= 0 && k2 < 12) {
        const int s0 = out_slot(0, 50 + (k2 >> 1));
        gemm_tile<1, 2, D2, PR2>(p, ws, call, lds, s0, s0, xcd, k2 & 1, 50 + (k2 >> 1));
      } else if (!own && gi < 24) {
        const int s0 = mate_slot((gi - 12) >> 2, 15);
        gemm_tile<1, 1, D1, PR1>(p, ws, call, lds, s0, s0, xcd & 3, gi & 3, 0);
      }
    }
  }
}

}

extern "C" void kernel_launch(int rank, int nranks, ncclComm_t comm,
                              hipStream_t stream,
                              void* const* d_in, const int* in_sizes, int n_in,
                              void* d_out, int out_size,
                              const int* in_dtypes, int out_dtype,
                              void* const* d_out_all,
                              void* const* d_ws_all, size_t ws_bytes) {
  using namespace agk;
  if (nranks != NR || (unsigned)rank >= (unsigned)NR || n_in < 2 || in_sizes[0] != MS * KD || in_sizes[1] != KD * NS ||
      out_size != 4096 * NS || ws_bytes < (size_t)WS_USED) {
    fprintf(stderr, "kernel_launch: unexpected problem shape\n");
    return;
  }
  HIP_CHECK(hipFuncSetAttribute((const void*)ag_gemm, hipFuncAttributeMaxDynamicSharedMemorySize, LDS_BYTES));
  Params p;
  p.x = (const float*)d_in[0];
  p.w = (const float*)d_in[1];
  p.out = (float*)d_out;
  p.me = d_ws_all[rank];
  for (int i = 0; i < NR; ++i) p.ws[i] = d_ws_all[i];
  p.rank = rank;
  p.pad = 0;
  hipLaunchKernelGGL(ag_gemm, dim3(256), dim3(256), LDS_BYTES, stream, p);
  HIP_CHECK(hipGetLastError());
}
```

```cpp
#include <hip/hip_runtime.h>
#include <rccl/rccl.h>
#include <cstdio>

#define HIP_CHECK(expr) do { hipError_t _e = (expr); if (_e != hipSuccess) { \
    fprintf(stderr, "HIP error %s at %s:%d\n", hipGetErrorString(_e), __FILE__, __LINE__); \
    return; } } while (0)

namespace agk {

typedef unsigned u32x4 __attribute__((ext_vector_type(4)));
typedef float f32x4 __attribute__((ext_vector_type(4)));
typedef float f32x16 __attribute__((ext_vector_type(16)));
typedef _Float16 f16x8 __attribute__((ext_vector_type(8)));
typedef __attribute__((address_space(1))) unsigned gu32;
typedef __amdgpu_buffer_rsrc_t rsrc_t;

constexpr int NR = 8, MS = 512, NS = 512, KD = 4096;
constexpr int BLK = 32 * KD * 2;
constexpr int A_SLOTS = 72;
constexpr int A_BYTES = A_SLOTS * BLK, B_BYTES = 32 * BLK, OX_BYTES = 4 << 20;
constexpr int EX_PIECE = 7168 + 256, EX_BYTES = 8 << 20;
constexpr int B_OFF = A_BYTES, OX_OFF = B_OFF + B_BYTES, EX_OFF = OX_OFF + OX_BYTES, BUF_BYTES = EX_OFF + EX_BYTES;
static_assert(1024 * EX_PIECE <= EX_BYTES, "export buffer");
constexpr int CNT_OFF = 0;
constexpr int TMO_OFF = 32768;
constexpr int TK_OFF = 33024;
constexpr int OQ_OFF = 34816;
constexpr int FA_OFF = 65536, FLG_PAR = 65536;
constexpr int FB_OFF = FA_OFF + A_SLOTS * 512;
constexpr int SF_OFF = 262144, SF_SRC = 32768;
constexpr int CTRL_BYTES = 1 << 20;
static_assert(FB_OFF + 4096 <= FA_OFF + FLG_PAR && FA_OFF + 2 * FLG_PAR <= SF_OFF && SF_OFF + 16 * SF_SRC <= CTRL_BYTES, "control block");
constexpr int WS_USED = CTRL_BYTES + 2 * BUF_BYTES;
constexpr int OUT_BYTES = 4096 * NS * 4;
constexpr int IN_BYTES = MS * KD * 4;
constexpr int AUX_SYS = 17;
constexpr int RSRC_W3 = 0x00020000;
constexpr unsigned long long DEADLINE = 30000000ull;
constexpr int LDS_RED = 4 * 128 * 68 * 4, LDS_BYTES = LDS_RED + 64;
constexpr int LDS_WAVE = 16 * 528 + 512;

constexpr int NPUSH = 16;
constexpr int NR0 = 24;
constexpr int PWIN = 2;
constexpr int WIN = 6;
constexpr int NSINGLE = 12;
constexpr int AHEAD = 2;
constexpr int NE = 56;
constexpr int P0 = 40;
constexpr int EP0 = 60;
constexpr int EPS = 8;
constexpr int NRI = 8;
constexpr int NSI = 24;
constexpr int D2 = 4;
constexpr int D1 = 4;
constexpr int PR2 = 4;
constexpr int PR1 = 8;
constexpr int SL1 = 2;
constexpr int SL2 = 1;
constexpr int NWC = 12;
constexpr int WWIN = 16;
constexpr int NFW = 8;
constexpr int N_HOME = 352, E_ALL = 480, NP_INTRA = 384, T_AWAY = 128, T_OUT = 288, N_OUT = 112, NP_CROSS = T_OUT + N_OUT;
constexpr unsigned D_RAW = 1u << 13;
__device__ __forceinline__ bool desc_ok(unsigned d, unsigned call) { return (d >> 18) == (call & 0x3fffu); }

struct Params {
  const float* x;
  const float* w;
  float* out;
  void* me;
  void* ws[NR];
  int rank;
  int pad;
};


__device__ __forceinline__ char* uni(const void* p) {
  const unsigned long long a = (unsigned long long)p;
  const unsigned lo = __builtin_amdgcn_readfirstlane((unsigned)a), hi = __builtin_amdgcn_readfirstlane((unsigned)(a >> 32));
  return (char*)(((unsigned long long)hi << 32) | lo);
}
__device__ __forceinline__ char* lane_ws(const Params& p, int r) {
  char* a = uni(p.ws[0]);
#pragma unroll
  for (int i = 1; i < NR; ++i) { char* b = uni(p.ws[i]); a = r == i ? b : a; }
  return a;
}
__device__ __forceinline__ rsrc_t mk(const void* p, int bytes) {
  return __builtin_amdgcn_make_buffer_rsrc((void*)uni(p), 0, bytes, RSRC_W3);
}
__device__ __forceinline__ unsigned ld_sys(const void* base, int off) {
  return __hip_atomic_load((gu32*)((char*)base + off), __ATOMIC_RELAXED, __HIP_MEMORY_SCOPE_SYSTEM);
}
__device__ __forceinline__ void st_sys(void* base, int off, unsigned v) {
  __hip_atomic_store((gu32*)((char*)base + off), v, __ATOMIC_RELAXED, __HIP_MEMORY_SCOPE_SYSTEM);
}
__device__ __forceinline__ unsigned ld_agent(const void* base, int off) {
  return __hip_atomic_load((gu32*)((char*)base + off), __ATOMIC_RELAXED, __HIP_MEMORY_SCOPE_AGENT);
}
__device__ __forceinline__ void st_agent(void* base, int off, unsigned v) {
  __hip_atomic_store((gu32*)((char*)base + off), v, __ATOMIC_RELAXED, __HIP_MEMORY_SCOPE_AGENT);
}
#define DRAIN() asm volatile("s_waitcnt vmcnt(0)" ::: "memory")

__device__ __forceinline__ u32x4 cvt8(f32x4 a, f32x4 b) {
  f16x8 h;
  h[0] = (_Float16)a[0]; h[1] = (_Float16)a[1]; h[2] = (_Float16)a[2]; h[3] = (_Float16)a[3];
  h[4] = (_Float16)b[0]; h[5] = (_Float16)b[1]; h[6] = (_Float16)b[2]; h[7] = (_Float16)b[3];
  return __builtin_bit_cast(u32x4, h);
}

__device__ __forceinline__ float silu(float y) {
  const float t = fminf(fmaxf(y, -60.f), 60.f);
  return y * __builtin_amdgcn_rcpf(1.f + __expf(-t));
}

__device__ __forceinline__ int slot_row(int rank, int slot) {
  const int grp = rank >> 2, j = rank & 3;
  int src, b;
  if (slot < 16) { src = rank; b = slot; }
  else if (slot < 52) { const int r = (slot - 16) / 12; src = grp * 4 + ((j + 1 + r) & 3); b = slot - 16 - r * 12; b = b == 11 ? 15 : b; }
  else { const int i = (slot - 52) / 5; src = (grp ^ 1) * 4 + i; b = 11 + slot - 52 - i * 5; }
  return src * MS + b * 32;
}
__device__ __forceinline__ int mate_slot(int r, int b) { return 16 + r * 12 + (b == 15 ? 11 : b); }
__device__ __forceinline__ int away_slot(int i, int b) { return 52 + i * 5 + (b - 11); }
__device__ __forceinline__ int out_slot(int ow, int ob) {
  if (ob < 11) return ob;
  if (ob < 26) { const int r = (ob - 11) / 5; return mate_slot(r, ob - 11 - r * 5); }
  if (ob < 38) return away_slot((ow + 1 + ((ob - 26) >> 2)) & 3, 11 + ((ob - 26) & 3));
  if (ob < 50) return mate_slot((ob - 38) >> 2, 5 + ((ob - 38) & 3));
  return mate_slot((ob - 50) >> 1, 9 + (ob & 1));
}
__device__ __forceinline__ int own_slot(int i) {
  if (i < 16) return i;
  if (i < 31) return out_slot(0, i - 5);
  if (i < 47) return away_slot((i - 31) >> 2, 11 + ((i - 31) & 3));
  if (i < 59) return out_slot(0, i - 9);
  if (i < 63) return away_slot(i - 59, 15);
  if (i < 69) return out_slot(0, i - 13);
  return mate_slot(i - 69, 15);
}
__device__ __forceinline__ int sf_pos(int t) {
  return t < NPUSH + NE ? 16384 + (t >> 1) * 128 + (t & 1) * 4 : (t >> 2) * 128 + (t & 3) * 4;
}
__device__ __forceinline__ int a_flag(int slot, int u) { return FA_OFF + slot * 512 + (u & 3) * 128 + (u >> 2) * 4; }
__device__ __forceinline__ int b_flag(int qslot, int h) { return FB_OFF + qslot * 512 + ((h >> 1) & 3) * 128 + (h & 1) * 64 + (h >> 3) * 4; }

__device__ __forceinline__ void x_piece(int e, int& b, int& j) {
  if (e < 160) { j = e / 5; b = e - j * 5; }
  else if (e < 288) { b = 5 + (e & 3); j = (e - 160) >> 2; }
  else if (e < N_HOME) { b = 9 + (e & 1); j = (e - 288) >> 1; }
  else if (e < E_ALL) { b = 11 + (e & 3); j = (e - N_HOME) >> 2; }
  else { b = 15; j = e - E_ALL; }
}
__device__ __forceinline__ void piece_load(f32x4 (&v)[16], const Params& p, bool isx, int lane, int b, int j, int q, int h) {
  static_assert(KD * 4 == 8 * NS * 4, "same lane offsets");
  const rsrc_t r = mk(isx ? p.x : p.w, IN_BYTES);
  const int voff = (lane >> 5) * (KD * 4) + (lane & 31) * 16;
  const int soff = isx ? b * (32 * KD * 4) + j * 512 : h * (32 * NS * 4) + q * 512;
  const int s1 = isx ? 2 * KD * 4 : NS * 4, s8 = isx ? 16 * KD * 4 : 16 * NS * 4;
#pragma unroll
  for (int i = 0; i < 16; ++i)
    v[i] = __builtin_bit_cast(f32x4, __builtin_amdgcn_raw_buffer_load_b128(r, voff, soff + (i >> 3) * s8 + (i & 7) * s1, 0));
  __builtin_amdgcn_sched_barrier(0);
}
__device__ __forceinline__ unsigned scale_piece(f32x4 (&v)[16]) {
  float m = 0.f;
#pragma unroll
  for (int i = 0; i < 16; ++i)
    m = fmaxf(fmaxf(m, fmaxf(fabsf(v[i][0]), fabsf(v[i][1]))), fmaxf(fabsf(v[i][2]), fabsf(v[i][3])));
  unsigned u = __builtin_bit_cast(unsigned, m);
  u = max(u, (unsigned)__builtin_amdgcn_update_dpp(0u, u, 0x111, 0xf, 0xf, false));
  u = max(u, (unsigned)__builtin_amdgcn_update_dpp(0u, u, 0x112, 0xf, 0xf, false));
  u = max(u, (unsigned)__builtin_amdgcn_update_dpp(0u, u, 0x114, 0xf, 0xf, false));
  u = max(u, (unsigned)__builtin_amdgcn_update_dpp(0u, u, 0x118, 0xf, 0xf, false));
  u = max(u, (unsigned)__builtin_amdgcn_update_dpp(0u, u, 0x142, 0xa, 0xf, false));
  u = max(u, (unsigned)__builtin_amdgcn_update_dpp(0u, u, 0x143, 0xc, 0xf, false));
  const unsigned top = __builtin_amdgcn_readlane(u, 63);
  const int e = top >> 23;
  if (top == 0 || e == 255 || (e >= 117 && e <= 141)) return 0;
  const int z = min((141 - e) >> 4, 7);
#pragma unroll
  for (int i = 0; i < 16; ++i)
#pragma unroll
    for (int k = 0; k < 4; ++k) v[i][k] = __builtin_amdgcn_ldexpf(v[i][k], 16 * z);
  return z & 15;
}
__device__ __forceinline__ void x_frags(u32x4 (&f)[8], const f32x4 (&v)[16], char* lw, int lane) {
  typedef _Float16 f16x4 __attribute__((ext_vector_type(4)));
  char* wp = lw + ((lane & 31) >> 1) * 528 + (lane >> 5) * 16 + (lane & 1) * 8;
#pragma unroll
  for (int i = 0; i < 16; ++i) {
    f16x4 h = {(_Float16)v[i][0], (_Float16)v[i][1], (_Float16)v[i][2], (_Float16)v[i][3]};
    *(f16x4*)(wp + i * 32) = h;
  }
  asm volatile("" ::: "memory");
  const char* rp = lw + (lane >> 5) * 528 + (lane & 31) * 16;
#pragma unroll
  for (int s = 0; s < 8; ++s) f[s] = *(const u32x4*)(rp + s * 1056);
  asm volatile("" ::: "memory");
}
__device__ __forceinline__ void w_frags(u32x4 (&f)[8], const f32x4 (&v)[16]) {
#pragma unroll
  for (int s = 0; s < 2; ++s)
#pragma unroll
    for (int e = 0; e < 4; ++e) {
      f32x4 a = {v[s * 8][e], v[s * 8 + 1][e], v[s * 8 + 2][e], v[s * 8 + 3][e]};
      f32x4 c = {v[s * 8 + 4][e], v[s * 8 + 5][e], v[s * 8 + 6][e], v[s * 8 + 7][e]};
      f[s * 4 + e] = cvt8(a, c);
    }
}
__device__ __forceinline__ int w_lane_off(int lane) { return ((lane & 31) >> 3) * BLK + ((lane >> 5) * 32 + (lane & 7)) * 16; }

__device__ __forceinline__ unsigned pack_piece(const u32x4 (&f)[8], rsrc_t rl, int exoff, char* lw, int lane) {
  typedef unsigned short u16x2 __attribute__((ext_vector_type(2)));
  unsigned* list = (unsigned*)(lw + 16 * 528);
  list[lane] = 0;
  unsigned se = 0;
#pragma unroll
  for (int d = 0; d < 4; ++d) se += ((f[0][d] >> 10) & 0x001f001fu) + ((f[4][d] >> 10) & 0x001f001fu);
  se = (se & 0xffffu) + (se >> 16);
  se += __builtin_amdgcn_update_dpp(0u, se, 0x111, 0xf, 0xf, false);
  se += __builtin_amdgcn_update_dpp(0u, se, 0x112, 0xf, 0xf, false);
  se += __builtin_amdgcn_update_dpp(0u, se, 0x114, 0xf, 0xf, false);
  se += __builtin_amdgcn_update_dpp(0u, se, 0x118, 0xf, 0xf, false);
  se += __builtin_amdgcn_update_dpp(0u, se, 0x142, 0xa, 0xf, false);
  se += __builtin_amdgcn_update_dpp(0u, se, 0x143, 0xc, 0xf, false);
  int base = (int)(((unsigned)__builtin_amdgcn_readlane(se, 63) + 1843 + 1023) >> 10) - 7;
  base = base < 0 ? 0 : base > 24 ? 24 : base;
  const unsigned b2 = (unsigned)base * 0x04000400u;
  unsigned pk[28], esc[8];
#pragma unroll
  for (int g = 0; g < 4; ++g) {
    unsigned t[8], wd[8];
#pragma unroll
    for (int i = 0; i < 8; ++i) {
      wd[i] = f[2 * g + (i >> 2)][i & 3];
      t[i] = __builtin_bit_cast(unsigned, (u16x2)(__builtin_bit_cast(u16x2, wd[i]) - __builtin_bit_cast(u16x2, b2)));
    }
    const unsigned z = (t[7] & 0x1fff1fffu) | ((wd[7] >> 2) & 0x20002000u);
#pragma unroll
    for (int i = 0; i < 7; ++i) pk[7 * g + i] = (t[i] & 0x1fff1fffu) | (wd[i] & 0x80008000u) | ((z << (13 - 2 * i)) & 0x60006000u);
#pragma unroll
    for (int u = 0; u < 2; ++u) {
      unsigned a = 0;
#pragma unroll
      for (int d = 0; d < 4; ++d) a |= ((t[4 * u + d] ^ (wd[4 * u + d] & 0x80008000u)) >> (4 * d)) & (0xe000e000u >> (4 * d));
      esc[2 * g + u] = a;
    }
  }
#pragma unroll
  for (int k = 0; k < 7; ++k)
    __builtin_amdgcn_raw_buffer_store_b128(u32x4{pk[4 * k], pk[4 * k + 1], pk[4 * k + 2], pk[4 * k + 3]}, rl,
                                           exoff + k * 1024 + lane * 16, 0, AUX_SYS);
  int cnt = 0;
  asm volatile("" ::: "memory");
#pragma unroll
  for (int u = 0; u < 8; ++u) {
    unsigned a = esc[u];
    unsigned long long m = __ballot(a != 0);
    do {
      const unsigned sh = __builtin_ctz(a | 0x80000000u) & 28u;
      const unsigned q = (a >> (sh + 1)) & 7u;
      const unsigned e = ((u * 512 + lane * 8 + 6) << 4) + (sh & 16u) - ((sh & 12u) << 3) + q;
      unsigned at = __builtin_amdgcn_mbcnt_hi((unsigned)(m >> 32), __builtin_amdgcn_mbcnt_lo((unsigned)m, cnt));
      at = (a != 0 && at < 128u) ? at : 128u + lane;
      ((unsigned short*)list)[at] = (unsigned short)e;
      a &= ~(0xfu << sh);
      cnt += __builtin_popcountll(m);
      m = __ballot(a != 0);
    } while (m != 0);
  }
  asm volatile("" ::: "memory");
  __builtin_amdgcn_raw_buffer_store_b32(list[lane], rl, exoff + 7168 + lane * 4, 0, AUX_SYS);
  asm volatile("" ::: "memory");
  return cnt > 128 ? D_RAW : (unsigned)(cnt << 5 | base);
}
__device__ __forceinline__ void unpack_piece(u32x4 (&r)[8], unsigned desc, unsigned patch, char* img, int lane) {
  const unsigned b2 = (desc & 31u) * 0x04000400u;
  unsigned pk[28];
#pragma unroll
  for (int i = 0; i < 28; ++i) pk[i] = r[i >> 2][i & 3];
#pragma unroll
  for (int g = 0; g < 4; ++g) {
    unsigned o[8], z = 0;
#pragma unroll
    for (int i = 0; i < 7; ++i) {
      o[i] = (pk[7 * g + i] & 0x9fff9fffu) + b2;
      z |= (pk[7 * g + i] >> (13 - 2 * i)) & (0x00030003u << (2 * i));
    }
    o[7] = ((z & 0x1fff1fffu) | ((z << 2) & 0x80008000u)) + b2;
    *(u32x4*)(img + g * 2048 + lane * 16) = u32x4{o[0], o[1], o[2], o[3]};
    *(u32x4*)(img + g * 2048 + 1024 + lane * 16) = u32x4{o[4], o[5], o[6], o[7]};
  }
  asm volatile("" ::: "memory");
#pragma unroll
  for (int k = 0; k < 2; ++k) {
    const unsigned e = k ? patch >> 16 : patch & 0xffffu;
    const unsigned hi = e << 29;
    if ((e & 7u) != 0)
      __hip_atomic_fetch_add((unsigned*)(img + (e >> 5) * 4), (e & 16u) ? hi : (unsigned)((int)hi >> 16),
                             __ATOMIC_RELAXED, __HIP_MEMORY_SCOPE_WORKGROUP);
  }
  asm volatile("" ::: "memory");
#pragma unroll
  for (int i = 0; i < 8; ++i) r[i] = *(const u32x4*)(img + i * 1024 + lane * 16);
  asm volatile("" ::: "memory");
}

__device__ __forceinline__ void convert(const Params& p, char* ws, char* lds, int e, bool agg, int w, int lane, unsigned& call, int round) {
  const int grp = p.rank >> 2, jme = p.rank & 3;
  const bool isx = e < 512, home = e < N_HOME;
  f32x4 v[16];
  int b = 0, j = 0;
  const int q = (e >> 7) & 3, h = e & 127;
  const unsigned cnt0 = ld_agent(ws, CNT_OFF + blockIdx.x * 128 + w * 4);
  if (isx) x_piece(e, b, j);
  piece_load(v, p, isx, lane, b, j, q, h);
  call = __builtin_amdgcn_readfirstlane(cnt0) + 1;
  const int t = home ? e : isx ? e - N_HOME + T_AWAY : h;
  const int d = lane >> 2, nd = home ? 3 : !isx ? 1 : e < E_ALL ? 4 : 7;
  const bool tomate = home || d >= 4;
  char* const sfp = lane_ws(p, tomate ? grp * 4 + ((jme + 1 + d) & 3) : (grp ^ 1) * 4 + (isx ? d : q)) +
                    SF_OFF + ((call & 1) * 8 + p.rank) * SF_SRC + sf_pos((tomate && !home ? t + 96 : t) + (lane & 3));
  const int base = CTRL_BYTES + (call & 1) * BUF_BYTES;
  const rsrc_t rl = mk(ws, WS_USED);
  char* lw = lds + w * LDS_WAVE;
  unsigned sc = 0;
  u32x4 f[8];
  for (;;) {
    if (isx) {
      x_frags(f, v, lw, lane);
      const int o = base + b * BLK + j * 8192 + lane * 16;
#pragma unroll
      for (int i = 0; i < 8; ++i) __builtin_amdgcn_raw_buffer_store_b128(f[i], rl, o + i * 1024, 0, AUX_SYS);
    } else {
      w_frags(f, v);
      const int o = base + B_OFF + q * 4 * BLK + w_lane_off(lane) + h * 2048;
#pragma unroll
      for (int i = 0; i < 8; ++i) __builtin_amdgcn_raw_buffer_store_b128(f[i], rl, o + (i >> 2) * 1024 + (i & 3) * 128, 0, AUX_SYS);
    }
    __builtin_amdgcn_sched_barrier(0);
    if (sc != 0 || (sc = scale_piece(v)) == 0) break;
    DRAIN();
  }
  unsigned desc = D_RAW;
  if (t >= NPUSH + NR0) desc = pack_piece(f, rl, base + EX_OFF + e * EX_PIECE, lw, lane);
  DRAIN();
  if (!agg) {
    if (d < nd && (lane & 3) == 0) st_sys(sfp, 0, call << 18 | sc << 14 | desc);
  } else if (t >= NPUSH) {
    unsigned* dl = (unsigned*)(lds + LDS_RED) + round * 4;
    if (lane == 0) dl[w] = call << 18 | sc << 14 | desc;
    __syncthreads();
    if (w == 0 && d < nd) st_sys(sfp, 0, dl[lane & 3]);
  }
  if (lane < (isx ? 2 : 1)) st_sys(ws, (call & 1) * FLG_PAR + (isx ? a_flag(b, 2 * j + lane) : b_flag(q, h)), call << 4 | sc);
}

__device__ __forceinline__ unsigned push_first(const Params& p, char* ws, char* lds, int link, int t, int w, int lane) {
  const int peer = p.rank ^ (link + 1), jme = p.rank & 3, jp = peer & 3;
  const bool isx = link < 3;
  const unsigned cnt0 = ld_agent(ws, CNT_OFF + blockIdx.x * 128 + w * 4);
  f32x4 v[16];
  int b = 0, j = 0;
  if (isx) { j = t / 5; b = t - j * 5; }
  piece_load(v, p, isx, lane, b, j, jp, t);
  const unsigned call = __builtin_amdgcn_readfirstlane(cnt0) + 1;
  const int base = CTRL_BYTES + (call & 1) * BUF_BYTES;
  const rsrc_t rp = mk(p.ws[peer], WS_USED);
  const int slot = mate_slot((jme - jp - 1) & 3, b);
  const int o0 = isx ? base + slot * BLK + j * 8192 + lane * 16
                     : base + B_OFF + (4 + jme) * 4 * BLK + w_lane_off(lane) + t * 2048;
  unsigned sc = 0;
  for (;;) {
    u32x4 f[8];
    if (isx) x_frags(f, v, lds, lane); else w_frags(f, v);
#pragma unroll
    for (int i = 0; i < 8; ++i) {
      if (i > PWIN) asm volatile("s_waitcnt vmcnt(%0)" :: "i"(PWIN) : "memory");
      __builtin_amdgcn_raw_buffer_store_b128(f[i], rp, o0 + (isx ? i * 1024 : (i >> 2) * 1024 + (i & 3) * 128), 0, AUX_SYS);
    }
    __builtin_amdgcn_sched_barrier(0);
    if (sc != 0 || (sc = scale_piece(v)) == 0) break;
    DRAIN();
  }
  DRAIN();
  if (lane < (isx ? 2 : 1))
    st_sys(p.ws[peer], (call & 1) * FLG_PAR + (isx ? a_flag(slot, 2 * j + lane) : b_flag(4 + jme, t)), call << 4 | sc);
  return call;
}

struct Piece {
  int src, ex, dst;
  int kind;
  int fa, fb;
};
__device__ __forceinline__ Piece in_piece(int rank, int peer, int t, int base) {
  Piece c;
  const int jme = rank & 3, jp = peer & 3;
  const bool intra = (peer >> 2) == (rank >> 2);
  if (intra || t >= T_AWAY) {
    int b, j;
    c.ex = intra ? (t < N_HOME ? t : t + (E_ALL - N_HOME)) : t - T_AWAY + N_HOME;
    x_piece(c.ex, b, j);
    const int slot = intra ? mate_slot((jp - jme - 1) & 3, b) : away_slot(jp, b);
    c.src = base + b * BLK + j * 8192; c.dst = base + slot * BLK + j * 8192; c.kind = 0;
    c.fa = a_flag(slot, 2 * j); c.fb = a_flag(slot, 2 * j + 1);
  } else {
    c.src = base + B_OFF + jme * 4 * BLK + t * 2048; c.ex = 512 + jme * 128 + t;
    c.dst = base + B_OFF + (4 + jp) * 4 * BLK + t * 2048; c.kind = 1;
    c.fa = c.fb = b_flag(4 + jp, t);
  }
  c.ex = base + EX_OFF + c.ex * EX_PIECE;
  return c;
}
__device__ __forceinline__ Piece out_piece(int rank, int peer, unsigned desc, int base) {
  Piece c;
  const int jme = rank & 3, jp = peer & 3;
  const int o = min((int)(desc & 127u), N_OUT - 1);
  c.ex = c.src = base + OX_OFF + jme * (1 << 20) + o * 8192;
  c.kind = o < 100 ? 2 : 3;
  c.dst = (slot_row(peer, out_slot(jme, o >> 1)) * NS + jp * 128 + (o & 1) * (o < 100 ? 16 * NS : 64)) * 4;
  c.fa = c.fb = 0;
  return c;
}

constexpr int NSLOT = 8, SLOT_B = 8192 + 256, F_CTL = NSLOT * SLOT_B;
enum { C_QN = 0, C_FIN = 1, C_ISSUED = 2, C_LANDED = 3, C_DONE = 4, C_ENT = 16 };
constexpr int OOB = 0x7ff00000;
__device__ __forceinline__ unsigned lds_ld(const unsigned* a) {
  return __builtin_amdgcn_readfirstlane(__hip_atomic_load(a, __ATOMIC_ACQUIRE, __HIP_MEMORY_SCOPE_WORKGROUP));
}
__device__ __forceinline__ void lds_st(unsigned* a, unsigned v) {
  asm volatile("" ::: "memory");
  __hip_atomic_store(a, v, __ATOMIC_RELEASE, __HIP_MEMORY_SCOPE_WORKGROUP);
  asm volatile("" ::: "memory");
}

__device__ __forceinline__ void fetch_schedule(const Params& p, char* ws, unsigned* ctl, unsigned call, int link, int lane) {
  const int rank = __builtin_amdgcn_readfirstlane(p.rank);
  const int peer = rank ^ (link + 1);
  const int np = link < 3 ? NP_INTRA : NP_CROSS;
  const rsrc_t rl = mk(ws, WS_USED);
  const int base = CTRL_BYTES + (call & 1) * BUF_BYTES;
  const int sf = SF_OFF + ((call & 1) * 8 + peer) * SF_SRC;
  const int nsg = link < 3 ? NSI : NSINGLE, v2 = (np - NPUSH - nsg) >> 1, nv = v2 + nsg;
  const unsigned tkbase = (call - 1) * (unsigned)nv - (unsigned)NFW;
  int tko = TK_OFF + link * 128;
  asm volatile("" : "+v"(tko));
  unsigned tk = 0;
  int v = blockIdx.x & 7, n = 0;
  bool fail = false;
  while ((unsigned)v < (unsigned)nv && !fail) {
    const int k = v < v2 ? 2 : 1, tf = NPUSH + v + min(v, v2);
    const int sfo = sf + sf_pos(tf + (lane & (k - 1)));
    while ((int)lds_ld(ctl + C_ISSUED) + AHEAD <= n || (int)lds_ld(ctl + C_DONE + (n & 1)) + NSLOT <= n ||
           (int)lds_ld(ctl + C_DONE + (~n & 1)) + NSLOT <= n + 1) __builtin_amdgcn_s_sleep(1);
    if (lane == 0) tk = __builtin_amdgcn_raw_ptr_buffer_atomic_add_i32(1, rl, tko, 0, 0);
    unsigned dv = ld_sys(ws, sfo);
    for (int i = 0; i < k; ++i) {
      const int t = tf + i;
      const bool isout = link >= 3 && t >= T_OUT;
      Piece c = in_piece(rank, peer, isout ? 0 : t, base);
      asm volatile("" :: "s"(c.src), "s"(c.ex), "s"(c.dst), "s"(c.fa), "s"(c.fb));
      unsigned dc = __builtin_amdgcn_readlane(dv, i);
      if (!desc_ok(dc, call)) {
        const unsigned long long ts = __builtin_amdgcn_s_memrealtime();
        for (unsigned it = 1;; ++it) {
          dv = ld_sys(ws, sfo);
          dc = __builtin_amdgcn_readlane(dv, i);
          if (desc_ok(dc, call)) break;
          if ((it & 15) == 0 && (__builtin_amdgcn_readfirstlane(ld_agent(ws, TMO_OFF)) == call ||
                                 __builtin_amdgcn_s_memrealtime() - ts > DEADLINE)) { fail = true; break; }
          __builtin_amdgcn_s_sleep(1);
        }
        if (fail) break;
      }
      if (link < 3 && t >= N_HOME - NRI && t < N_HOME) dc |= D_RAW;
      if (isout) c = out_piece(rank, peer, dc, base);
      const bool wide = link >= 3 && t >= NP_CROSS - NWC;
      unsigned* e = ctl + C_ENT + (n & (NSLOT - 1)) * 8;
      *(u32x4*)e = u32x4{(unsigned)((dc & D_RAW) ? c.src : c.ex), dc, (unsigned)(c.kind | (wide ? 4 : 0)), (unsigned)c.dst};
      *(u32x4*)(e + 4) = u32x4{(unsigned)c.fa, (unsigned)c.fb, (unsigned)t, 0u};
      lds_st(ctl + C_QN, ++n);
    }
    v = (int)(__builtin_amdgcn_readfirstlane(tk) - tkbase);
  }
  lds_st(ctl + C_FIN, n + 1);
  if (fail) {
    st_agent(ws, TMO_OFF, call);
    if (lane == 0) p.out[rank * MS * NS] = __builtin_nanf("");
  }
}

#define DMA(W, op, voff, ldsaddr) do { unsigned keep_; \
  asm volatile("s_mov_b32 %0, m0\n\ts_mov_b32 m0, %2\n\ts_waitcnt vmcnt(%4)\n\t" op " %1, %3, 0 offen sc0 sc1 lds\n\t" \
               "s_mov_b32 m0, %0" : "=&s"(keep_) : "v"(voff), "s"(ldsaddr), "s"(rp), "i"((W) - 1) : "memory"); } while (0)
__device__ __forceinline__ void fetch_load(const Params& p, char* lds, unsigned* ctl, int link, int lane) {
  const rsrc_t rp = mk(p.ws[p.rank ^ (link + 1)], WS_USED);
  const unsigned ring = (unsigned)(unsigned long long)(__attribute__((address_space(3))) char*)lds;
  const int lx = lane * 16, lw = w_lane_off(lane);
  unsigned q = 0;
  u32x4 e = {0, 0, 0, 0};
  int pub = 0;
  for (int n = 0;; ++n) {
    if ((int)__builtin_amdgcn_readfirstlane(q) <= n) {
      if ((int)lds_ld(ctl + C_QN) <= n) {
        if (pub < n - 1) {
          asm volatile("s_waitcnt vmcnt(8)" ::: "memory");
          lds_st(ctl + C_LANDED, n - 1);
        }
        DRAIN();
        pub = n;
        lds_st(ctl + C_LANDED, n);
        bool end = false;
        while ((int)lds_ld(ctl + C_QN) <= n) {
          if ((int)lds_ld(ctl + C_FIN) == n + 1) { end = true; break; }
          __builtin_amdgcn_s_sleep(1);
        }
        if (end) break;
      }
      asm volatile("" ::: "memory");
      e = *(const u32x4*)(ctl + C_ENT + (n & (NSLOT - 1)) * 8);
    }
    lds_st(ctl + C_ISSUED, n + 1);
    const unsigned dc = __builtin_amdgcn_readfirstlane(e[1]);
    asm volatile("" :: "v"(e));
    const unsigned kind = __builtin_amdgcn_readfirstlane(e[2]);
    const bool raw = dc & D_RAW, wraw = raw && (kind & 3) == 1;
    const int src = __builtin_amdgcn_readfirstlane(e[0]);
    const int a0 = (wraw ? lw : lx) + src, st = wraw ? 128 : 1024, hb = wraw ? 1024 : 4096;
    const unsigned slot = ring + (n & (NSLOT - 1)) * SLOT_B;
#define PIECE(W) _Pragma("unroll") for (int j = 0; j < 8; ++j) { \
      if (j < 7 || raw) DMA(W, "buffer_load_dwordx4", a0 + (j >> 2) * hb + (j & 3) * st, slot + j * 1024); \
      else DMA(W, "buffer_load_dword", (lane == 0 || 2 * lane < (int)((dc >> 5) & 255u)) ? src + 7168 + lane * 4 : OOB, slot + 8192); \
      if (j == ((W) - 1) % 8) {                      \
        const int l = n - ((W) - 1) / 8; \
        if (l > pub) { pub = l; lds_st(ctl + C_LANDED, l); } \
      } \
      if (j == 4) {                                  \
        q = __hip_atomic_load(ctl + C_QN, __ATOMIC_RELAXED, __HIP_MEMORY_SCOPE_WORKGROUP); \
        asm volatile("" ::: "memory"); \
        e = *(const u32x4*)(ctl + C_ENT + ((n + 1) & (NSLOT - 1)) * 8); \
      } }
    if (kind & 4) { PIECE(WWIN) } else { PIECE(WIN) }
#undef PIECE
  }
}

__device__ __forceinline__ void fetch_work(const Params& p, char* ws, char* lds, unsigned* ctl, unsigned call, int link, int k, int lane) {
  const rsrc_t rl = mk(ws, WS_USED), ro = mk(p.out, OUT_BYTES);
  const int lx = lane * 16, lw = w_lane_off(lane), lo = (lane >> 5) * (NS * 4) + (lane & 31) * 16,
            lo3 = (lane >> 4) * (NS * 4) + (lane & 15) * 16;
  for (int n = k;; n += 2) {
    bool end = false;
    while ((int)lds_ld(ctl + C_LANDED) <= n) {
      const int fin = (int)lds_ld(ctl + C_FIN);
      if (fin != 0 && fin - 1 <= n) { end = true; break; }
      __builtin_amdgcn_s_sleep(1);
    }
    if (end) break;
    asm volatile("" ::: "memory");
    char* img = lds + (n & (NSLOT - 1)) * SLOT_B;
    const u32x4 e0 = *(const u32x4*)(ctl + C_ENT + (n & (NSLOT - 1)) * 8), e1 = *(const u32x4*)(ctl + C_ENT + (n & (NSLOT - 1)) * 8 + 4);
    u32x4 r[8];
#pragma unroll
    for (int i = 0; i < 8; ++i) r[i] = *(const u32x4*)(img + i * 1024 + lane * 16);
    const unsigned pt = *(const unsigned*)(img + 8192 + lane * 4);
    asm volatile("" ::: "memory");
    const unsigned dc = __builtin_amdgcn_readfirstlane(e0[1]);
    const int kind = __builtin_amdgcn_readfirstlane(e0[2]) & 3;
    if (!(dc & D_RAW)) unpack_piece(r, dc, 2 * lane < (int)((dc >> 5) & 255u) ? pt : 0u, img, lane);
    const int d0 = (kind == 0 ? lx : kind == 1 ? lw : kind == 2 ? lo : lo3) + (int)e0[3];
    const int dst_st = kind == 0 ? 1024 : kind == 1 ? 128 : kind == 2 ? 2 * NS * 4 : 4 * NS * 4, dst_hb = kind == 1 ? 1024 : 4 * dst_st;
    const rsrc_t rd = kind >= 2 ? ro : rl;
#pragma unroll
    for (int i = 0; i < 8; ++i)
      __builtin_amdgcn_raw_buffer_store_b128(r[i], rd, d0 + (i >> 2) * dst_hb + (i & 3) * dst_st, 0, AUX_SYS);
    lds_st(ctl + C_DONE + k, n + 1);
    DRAIN();
    if (lane < 2 && kind < 2) st_sys(ws, (call & 1) * FLG_PAR + (lane ? e1[1] : e1[0]), call << 4 | ((dc >> 14) & 15u));
  }
}

struct Ready { int na, nb; unsigned f; };
template <int SLEEP, bool ISA, int SLEEP0 = 16>
__device__ __forceinline__ void wait_ready(void* ws, int off, unsigned call, int need, Ready& r) {
  unsigned long long t0 = __builtin_amdgcn_s_memrealtime();
  for (unsigned it = 1;; ++it) {
    r.f = ld_sys(ws, off);
    const unsigned long long m = __ballot(r.f >= call << 4);
    r.na = __builtin_ctz(~((unsigned)(m & (m >> 16)) & 0xffffu));
    r.nb = __builtin_ctz(~((unsigned)((m >> 32) & (m >> 48)) & 0xffffu));
    if ((ISA ? r.na : r.nb) > need) break;
    if ((it & 15) == 0 && (ld_agent(ws, TMO_OFF) == call || __builtin_amdgcn_s_memrealtime() - t0 > DEADLINE)) {
      st_agent(ws, TMO_OFF, call);
      r.na = r.nb = 17;
      break;
    }
    if (need == 0) __builtin_amdgcn_s_sleep(SLEEP0); else if (need < 12) __builtin_amdgcn_s_sleep(SLEEP); else __builtin_amdgcn_s_sleep(SL2);
  }
}
__device__ __forceinline__ int scale_of(unsigned f, int lane) { return (int)((unsigned)__builtin_amdgcn_readlane(f, lane) << 28) >> 24; }

#define MMA(B, A, C) __builtin_amdgcn_mfma_f32_32x32x16_f16(__builtin_bit_cast(f16x8, B), __builtin_bit_cast(f16x8, A), C, 0, 0, 0)
template <int NB>
__device__ __forceinline__ void unscale_partials(rsrc_t rs, unsigned f, float* dst, int mb, int a0, int a1, int b0, int lane) {
  constexpr int LD = NB * 32 + 4;
  const unsigned lead = __shfl(f, lane < 32 ? lane & 16 : 32);
  const bool uniform = __ballot(((f ^ lead) & 15u) != 0) == 0;
#pragma clang loop unroll(disable)
  for (int mn = 0; mn < mb * NB; ++mn) {
    {
      const int m = mn / NB, n = mn % NB;
      float* d = dst + m * 32 * LD + n * 32;
      if (uniform) {
        const int sh = -scale_of(f, 16 * m) - scale_of(f, 32);
        for (int i = 0; i < 16; ++i) d[i] = __builtin_amdgcn_ldexpf(d[i], sh);
      } else {
        f32x16 acc = {0, 0, 0, 0, 0, 0, 0, 0, 0, 0, 0, 0, 0, 0, 0, 0};
#pragma clang loop unroll(disable)
        for (int kc = 0; kc < 16; ++kc) {
          u32x4 a[4], b[4];
#pragma unroll
          for (int s = 0; s < 4; ++s) {
            a[s] = __builtin_amdgcn_raw_buffer_load_b128(rs, lane * 16 + s * 1024 + (m ? a1 : a0) + kc * 16384, 0, AUX_SYS);
            b[s] = __builtin_amdgcn_raw_buffer_load_b128(rs, lane * 16 + s * 1024 + b0 + n * BLK + kc * 16384, 0, AUX_SYS);
          }
#pragma unroll
          for (int hh = 0; hh < 2; ++hh) {
            const f32x16 z = {0, 0, 0, 0, 0, 0, 0, 0, 0, 0, 0, 0, 0, 0, 0, 0};
            const f32x16 t = MMA(b[2 * hh + 1], a[2 * hh + 1], MMA(b[2 * hh], a[2 * hh], z));
            const int sh = -scale_of(f, 16 * m + kc) - scale_of(f, 32 + 16 * hh + kc);
#pragma unroll
            for (int r = 0; r < 16; ++r) acc[r] += __builtin_amdgcn_ldexpf(t[r], sh);
          }
        }
#pragma unroll
        for (int r = 0; r < 16; ++r) d[(r & 3) * 4 + (r >> 2)] = acc[r];
      }
    }
  }
}

template <int MB, int NB, int D, int PRE>
__device__ __forceinline__ void gemm_tile(const Params& p, char* ws, unsigned call, float* lds, int slot0, int slot1, int qslot,
                                          int nt, int ob) {
  const int lane = threadIdx.x & 63;
  const int w = __builtin_amdgcn_readfirstlane(threadIdx.x >> 6);
  const rsrc_t rs = mk(ws, WS_USED);
  const int base = CTRL_BYTES + (call & 1) * BUF_BYTES;
  const int abase[2] = {base + slot0 * BLK + w * 4096, base + slot1 * BLK + w * 4096};
  const int bbase = base + B_OFF + (qslot * 4 + nt * NB) * BLK + w * 4096;
  const int foff = (call & 1) * FLG_PAR + (lane < 32 ? a_flag((lane & 16) ? slot1 : slot0, w) + (lane & 15) * 4
                             : FB_OFF + qslot * 512 + w * 128 + (lane & 31) * 4);
  const int voff = lane * 16;
  const bool remote = qslot >= 4;
  const rsrc_t ro = remote ? rs : mk(p.out, OUT_BYTES);
  constexpr int LPR = NB * 8;
  constexpr int RPI = 64 / LPR;
  constexpr int RPW = MB * 8;
  const int grow = slot_row(p.rank, (w * RPW) >> 5 ? slot1 : slot0) + ((w * RPW) & 31);
  int o0 = remote ? base + OX_OFF + (qslot - 4) * (1 << 20) + (2 * ob + nt) * 8192 + w * (RPW * NB * 128) + lane * 16
                        : ((grow + lane / LPR) * NS + qslot * 128 + nt * NB * 32) * 4 + (lane % LPR) * 16;
  const int ost = remote ? 1024 : RPI * NS * 4;
  char* const owner = uni(p.ws[(((p.rank >> 2) ^ 1) * 4 + qslot - 4) & 7]);
  asm volatile("" : "+v"(o0) : "s"(ro), "s"(owner));

  f32x16 acc[MB][NB];
#pragma unroll
  for (int m = 0; m < MB; ++m)
#pragma unroll
    for (int n = 0; n < NB; ++n)
#pragma unroll
      for (int r = 0; r < 16; ++r) acc[m][n][r] = 0.f;
  u32x4 fa[D][4][MB], fb[D][4][NB];

#define ISSUE_B(FB, kc, FROM_LDS) do { \
    if (FROM_LDS) { \
      _Pragma("unroll") for (int s = 0; s < 4; ++s) \
        _Pragma("unroll") for (int n = 0; n < NB; ++n) \
          FB[s][n] = *(const u32x4*)(pre + (((kc) - (16 - PRE)) * 4 * NB + s * NB + n) * 1024); \
    } else { \
      if (!PRE && rd.nb <= (kc)) wait_ready<(MB == 2 ? 8 : SL1), false>(ws, foff, call, (kc), rd); \
      asm volatile("" ::: "memory"); \
      _Pragma("unroll") for (int s = 0; s < 4; ++s) \
        _Pragma("unroll") for (int n = 0; n < NB; ++n) \
          FB[s][n] = __builtin_amdgcn_raw_buffer_load_b128(rs, voff + s * 1024, bbase + n * BLK + (kc) * 16384, AUX_SYS); \
    } \
    __builtin_amdgcn_sched_barrier(0); } while (0)
#define ISSUE_A(FA, kc) do { \
    if (rd.na <= (kc)) wait_ready<(MB == 2 ? 8 : SL1), true, (PRE ? 4 : 16)>(ws, foff, call, (kc), rd); \
    asm volatile("" ::: "memory"); \
    _Pragma("unroll") for (int s = 0; s < 4; ++s) \
      _Pragma("unroll") for (int m = 0; m < MB; ++m) \
        FA[s][m] = __builtin_amdgcn_raw_buffer_load_b128(rs, voff + s * 1024, abase[m] + (kc) * 16384, AUX_SYS); \
    __builtin_amdgcn_sched_barrier(0); } while (0)
#define ISSUE(FA, FB, kc, FROM_LDS) do { ISSUE_B(FB, kc, FROM_LDS); ISSUE_A(FA, kc); } while (0)
#define COMPUTE(FA, FB) do { \
    _Pragma("unroll") for (int s = 0; s < 4; ++s) \
      _Pragma("unroll") for (int m = 0; m < MB; ++m) \
        _Pragma("unroll") for (int n = 0; n < NB; ++n) acc[m][n] = MMA(FB[s][n], FA[s][m], acc[m][n]); \
    __builtin_amdgcn_sched_barrier(0); } while (0)

  Ready rd = {0, 0, 0u};
  constexpr int PSTR = PRE ? LDS_RED / 16 : MB * 32 * (NB * 32 + 4);
  static_assert(PRE % D == 0 && PRE * 4 * NB * 1024 <= LDS_RED / 4, "LDS region of a wave");
  char* const pre = (char*)(lds + w * PSTR) + lane * 16;
  if (PRE) {
    wait_ready<8, false>(ws, foff, call, 15, rd);
    asm volatile("" ::: "memory");
#pragma clang loop unroll(disable)
    for (int kc = 16 - PRE; kc < 16; ++kc) {
#pragma unroll
      for (int i = 0; i < 4 * NB; ++i)
        *(u32x4*)(pre + ((kc - (16 - PRE)) * 4 * NB + i) * 1024) = __builtin_amdgcn_raw_buffer_load_b128(
            rs, voff + (i / NB) * 1024, bbase + (i % NB) * BLK + kc * 16384, AUX_SYS);
    }
  }
#pragma unroll
  for (int d = 0; d < D; ++d) { ISSUE_B(fb[d], d, false); if (!PRE) ISSUE_A(fa[d], d); }
  if (PRE) {
#pragma unroll
    for (int d = 0; d < D; ++d) ISSUE_A(fa[d], d);
  }
#pragma clang loop unroll(disable)
  for (int kc = D; kc < 16 - PRE; kc += D) {
#pragma unroll
    for (int d = 0; d < D; ++d) {
      COMPUTE(fa[d], fb[d]);
      ISSUE(fa[d], fb[d], kc + d, false);
    }
  }
  if (PRE) {
#pragma clang loop unroll(disable)
    for (int kc = 16 - PRE; kc < 16; kc += D) {
#pragma unroll
      for (int d = 0; d < D; ++d) {
        COMPUTE(fa[d], fb[d]);
        ISSUE(fa[d], fb[d], kc + d, true);
      }
    }
  }
  const unsigned tmo = ld_agent(ws, TMO_OFF);
#pragma unroll
  for (int d = 0; d < D; ++d) COMPUTE(fa[d], fb[d]);
#undef ISSUE
#undef ISSUE_A
#undef ISSUE_B
#undef COMPUTE

  constexpr int LD = NB * 32 + 4;
  unsigned oq = 0;
  if (remote && threadIdx.x == 0) {
    int oo = OQ_OFF + (qslot - 4) * 128;
    asm volatile("" : "+v"(oo));
    oq = __builtin_amdgcn_raw_ptr_buffer_atomic_add_i32(MB * NB / 2, rs, oo, 0, 0);
  }
  {
    const bool bad = rd.na > 16 || tmo == call;
    float* dst = lds + w * PSTR + (lane & 31) * LD + (lane >> 5) * 16;
#pragma unroll
    for (int m = 0; m < MB; ++m)
#pragma unroll
      for (int n = 0; n < NB; ++n)
#pragma unroll
        for (int t = 0; t < 4; ++t) {
          f32x4 o = {acc[m][n][t], acc[m][n][4 + t], acc[m][n][8 + t], acc[m][n][12 + t]};
          *(f32x4*)(dst + m * 32 * LD + n * 32 + t * 4) = o;
        }
    if (__ballot((rd.f & 15u) != 0) != 0) unscale_partials<NB>(rs, rd.f, dst, MB, abase[0], abase[1], bbase, lane);
    if (bad) {
      const float q = __builtin_nanf("");
      for (int m = 0; m < MB; ++m)
        for (int i = 0; i < NB * 4; ++i) *(f32x4*)(dst + m * 32 * LD + (i >> 2) * 32 + (i & 3) * 4) = f32x4{q, q, q, q};
    }
  }
  __syncthreads();
#pragma unroll
  for (int i = 0; i < RPW / RPI; ++i) {
    const int row = w * RPW + i * RPI + lane / LPR, c4 = (lane % LPR) * 4;
    f32x4 s = *(const f32x4*)(lds + row * LD + c4);
#pragma unroll
    for (int k = 1; k < 4; ++k) s += *(const f32x4*)(lds + k * PSTR + row * LD + c4);
    f32x4 o;
#pragma unroll
    for (int e = 0; e < 4; ++e) o[e] = silu(s[e]);
    __builtin_amdgcn_raw_buffer_store_b128(__builtin_bit_cast(u32x4, o), ro, o0 + i * ost, 0, AUX_SYS);
  }
  DRAIN();
  __syncthreads();
  if (remote && w == 0) {
    const unsigned at = __builtin_amdgcn_readfirstlane(oq) - (call - 1) * (unsigned)N_OUT + lane;
    if (lane < MB * NB / 2 && at < (unsigned)N_OUT)
      st_sys(owner, SF_OFF + ((call & 1) * 8 + p.rank) * SF_SRC + sf_pos(T_OUT + at),
             call << 18 | D_RAW | (2 * ob + nt + lane));
  }
}

__global__ void __launch_bounds__(256, 1) ag_gemm(Params p) {
  extern __shared__ __attribute__((aligned(16))) float lds[];
  const int lane = threadIdx.x & 63;
  const int w = __builtin_amdgcn_readfirstlane(threadIdx.x >> 6);
  const int xcd = blockIdx.x & 7, idx = blockIdx.x >> 3;
  asm volatile("" :: "s"(p.x), "s"(p.w), "s"(p.me), "s"(p.rank));
  char* const ws = uni(p.me);

  if (idx < 7) {
    const int link = (idx + xcd) % 7;
    unsigned* ctl = (unsigned*)((char*)lds + F_CTL);
    unsigned call = 0;
    if (w == 0) {
      if (lane < 16) ctl[lane] = 0;
      asm volatile("s_waitcnt lgkmcnt(0)\n\ts_barrier" ::: "memory");
      call = __builtin_amdgcn_readfirstlane(ld_agent(ws, CNT_OFF + blockIdx.x * 128)) + 1;
      fetch_schedule(p, ws, ctl, call, link, lane);
      if (lane == 0) st_agent(ws, CNT_OFF + blockIdx.x * 128, call);
    } else if (w == 1) {
      asm volatile("s_barrier" ::: "memory");
      fetch_load(p, (char*)lds, ctl, link, lane);
    } else {
      asm volatile("s_barrier" ::: "memory");
      call = __builtin_amdgcn_readfirstlane(ld_agent(ws, CNT_OFF + blockIdx.x * 128 + w * 4)) + 1;
      if (lane == 0) st_agent(ws, CNT_OFF + blockIdx.x * 128 + w * 4, call);
      fetch_work(p, ws, (char*)lds, ctl, call, link, w - 2, lane);
    }
  } else {
    unsigned call = 0;
    {
      const int g = (idx - 7) * 8 + xcd;
      constexpr int A0 = NPUSH / 4, A1 = (NPUSH + NE) / 4, NG = 32 - NE / 4, NTASK = 5 * NG + 96;
      auto task_piece = [](int o) {
        if (o < 5 * NG) {
          int a = o / 5; const int r = o - a * 5;
          a = a < 8 ? a + A1 : a < 8 + A0 ? a - 8 : a + NE / 4;
          return r == 0 ? a * 4 : 384 + r * 128 + a * 4;
        }
        const int r = o - 5 * NG;
        return r < 80 ? ((r & 1) ? N_HOME + (r >> 1) * 4 : (32 + (r >> 1)) * 4) : (r - 8) * 4;
      };
      const int ei = g < 80 ? (w == 0 ? g : w == 2 ? g + 80 : -1) : (w == 0 && g + 80 < 5 * NE ? g + 80 : -1);
      const int pk = g - (200 - 7 * NPUSH);
      if (w == 1 && pk >= 0) call = push_first(p, ws, (char*)lds + LDS_WAVE, pk % 7, pk / 7, w, lane);
      if (ei < 0) for (int i = call ? 64 : -P0; i < g; ++i) __builtin_amdgcn_s_sleep(1);
      else for (int i = EP0; i < ei; i += EPS) __builtin_amdgcn_s_sleep(1);
      const int r1 = g >= NTASK ? 1 : g >= 400 - NTASK ? 3 : 2;
#pragma clang loop unroll(disable)
      for (int r = ei >= 0 ? 0 : 1; r < r1; ++r) {
        const int s5 = ei % 5;
        const int e = r == 0 ? (s5 == 0 ? 0 : 384 + s5 * 128) + NPUSH + ei / 5 : task_piece(r == 1 ? g : g + NTASK - 200) + w;
        convert(p, ws, (char*)lds, e, r != 0, w, lane, call, r);
      }
      if (call == 0) call = __builtin_amdgcn_readfirstlane(ld_agent(ws, CNT_OFF + blockIdx.x * 128 + w * 4)) + 1;
      if (lane == 0) st_agent(ws, CNT_OFF + blockIdx.x * 128 + w * 4, call);
    }
    {
      const int gi = idx - 7;
      const bool own = xcd < 4;
      __syncthreads();
#pragma clang loop unroll(disable)
      for (int ob = 2 * gi; ob < (own ? 58 : 50); ob += 50) {
        gemm_tile<2, 4, 2, 0>(p, ws, call, lds, own ? own_slot(ob) : out_slot(xcd - 4, ob), own ? own_slot(ob + 1) : out_slot(xcd - 4, ob + 1),
                        xcd, 0, ob);
      }
      const int k2 = own ? gi - 9 : gi;
      if (own && gi >= 4 && gi < 9) {
        const int s0 = own_slot(54 + gi);
        gemm_tile<1, 4, 2, 0>(p, ws, call, lds, s0, s0, xcd, 0, 0);
      } else if (k2 >= 0 && k2 < 12) {
        const int s0 = out_slot(0, 50 + (k2 >> 1));
        gemm_tile<1, 2, D2, PR2>(p, ws, call, lds, s0, s0, xcd, k2 & 1, 50 + (k2 >> 1));
      } else if (!own && gi < 24) {
        const int s0 = mate_slot((gi - 12) >> 2, 15);
        gemm_tile<1, 1, D1, PR1>(p, ws, call, lds, s0, s0, xcd & 3, gi & 3, 0);
      }
    }
  }
}

}

extern "C" void kernel_launch(int rank, int nranks, ncclComm_t comm,
                              hipStream_t stream,
                              void* const* d_in, const int* in_sizes, int n_in,
                              void* d_out, int out_size,
                              const int* in_dtypes, int out_dtype,
                              void* const* d_out_all,
                              void* const* d_ws_all, size_t ws_bytes) {
  using namespace agk;
  if (nranks != NR || (unsigned)rank >= (unsigned)NR || n_in < 2 || in_sizes[0] != MS * KD || in_sizes[1] != KD * NS ||
      out_size != 4096 * NS || ws_bytes < (size_t)WS_USED) {
    fprintf(stderr, "kernel_launch: unexpected problem shape\n");
    return;
  }
  HIP_CHECK(hipFuncSetAttribute((const void*)ag_gemm, hipFuncAttributeMaxDynamicSharedMemorySize, LDS_BYTES));
  Params p;
  p.x = (const float*)d_in[0];
  p.w = (const float*)d_in[1];
  p.out = (float*)d_out;
  p.me = d_ws_all[rank];
  for (int i = 0; i < NR; ++i) p.ws[i] = d_ws_all[i];
  p.rank = rank;
  p.pad = 0;
  hipLaunchKernelGGL(ag_gemm, dim3(256), dim3(256), LDS_BYTES, stream, p);
  HIP_CHECK(hipGetLastError());
}
```
